```python
import numpy as np

kernel_name = "gpu_gemm_ar_m4096_k4096_n16384_bf16_silu"


ABSMAX_RTOL = 0.008

COMPUTE_DTYPE = "bfloat16"
EPILOGUE = "silu"
IN_QUANT = None
OUT_QUANT = None
IN_DTYPES = {"x": "bfloat16", "w_mat": "bfloat16"}
WORLD_SIZE = 8

def _to_bf16(x):
    u = np.ascontiguousarray(x, dtype=np.float32).view(np.uint32)
    return ((u + 0x7FFF + ((u >> 16) & 1)) & 0xFFFF0000).view(np.float32)

def setup_inputs():

    x = _to_bf16(np.random.randn(4096, 4096).astype(np.float32))
    w_mat = _to_bf16((np.random.randn(4096, 16384) / np.sqrt(4096)).astype(np.float32))
    return {"x": x, "w_mat": w_mat}

def _epilogue(y):
    return (y / (1.0 + np.exp(-np.clip(y, -60.0, 60.0)))).astype(np.float32)

def reference(x, w_mat):
    y = (x @ w_mat).astype(np.float32)
    return _epilogue(y)
```

```cpp
#include <hip/hip_runtime.h>
#include <rccl/rccl.h>
#include <cstdio>
#include <cstdint>

#define HIP_CHECK(expr) do { hipError_t _e = (expr); if (_e != hipSuccess) { \
    fprintf(stderr, "HIP error %s at %s:%d\n", hipGetErrorString(_e), __FILE__, __LINE__); \
    return; } } while (0)

#define NRB 0
#define NPUSH 16

namespace {
#define GAS __attribute__((address_space(1)))
#define LAS __attribute__((address_space(3)))
typedef short bf16x8 __attribute__((ext_vector_type(8)));
typedef float f32x4 __attribute__((ext_vector_type(4)));
typedef unsigned u32x4 __attribute__((ext_vector_type(4)));
typedef GAS unsigned gu32;

constexpr int NR = 8, MM = 4096, KK = 4096, NN = 16384, KS = KK / NR;
constexpr int NPAN = MM / 256, NBLK = NN / 256;
constexpr int DB = (NBLK - NRB) / NR;
constexpr int NLB = DB + NRB;
constexpr int NIN = NPAN + NLB;
constexpr int NTILE = NPAN * NLB, ND = NPAN * DB;
constexpr int NWG = 256, NCOMP = NWG - NPUSH, PW = NPUSH * 8, RPU = 256 / PW;
static_assert(NRB % 8 == 0 && DB >= 1 && (PW == 128 || PW == 256), "config");
constexpr unsigned CHUNK_B = 256u << 10;
constexpr unsigned IMG_B = 2u << 20;
constexpr unsigned CTRL_B = 1u << 20;
constexpr unsigned SCR_OFF = CTRL_B + 2u * NIN * IMG_B;
constexpr size_t WS_NEED = (size_t)SCR_OFF + (size_t)NR * DB * CHUNK_B;
constexpr unsigned OUT_B = (unsigned)((size_t)MM * NN * 2);
constexpr int LINE = 32;
constexpr int C_CALL = 0;
constexpr int C_CNT = C_CALL + NWG * LINE;
constexpr int C_STAGE = C_CNT + 2 * 80 * LINE;
constexpr int C_YF = C_STAGE + 8 * LINE;
constexpr int C_DONE = C_YF + 128;
constexpr int C_TMO = C_DONE + LINE;
static_assert((C_TMO + LINE) * 4 <= (int)CTRL_B && NIN <= 80 && ND <= 128, "ctrl");
constexpr unsigned long long DEADLINE_TICKS = 50000000ull;
constexpr int HTB = 16384, STAGE_BYTES = 8 * HTB, LDS_BYTES = STAGE_BYTES + 1024;

struct Params {
    const char* x; const char* w;
    char* out_self; char* ws_self;
    char* out[NR]; char* ws[NR];
    int rank;
    unsigned uorder4[80 / 4];
    unsigned yorder4[128 / 4];
    unsigned tile2[NTILE / 2];
    __device__ __forceinline__ int uorder(int i) const { return (uorder4[i >> 2] >> ((i & 3) * 8)) & 255; }
    __device__ __forceinline__ int yorder(int i) const { return (yorder4[i >> 2] >> ((i & 3) * 8)) & 255; }
    __device__ __forceinline__ int tile(int i) const { return (tile2[i >> 1] >> ((i & 1) * 16)) & 65535; }
};

#define RLX_AGENT __ATOMIC_RELAXED, __HIP_MEMORY_SCOPE_AGENT
#define RLX_SYS __ATOMIC_RELAXED, __HIP_MEMORY_SCOPE_SYSTEM
#define WAIT_V(n) asm volatile("s_waitcnt vmcnt(" #n ")" ::: "memory")
#define WAIT_L(n) asm volatile("s_waitcnt lgkmcnt(" #n ")" ::: "memory")
#define WAIT_VC(n) do { asm volatile("" ::: "memory"); __builtin_amdgcn_s_waitcnt(0x0F70 | ((n) & 15) | (((n) >> 4) << 14)); asm volatile("" ::: "memory"); } while (0)
#define BAR __builtin_amdgcn_s_barrier()
#define SCHED __builtin_amdgcn_sched_barrier(0)

typedef GAS char gchar;
typedef const GAS u32x4 gvec;
__device__ __forceinline__ gchar* uni(const void* p) {
    const unsigned long long v = (unsigned long long)p;
    const unsigned lo = __builtin_amdgcn_readfirstlane((unsigned)v), hi = __builtin_amdgcn_readfirstlane((unsigned)(v >> 32));
    return (gchar*)(((unsigned long long)hi << 32) | lo);
}
struct Self { gchar* out; gchar* ws; int rank; unsigned call; };
__device__ __forceinline__ unsigned img_off(unsigned par, unsigned uid) { return CTRL_B + (par * NIN + uid) * IMG_B; }
template <class P> __device__ __forceinline__ __amdgpu_buffer_rsrc_t mkrsrc(P p, unsigned bytes) {
    return __builtin_amdgcn_make_buffer_rsrc((void*)p, 0, bytes, 0x00020000);
}
__device__ __forceinline__ unsigned cvt_pk_bf16(float lo, float hi) { unsigned r; asm("v_cvt_pk_bf16_f32 %0, %1, %2" : "=v"(r) : "v"(lo), "v"(hi)); return r; }
__device__ __forceinline__ float silu(float y) {
    const float c = __builtin_amdgcn_fmed3f(y, -60.0f, 60.0f);
    return y * __builtin_amdgcn_rcpf(1.0f + __builtin_amdgcn_exp2f(c * -1.44269504089f));
}
__device__ __forceinline__ int gblock(int rank, int jb) { return jb < DB ? rank * DB + jb : NR * DB + (jb - DB); }

template <bool SYS> __device__ __forceinline__ bool wave_wait_ge(gu32* word, unsigned want, gu32* tmo, unsigned call) {
    const unsigned long long t0 = __builtin_amdgcn_s_memrealtime();
    for (;;) {
        const unsigned v = SYS ? __hip_atomic_load(word, RLX_SYS) : __hip_atomic_load(word, RLX_AGENT);
        if (__all(v >= want)) return true;
        if ((unsigned)__builtin_amdgcn_readfirstlane(__hip_atomic_load(tmo, RLX_AGENT)) == call ||
            __builtin_amdgcn_s_memrealtime() - t0 > DEADLINE_TICKS) {
            __hip_atomic_store(tmo, call, RLX_AGENT);
            return false;
        }
        __builtin_amdgcn_s_sleep(4);
    }
}

__device__ __forceinline__ void pusher(const Params& p, const Self& me, LAS unsigned* lds) {
    const unsigned call = me.call;
    const int tid = threadIdx.x, lane = tid & 63, wid = __builtin_amdgcn_readfirstlane(tid >> 6);
    const int gw = blockIdx.x * 8 + wid, rank = me.rank;
    const unsigned par = call & 1u;
    gu32* ctl = (gu32*)me.ws;
    gu32* tmo = ctl + C_TMO;
    gu32* peer_ctl = (gu32*)p.ws[lane & 7];
    const int rot = gw % 7;
    int dst[7];
#pragma unroll
    for (int j = 0; j < 7; ++j) dst[j] = (rank + 1 + (j + rot) % 7) & 7;
    const unsigned l16 = (unsigned)lane * 16u;

    auto signal = [&](int pos, int uid) {
        unsigned old = 0;
        if (lane == 0) old = __hip_atomic_fetch_add(lds + pos, 1u, __ATOMIC_RELAXED, __HIP_MEMORY_SCOPE_WORKGROUP);
        if (__builtin_amdgcn_readfirstlane(old) == 7u && lane < NR)
            __hip_atomic_fetch_add(peer_ctl + C_CNT + (par * 80 + uid) * LINE, 1u, RLX_SYS);
    };
    {
        __amdgpu_buffer_rsrc_t rs[7];
#pragma unroll
        for (int j = 0; j < 7; ++j) rs[j] = mkrsrc(p.ws[dst[j]], (unsigned)WS_NEED);
        const gchar* wsl = me.ws;
        auto bsrc = [&](int r) -> gvec* {
            const int uid = p.uorder(r / RPU), pc = gw + PW * (r % RPU);
            const int row = uid * 256 + ((pc >> 1) & 15) * 16 + (lane & 15), k = (pc >> 5) * 64 + (pc & 1) * 32 + (lane >> 4) * 8;
            const gchar* a = (const gchar*)p.x + ((size_t)row * KS + k) * 2;
            const gchar* b = wsl + img_off(par, uid) + rank * CHUNK_B + pc * 1024 + l16;
            return (gvec*)(uid < NPAN ? a : b);
        };
        auto bcast = [&](int r0, int r1) {
            u32x4 cur = *bsrc(r0);
            WAIT_VC(0);
#pragma unroll 1
            for (int r = r0; r < r1; ++r) {
                const u32x4 nxt = *bsrc(r + 1 < r1 ? r + 1 : r);
                const int uid = p.uorder(r / RPU), pc = gw + PW * (r % RPU);
                const unsigned so = img_off(par, uid) + rank * CHUNK_B + pc * 1024;
#pragma unroll
                for (int j = 0; j < 7; ++j) __builtin_amdgcn_raw_buffer_store_b128(cur, rs[j], l16, so, 17);
                __builtin_amdgcn_raw_buffer_store_b128(cur, mkrsrc(wsl, uid < NPAN ? (unsigned)WS_NEED : 0u), l16, so, 17);
                WAIT_VC(8);
                if (r > r0 && (r % RPU) == 0) signal(r / RPU - 1, p.uorder(r / RPU - 1));
                cur = nxt;
            }
            WAIT_V(0);
            signal(r1 / RPU - 1, p.uorder(r1 / RPU - 1));
        };
        bcast(0, 2 * RPU);
        {
            const unsigned want = call * (unsigned)(NCOMP / 8);
            wave_wait_ge<false>(ctl + C_STAGE + (lane & 7) * LINE, want, tmo, call);
            __builtin_amdgcn_fence(__ATOMIC_ACQUIRE, "agent");
        }
        {
            const int r0 = 2 * RPU, r1 = (2 + DB) * RPU;
            auto ld = [&](u32x4 (&v)[7], int r) {
                const int jb = r / RPU - 2, pc = gw + PW * (r % RPU);
#pragma unroll
                for (int j = 0; j < 7; ++j) v[j] = *(gvec*)(wsl + SCR_OFF + (unsigned)(dst[j] * DB + jb) * CHUNK_B + pc * 1024 + l16);
            };
            u32x4 cur[7], nxt[7];
            ld(cur, r0);
            WAIT_VC(0);
#pragma unroll 1
            for (int r = r0; r < r1; ++r) {
                ld(nxt, r + 1 < r1 ? r + 1 : r);
                const int jb = r / RPU - 2, pc = gw + PW * (r % RPU);
                const unsigned so = img_off(par, NPAN + jb) + rank * CHUNK_B + pc * 1024;
#pragma unroll
                for (int j = 0; j < 7; ++j) __builtin_amdgcn_raw_buffer_store_b128(cur[j], rs[j], l16, so, 17);
                WAIT_VC(7);
                if (r > r0 && (r % RPU) == 0) signal(r / RPU - 1, NPAN + jb - 1);
#pragma unroll
                for (int j = 0; j < 7; ++j) cur[j] = nxt[j];
            }
            WAIT_V(0);
            signal(r1 / RPU - 1, NPAN + DB - 1);
        }
        bcast((2 + DB) * RPU, NIN * RPU);
    }
    {
        __amdgpu_buffer_rsrc_t rs[7];
#pragma unroll
        for (int j = 0; j < 7; ++j) rs[j] = mkrsrc(p.out[dst[j]], OUT_B);
        const gchar* outl = me.out;
        const unsigned lo = (unsigned)(lane >> 5) * (NN * 2) + (unsigned)(lane & 31) * 16u;
        int ready = 0;
        constexpr int NRND = ND * 128 / PW;
        auto yoff = [&](int r) -> unsigned {
            const int g = r * PW + gw, y = g >> 7, pc = g & 127, t = p.yorder(y);
            if (y >= ready) {
                const unsigned long long t0 = __builtin_amdgcn_s_memrealtime();
                for (;;) {
                    const int yy = y + lane < ND ? y + lane : ND - 1;
                    const unsigned v = __hip_atomic_load(ctl + C_YF + p.yorder(yy), RLX_AGENT);
                    const unsigned long long m = ~__ballot(v == call);
                    const int n = m ? __builtin_ctzll(m) : 64;
                    if (n > 0) { ready = y + n; break; }
                    if ((unsigned)__builtin_amdgcn_readfirstlane(__hip_atomic_load(tmo, RLX_AGENT)) == call ||
                        __builtin_amdgcn_s_memrealtime() - t0 > DEADLINE_TICKS) {
                        __hip_atomic_store(tmo, call, RLX_AGENT); ready = ND; break;
                    }
                    __builtin_amdgcn_s_sleep(8);
                }
                __builtin_amdgcn_fence(__ATOMIC_ACQUIRE, "agent");
            }
            return (unsigned)((t / DB) * 256 + pc * 2) * (NN * 2) + (unsigned)(rank * DB + t % DB) * 512u;
        };
        unsigned so = yoff(0);
        u32x4 cur = *(gvec*)(outl + so + lo);
        WAIT_VC(0);
#pragma unroll 1
        for (int r = 0; r < NRND; ++r) {
            const unsigned sn = yoff(r + 1 < NRND ? r + 1 : r);
            const u32x4 nxt = *(gvec*)(outl + sn + lo);
#pragma unroll
            for (int j = 0; j < 7; ++j) __builtin_amdgcn_raw_buffer_store_b128(cur, rs[j], lo, so, 17);
            WAIT_VC(7);
            cur = nxt; so = sn;
        }
        WAIT_V(0);
        unsigned old = 0;
        if (lane == 0) old = __hip_atomic_fetch_add(lds + 80, 1u, __ATOMIC_RELAXED, __HIP_MEMORY_SCOPE_WORKGROUP);
        if (__builtin_amdgcn_readfirstlane(old) == 7u) {
            if (lane < NR) __hip_atomic_fetch_add(peer_ctl + C_DONE, 1u, RLX_SYS);
            if (blockIdx.x == 0) {
                const bool ok = wave_wait_ge<true>(ctl + C_DONE, call * (unsigned)(NR * NPUSH), tmo, call);
                if ((!ok || __hip_atomic_load(tmo, RLX_AGENT) == call) && lane == 0)
                    *(GAS unsigned short*)(me.out + (size_t)rank * DB * 512) = (unsigned short)0x7fc0;
            }
        }
    }
}

__device__ __forceinline__ unsigned bpos(unsigned fq, unsigned q, unsigned r) { return r * 256u + (((fq * 4u + q) ^ ((r & 1u) | ((r & 2u) << 2))) * 16u); }

__device__ __forceinline__ void stage_w(const Params& p, const Self& me) {
    const unsigned call = me.call;
    const int tid = threadIdx.x, lane = tid & 63, wid = __builtin_amdgcn_readfirstlane(tid >> 6);
    const int rank = me.rank, cw = (blockIdx.x - NPUSH) * 8 + wid;
    const unsigned par = call & 1u;
    const __amdgpu_buffer_rsrc_t rs = mkrsrc(me.ws, (unsigned)WS_NEED);
    const int fq = lane >> 4, c = lane & 15;
    for (int task = cw; task < 2048; task += NCOMP * 8) {
        const int B = task >> 5, ktl = (task >> 2) & 7, ks = (task >> 1) & 1, g = task & 1;
        const char* src = p.w + ((size_t)(ktl * 64 + ks * 32 + fq * 8) * NN + B * 256 + g * 128 + c * 8) * 2;
        u32x4 v[8];
#pragma unroll
        for (int j = 0; j < 8; ++j) v[j] = *(const u32x4*)(src + (size_t)j * NN * 2);
        unsigned base;
        if (B >= NR * DB) base = img_off(par, NPAN + DB + (B - NR * DB)) + rank * CHUNK_B;
        else if (B / DB == rank) base = img_off(par, NPAN + B % DB) + rank * CHUNK_B;
        else base = SCR_OFF + (unsigned)B * CHUNK_B;
        base += (unsigned)ktl * 32768u + (unsigned)ks * 1024u;
        const unsigned lb = (unsigned)((c >> 2) & 1) * HTB + (unsigned)(2 * g + (c >> 3)) * 4096u;
#pragma unroll
        for (int e = 0; e < 8; ++e) {
            u32x4 o;
#pragma unroll
            for (int d = 0; d < 4; ++d)
                o[d] = __builtin_amdgcn_perm(v[2 * d + 1][e >> 1], v[2 * d][e >> 1], (e & 1) ? 0x07060302u : 0x05040100u);
            __builtin_amdgcn_raw_buffer_store_b128(o, rs, lb + (unsigned)(e >> 2) * 2048u + bpos(fq, c & 3, e & 3), base, 16);
        }
    }
    WAIT_V(0);
    __syncthreads();
    if (tid == 0) __hip_atomic_fetch_add((gu32*)me.ws + C_STAGE + (blockIdx.x & 7) * LINE, 1u, RLX_AGENT);
}

__device__ __forceinline__ void compute(const Params& p, const Self& me, LAS unsigned char* lds) {
    const unsigned call = me.call;
    const int tid = threadIdx.x, wid = __builtin_amdgcn_readfirstlane(tid >> 6), lane = tid & 63, wr = wid >> 2, wc = wid & 3, fr = lane & 15, fq = lane >> 4;
    const int rank = me.rank, cwg = blockIdx.x - NPUSH;
    const unsigned par = call & 1u, want = ((call + 1u) >> 1) * (unsigned)(NR * NPUSH);
    if (cwg >= NTILE) return;
    gu32* ctl = (gu32*)me.ws;
    LAS unsigned* badf = (LAS unsigned*)(lds + STAGE_BYTES);
    const __amdgpu_buffer_rsrc_t rw = mkrsrc(me.ws, (unsigned)WS_NEED);
    const unsigned img = img_off(par, 0);
    const unsigned voff = (unsigned)tid * 16u, ldsw = (unsigned)wid * 1024u;
    const int aoff = wr * 8192 + lane * 16, boff = wc * 4096 + (int)bpos(fq, fr >> 2, fr & 3);
    constexpr unsigned kstep = 2 * HTB, hstep = HTB;
#define SA(b, h) (((b) * 2 + (h)) * HTB)
#define SB(b, h) ((4 + (b) * 2 + (h)) * HTB)
#define STAGE(bufoff, gbase) do { _Pragma("unroll") for (int _i = 0; _i < 2; ++_i) \
        __builtin_amdgcn_raw_ptr_buffer_load_lds(rw, (LAS void*)(lds + (bufoff) + ldsw + _i * 8192), 16, voff, (gbase) + _i * 8192, 0, 0); } while (0)
#define LDA(dst, b, h) do { _Pragma("unroll") for (int m = 0; m < 4; ++m) _Pragma("unroll") for (int k = 0; k < 2; ++k) dst[m][k] = *(const LAS bf16x8*)(lds + SA(b, h) + aoff + m * 2048 + k * 1024); } while (0)
#define LDB(dst, b, h) do { _Pragma("unroll") for (int n = 0; n < 2; ++n) _Pragma("unroll") for (int k = 0; k < 2; ++k) dst[n][k] = *(const LAS bf16x8*)(lds + SB(b, h) + boff + n * 2048 + k * 1024); } while (0)
#define MMA(ai, bj, At, Bt) do { __builtin_amdgcn_s_setprio(1); _Pragma("unroll") for (int m = 0; m < 4; ++m) _Pragma("unroll") for (int n = 0; n < 2; ++n) _Pragma("unroll") for (int k = 0; k < 2; ++k) \
        acc[ai][bj][m][n] = __builtin_amdgcn_mfma_f32_16x16x32_bf16(Bt[n][k], At[m][k], acc[ai][bj][m][n], 0, 0, 0); __builtin_amdgcn_s_setprio(0); } while (0)
    auto ready = [&](int tl) {
        if (wid == 0) {
            const int uid = (lane & 1) ? NPAN + (tl >> 4) : (tl & 15);
            if (!wave_wait_ge<true>(ctl + C_CNT + (par * 80 + uid) * LINE, want, ctl + C_TMO, call)) { if (lane == 0) *badf = 1u; }
            __builtin_amdgcn_fence(__ATOMIC_ACQUIRE, "");
            asm volatile("s_waitcnt vmcnt(0) lgkmcnt(0)" ::: "memory");
        }
        asm volatile("" ::: "memory"); BAR; asm volatile("" ::: "memory");
    };
    int q = cwg, cur = p.tile(q), pend = -1;
    f32x4 acc[2][2][4][2];
#pragma unroll
    for (int a = 0; a < 2; ++a)
#pragma unroll
        for (int b = 0; b < 2; ++b)
#pragma unroll
            for (int m = 0; m < 4; ++m)
#pragma unroll
                for (int n = 0; n < 2; ++n) acc[a][b][m][n] = (f32x4){0.f, 0.f, 0.f, 0.f};
    bf16x8 At[4][2], B0[2][2], B1[2][2];
    unsigned cA = img + (unsigned)(cur & 15) * IMG_B, cB = img + (unsigned)(NPAN + (cur >> 4)) * IMG_B;
    ready(cur);
    STAGE(SB(0, 0), cB); STAGE(SA(0, 0), cA); STAGE(SB(0, 1), cB + hstep); STAGE(SA(0, 1), cA + hstep);
    if (wr == 1) BAR;
    WAIT_V(4); BAR;
    STAGE(SB(1, 0), cB + kstep); STAGE(SA(1, 0), cA + kstep); STAGE(SB(1, 1), cB + hstep + kstep);
    WAIT_V(6); BAR;
    constexpr int nt = KK / 64;
    for (;;) {
        const bool has_next = q + NCOMP < NTILE;
        const int nxt = has_next ? p.tile(q + NCOMP) : cur;
        const unsigned nA = img + (unsigned)(nxt & 15) * IMG_B, nB = img + (unsigned)(NPAN + (nxt >> 4)) * IMG_B;
        for (int t = 0; t < nt; t += 2) {
            const bool last = (t == nt - 2);
            const unsigned a1 = cA + (unsigned)(t + 1) * kstep;
            const unsigned a2 = last ? nA : cA + (unsigned)(t + 2) * kstep, b2 = last ? nB : cB + (unsigned)(t + 2) * kstep;
            const unsigned a3 = a2 + kstep, b3 = b2 + kstep;
            if (last && has_next) ready(nxt);
            if (t == 2 && pend >= 0) { if (tid == 0) __hip_atomic_store(ctl + C_YF + pend, call, RLX_AGENT); pend = -1; }
            LDB(B0, 0, 0); SCHED; LDA(At, 0, 0); STAGE(SA(1, 1), a1 + hstep);
            WAIT_L(8); BAR; WAIT_L(0); MMA(0, 0, At, B0); BAR; SCHED;
            LDB(B1, 0, 1); STAGE(SB(0, 0), b2);
            BAR; WAIT_L(0); MMA(0, 1, At, B1); BAR;
            LDA(At, 0, 1); STAGE(SA(0, 0), a2);
            BAR; WAIT_L(0); MMA(1, 0, At, B0); BAR; SCHED;
            STAGE(SB(0, 1), b2 + hstep);
            WAIT_V(6); BAR; MMA(1, 1, At, B1); BAR;
            LDB(B0, 1, 0); SCHED; LDA(At, 1, 0); STAGE(SA(0, 1), a2 + hstep);
            WAIT_L(8); BAR; WAIT_L(0); MMA(0, 0, At, B0); BAR; SCHED;
            LDB(B1, 1, 1); STAGE(SB(1, 0), b3);
            BAR; WAIT_L(0); MMA(0, 1, At, B1); BAR;
            LDA(At, 1, 1); STAGE(SA(1, 0), a3);
            BAR; WAIT_L(0); MMA(1, 0, At, B0); BAR; SCHED;
            STAGE(SB(1, 1), b3 + hstep);
            WAIT_V(6); BAR; MMA(1, 1, At, B1); BAR;
        }
        {
            const bool bad = *badf != 0u;
            const __amdgpu_buffer_rsrc_t ro = mkrsrc(me.out, OUT_B);
            const unsigned so = (unsigned)((cur & 15) * 256 + wr * 64) * (NN * 2) + (unsigned)gblock(rank, cur >> 4) * 512u + (unsigned)wc * 128u;
            const unsigned vo = (unsigned)fr * (NN * 2) + (unsigned)fq * 16u;
#pragma unroll
            for (int ai = 0; ai < 2; ++ai)
#pragma unroll
                for (int m = 0; m < 4; ++m)
#pragma unroll
                    for (int bj = 0; bj < 2; ++bj) {
                        const f32x4 v0 = acc[ai][bj][m][0], v1 = acc[ai][bj][m][1];
                        u32x4 w;
                        w.x = cvt_pk_bf16(silu(v0[0]), silu(v0[1])); w.y = cvt_pk_bf16(silu(v0[2]), silu(v0[3]));
                        w.z = cvt_pk_bf16(silu(v1[0]), silu(v1[1])); w.w = cvt_pk_bf16(silu(v1[2]), silu(v1[3]));
                        if (bad) w = (u32x4){0x7fc07fc0u, 0x7fc07fc0u, 0x7fc07fc0u, 0x7fc07fc0u};
                        __builtin_amdgcn_raw_buffer_store_b128(w, ro, vo + bj * 64, so + (unsigned)(ai * 128 + m * 16) * (NN * 2), 16);
                    }
            if ((cur >> 4) < DB) pend = (cur & 15) * DB + (cur >> 4);
        }
        if (!has_next) break;
#pragma unroll
        for (int a = 0; a < 2; ++a)
#pragma unroll
            for (int b = 0; b < 2; ++b)
#pragma unroll
                for (int m = 0; m < 4; ++m)
#pragma unroll
                    for (int n = 0; n < 2; ++n) acc[a][b][m][n] = (f32x4){0.f, 0.f, 0.f, 0.f};
        cur = nxt; cA = nA; cB = nB; q += NCOMP;
    }
    WAIT_V(0);
    if (wr == 0) BAR;
    BAR;
    if (pend >= 0 && tid == 0) __hip_atomic_store(ctl + C_YF + pend, call, RLX_AGENT);
}

__global__ __launch_bounds__(512, 2) void fused_gemm_silu(const Params p) {
    extern __shared__ __attribute__((aligned(16))) unsigned char shm[];
    LAS unsigned char* lds = (LAS unsigned char*)shm;
    LAS unsigned* ctlw = (LAS unsigned*)(lds + STAGE_BYTES);
    Self me;
    me.out = uni(p.out_self); me.ws = uni(p.ws_self); me.rank = __builtin_amdgcn_readfirstlane(p.rank);
    gu32* cc = (gu32*)me.ws + C_CALL + blockIdx.x * LINE;
    if (threadIdx.x < 128) ctlw[threadIdx.x] = 0u;
    if (threadIdx.x == 128) { const unsigned c = __hip_atomic_load(cc, RLX_AGENT) + 1u; ctlw[128] = c; __hip_atomic_store(cc, c, RLX_AGENT); }
    __syncthreads();
    me.call = __builtin_amdgcn_readfirstlane(ctlw[128]);
    if (blockIdx.x < NPUSH) {
        pusher(p, me, ctlw);
    } else {
        stage_w(p, me);
        compute(p, me, lds);
    }
}

void build_tables(Params& pp) {
    struct { unsigned char* uorder; unsigned char* yorder; unsigned short* tile; } p{(unsigned char*)pp.uorder4, (unsigned char*)pp.yorder4, (unsigned short*)pp.tile2};
    int n = 0, nx = 0, nw = 0;
    p.uorder[n++] = nx++; p.uorder[n++] = nx++;
    while (nw < DB) p.uorder[n++] = NPAN + nw++;
    while (n < NIN) {
        if (nx < NPAN && (nx <= nw || nw == NLB)) p.uorder[n++] = nx++;
        else p.uorder[n++] = NPAN + nw++;
    }
    for (int i = NIN; i < 80; ++i) p.uorder[i] = 0;
    int upos[NIN];
    for (int i = 0; i < NIN; ++i) upos[p.uorder[i]] = i;
    int t = 0, y = 0;
    for (int pos = 0; pos < NIN; ++pos)
        for (int jb = 0; jb < NLB; ++jb)
            for (int i = 0; i < NPAN; ++i) {
                const int key = upos[i] > upos[NPAN + jb] ? upos[i] : upos[NPAN + jb];
                if (key != pos) continue;
                p.tile[t++] = (unsigned short)(i | jb << 4);
                if (jb < DB) p.yorder[y++] = (unsigned char)(i * DB + jb);
            }
    for (; y < 128; ++y) p.yorder[y] = 0;
}
}

extern "C" void kernel_launch(int rank, int nranks, ncclComm_t comm, hipStream_t stream,
                              void* const* d_in, const int* in_sizes, int n_in,
                              void* d_out, int out_size, const int* in_dtypes, int out_dtype,
                              void* const* d_out_all, void* const* d_ws_all, size_t ws_bytes) {
    if (nranks != NR || (unsigned)rank >= (unsigned)NR || n_in < 2 || in_sizes[0] != MM * KS || in_sizes[1] != KS * NN ||
        out_size != MM * NN || ws_bytes < WS_NEED) {
        fprintf(stderr, "unexpected problem shape\n");
        return;
    }
    static bool attr_set = false;
    if (!attr_set) {
        HIP_CHECK(hipFuncSetAttribute((const void*)fused_gemm_silu, hipFuncAttributeMaxDynamicSharedMemorySize, LDS_BYTES));
        attr_set = true;
    }
    Params p;
    p.x = (const char*)d_in[0]; p.w = (const char*)d_in[1]; p.rank = rank;
    p.out_self = (char*)d_out_all[rank]; p.ws_self = (char*)d_ws_all[rank];
    for (int r = 0; r < NR; ++r) { p.out[r] = (char*)d_out_all[r]; p.ws[r] = (char*)d_ws_all[r]; }
    build_tables(p);
    hipLaunchKernelGGL(fused_gemm_silu, dim3(NWG), dim3(512), LDS_BYTES, stream, p);
    HIP_CHECK(hipGetLastError());
}
```

```cpp
#include <hip/hip_runtime.h>
#include <rccl/rccl.h>
#include <cstdio>
#include <cstdint>

#define HIP_CHECK(expr) do { hipError_t _e = (expr); if (_e != hipSuccess) { \
    fprintf(stderr, "HIP error %s at %s:%d\n", hipGetErrorString(_e), __FILE__, __LINE__); \
    return; } } while (0)

#define NRB 32
#define NPUSH 24

#define YMODE 1

#define KCLK 1
#define XMAP 1

namespace {
#define GAS __attribute__((address_space(1)))
#define LAS __attribute__((address_space(3)))
typedef short bf16x8 __attribute__((ext_vector_type(8)));
typedef float f32x4 __attribute__((ext_vector_type(4)));
typedef unsigned u32x4 __attribute__((ext_vector_type(4)));
typedef GAS unsigned gu32;

constexpr int NR = 8, MM = 4096, KK = 4096, NN = 16384, KS = KK / NR;
constexpr int NPAN = MM / 256, NBLK = NN / 256;
constexpr int DB = (NBLK - NRB) / NR;
constexpr int NLB = DB + NRB;
constexpr int NIN = NPAN + NLB;
constexpr int NTILE = NPAN * NLB, ND = NPAN * DB;
constexpr int NWG = 256, NCOMP = NWG - NPUSH;
constexpr int NBU = NPAN + NRB;
constexpr int NITEM_IN = NBU * 32, NITEM = NITEM_IN + ND * 16;
static_assert(NRB % 8 == 0 && DB >= 1 && NRB <= 56 && NPUSH % 8 == 0 && 2 * NPUSH <= 64 && NCOMP >= 128, "config");
constexpr unsigned CHUNK_B = 256u << 10;
constexpr unsigned IMG_B = 2u << 20;
constexpr unsigned CTRL_B = 1u << 20;
constexpr size_t WS_NEED = (size_t)CTRL_B + 2u * NIN * IMG_B;
constexpr unsigned OUT_B = (unsigned)((size_t)MM * NN * 2);
constexpr int LINE = 32;
constexpr int C_CALL = 0;
constexpr int C_CNT = C_CALL + NWG * LINE;
constexpr int C_YF = C_CNT + 2 * 80 * 8 * LINE;
constexpr int C_DONE = C_YF + 192;
constexpr int C_TMO = C_DONE + LINE;
constexpr int C_CLAIM = C_TMO + LINE;
constexpr int C_KCLK = C_CLAIM + LINE;
static_assert((C_KCLK + 16 * LINE) * 4 <= (512 << 10) && NIN <= 80 && ND <= 128, "ctrl");
constexpr unsigned long long DEADLINE_TICKS = 50000000ull;
constexpr int HTB = 16384, STAGE_BYTES = 8 * HTB, LDS_BYTES = STAGE_BYTES + 1024;

struct Params {
    const char* x; const char* w;
    char* out_self; char* ws_self;
    char* buf[2 * NR + 1];
    int rank;
    unsigned uorder4[80 / 4];
    unsigned yorder4[128 / 4];
    unsigned ypos4[128 / 4];
    unsigned tile2[NTILE / 2];
};
typedef const __attribute__((address_space(4))) Params* KP;
__device__ __forceinline__ int t_uorder(KP p, int i) { return (p->uorder4[i >> 2] >> ((i & 3) * 8)) & 255; }
__device__ __forceinline__ int t_yorder(KP p, int i) { return (p->yorder4[i >> 2] >> ((i & 3) * 8)) & 255; }
__device__ __forceinline__ int t_ypos(KP p, int i) { return (p->ypos4[i >> 2] >> ((i & 3) * 8)) & 255; }
__device__ __forceinline__ int t_tile(KP p, int i) { return (p->tile2[i >> 1] >> ((i & 1) * 16)) & 65535; }

#define RLX_AGENT __ATOMIC_RELAXED, __HIP_MEMORY_SCOPE_AGENT
#define RLX_SYS __ATOMIC_RELAXED, __HIP_MEMORY_SCOPE_SYSTEM
#define WAIT_V(n) asm volatile("s_waitcnt vmcnt(" #n ")" ::: "memory")
#define WAIT_L(n) asm volatile("s_waitcnt lgkmcnt(" #n ")" ::: "memory")
#define WAIT_VC(n) do { asm volatile("" ::: "memory"); __builtin_amdgcn_s_waitcnt(0x0F70 | ((n) & 15) | (((n) >> 4) << 14)); asm volatile("" ::: "memory"); } while (0)
#define BAR __builtin_amdgcn_s_barrier()
#define SCHED __builtin_amdgcn_sched_barrier(0)

#define STAMP(slot) do { } while (0)
#define STAMPC(slot) do { } while (0)
typedef GAS char gchar;
typedef const GAS u32x4 gvec;
__device__ __forceinline__ gchar* uni(const void* p) {
    const unsigned long long v = (unsigned long long)p;
    const unsigned lo = __builtin_amdgcn_readfirstlane((unsigned)v), hi = __builtin_amdgcn_readfirstlane((unsigned)(v >> 32));
    return (gchar*)(((unsigned long long)hi << 32) | lo);
}
struct Self { gchar* out; gchar* ws; int rank; unsigned call; };
__device__ __forceinline__ int cnt_idx(unsigned par, int uid, int sender) { return C_CNT + (((int)par * 80 + uid) * 8 + sender) * LINE; }
__device__ __forceinline__ unsigned img_off(unsigned par, unsigned uid) { return CTRL_B + (par * NIN + uid) * IMG_B; }
template <class P> __device__ __forceinline__ __amdgpu_buffer_rsrc_t mkrsrc(P p, unsigned bytes) {
    return __builtin_amdgcn_make_buffer_rsrc((void*)p, 0, bytes, 0x00020000);
}
__device__ __forceinline__ unsigned cvt_pk_bf16(float lo, float hi) { unsigned r; asm("v_cvt_pk_bf16_f32 %0, %1, %2" : "=v"(r) : "v"(lo), "v"(hi)); return r; }
__device__ __forceinline__ float silu(float y) {
    const float c = __builtin_amdgcn_fmed3f(y, -60.0f, 60.0f);
    return y * __builtin_amdgcn_rcpf(1.0f + __builtin_amdgcn_exp2f(c * -1.44269504089f));
}
__device__ __forceinline__ int gblock(int rank, int jb) { return jb < DB ? rank * DB + jb : NR * DB + (jb - DB); }

template <bool SYS> __device__ __forceinline__ bool wave_wait_ge(gu32* word, unsigned want, gu32* tmo, unsigned call, unsigned& v) {
    const unsigned long long t0 = __builtin_amdgcn_s_memrealtime();
    for (;;) {
        v = SYS ? __hip_atomic_load(word, RLX_SYS) : __hip_atomic_load(word, RLX_AGENT);
        if (__all(v >= want)) return true;
        if ((unsigned)__builtin_amdgcn_readfirstlane(__hip_atomic_load(tmo, RLX_AGENT)) == call ||
            __builtin_amdgcn_s_memrealtime() - t0 > DEADLINE_TICKS) {
            __hip_atomic_store(tmo, call, RLX_AGENT);
            return false;
        }
        __builtin_amdgcn_s_sleep(4);
    }
}

template <bool SYS> __device__ __forceinline__ bool wave_wait_ge(gu32* word, unsigned want, gu32* tmo, unsigned call) {
    unsigned v;
    return wave_wait_ge<SYS>(word, want, tmo, call, v);
}

constexpr int YGRP = YMODE ? DB : 1;
__device__ __forceinline__ int y_need(int c) { return ((c - NITEM_IN) / (16 * YGRP) + 1) * YGRP; }
__device__ __forceinline__ void item_decode(KP p, int c, unsigned par, int rank, int wid, int lane, int& uid, unsigned& so, unsigned& vo) {
    if (c < NITEM_IN) {
        uid = t_uorder(p, c >> 5); so = img_off(par, uid) + rank * CHUNK_B + (unsigned)((c & 31) * 8 + wid) * 1024u; vo = (unsigned)lane * 16u;
    } else {
        const int cy = (c < NITEM ? c : NITEM - 1) - NITEM_IN, t = t_yorder(p, cy / (16 * YGRP) * YGRP);
        const unsigned o = (unsigned)((cy % (16 * YGRP)) * 8 + wid) * 1024u + (unsigned)lane * 16u;
        uid = -1; so = (unsigned)((t / DB) * 256) * (NN * 2) + (unsigned)(rank * DB + t % DB) * 512u;
        vo = (o / (YGRP * 512u)) * (NN * 2) + o % (YGRP * 512u);
    }
}
__device__ __forceinline__ u32x4 item_load(KP p, int c, int uid, unsigned so, unsigned vo, bool on, int rank, int wid, int lane) {
    const bool isx = (unsigned)uid < (unsigned)NPAN, isy = uid < 0;
    const int pc = (c & 31) * 8 + wid;
    const unsigned sx = (unsigned)(uid * 256 + ((pc >> 1) & 15) * 16) * (KS * 2) + (unsigned)((pc >> 5) * 128 + (pc & 1) * 64);
    const char* base = p->buf[isx ? 2 * NR : isy ? NR + rank : rank];
    const unsigned bytes = !on ? 0u : isx ? (unsigned)(MM * KS * 2) : isy ? OUT_B : (unsigned)WS_NEED;
    const unsigned vx = (unsigned)(lane & 15) * (KS * 2) + (unsigned)(lane >> 4) * 16u;
    return __builtin_amdgcn_raw_buffer_load_b128(mkrsrc(base, bytes), isx ? vx : vo, isx ? sx : so, 16);
}
__device__ __forceinline__ void item_signal(gu32* peer_ctl, int uid, unsigned par, int rank) {
    const int tid = threadIdx.x;
    if (uid >= 0 && tid < NR && (uid < NPAN || tid != rank)) __hip_atomic_fetch_add(peer_ctl + cnt_idx(par, uid, rank), 1u, RLX_SYS);
}
__device__ __forceinline__ void pusher(KP p, const Self me, LAS unsigned* lds) {
    const unsigned call = me.call, par = call & 1u, want = ((call + 1u) >> 1) * 32u;
    const int tid = threadIdx.x, lane = tid & 63, wid = __builtin_amdgcn_readfirstlane(tid >> 6), rank = me.rank;
    gu32* ctl = (gu32*)me.ws;
    gu32* tmo = ctl + C_TMO;
    gu32* peer_ctl = (gu32*)p->buf[lane & 7];
    const int rot = (blockIdx.x * 8 + wid) % 7;
    unsigned dsts = 0;
#pragma unroll
    for (int j = 0; j < 7; ++j) dsts |= (unsigned)((rank + 1 + (j + rot) % 7) & 7) << (4 * j);
    unsigned long long dp0, dp1, dp2, dp3, dp4, dp5, dp6;
#define LOAD_DP(o) do { dp0 = (unsigned long long)p->buf[(dsts & 7) + (o)]; dp1 = (unsigned long long)p->buf[((dsts >> 4) & 7) + (o)]; \
        dp2 = (unsigned long long)p->buf[((dsts >> 8) & 7) + (o)]; dp3 = (unsigned long long)p->buf[((dsts >> 12) & 7) + (o)]; \
        dp4 = (unsigned long long)p->buf[((dsts >> 16) & 7) + (o)]; dp5 = (unsigned long long)p->buf[((dsts >> 20) & 7) + (o)]; \
        dp6 = (unsigned long long)p->buf[((dsts >> 24) & 7) + (o)]; } while (0)
    LOAD_DP(0);
    bool dp_out = false;
    const int stg_idx = cnt_idx(par, NPAN + DB + (lane < NRB ? lane : 0), rank);
    const unsigned cbase = (call - 1u) * (unsigned)NITEM - 2u * NPUSH;
    int c0 = blockIdx.x, c1 = NPUSH + blockIdx.x, prev = -1, it_n = 0;
    int staged = NRB == 0, yready = 0;
    bool have0 = true;
    int u0; unsigned so0, vo0;
    item_decode(p, c0, par, rank, wid, lane, u0, so0, vo0);
    u32x4 cur = item_load(p, c0, u0, so0, vo0, true, rank, wid, lane);
    WAIT_VC(0);
#pragma unroll 1
    while (c0 < NITEM) {
        LAS unsigned* sl = lds + (it_n & 1) * 4;
        if (!have0) {
            WAIT_V(0); BAR;
            item_signal(peer_ctl, prev, par, rank); prev = -1;
            if (wid == 0) wave_wait_ge<false>(ctl + (u0 >= 0 ? stg_idx : C_YF + y_need(c0) - 1 - lane % YGRP), u0 >= 0 ? want : call, tmo, call);
            asm volatile("" ::: "memory"); BAR; asm volatile("" ::: "memory");
            if (u0 >= 0) staged = 1; else yready = y_need(c0);
            cur = item_load(p, c0, u0, so0, vo0, true, rank, wid, lane);
            WAIT_VC(0);
        }
        unsigned aux = 1u;
        if (wid == 0) { if (lane == 0) asm volatile("global_atomic_add %0, %1, %0, off sc0" : "+v"(aux) : "v"(ctl + C_CLAIM) : "memory"); }
        else if (wid <= 2) asm volatile("global_load_dword %0, %1, off sc1" : "+v"(aux) : "v"(ctl + (wid == 1 ? stg_idx : C_YF + yready + lane)) : "memory");
        int u1; unsigned so1, vo1;
        item_decode(p, c1, par, rank, wid, lane, u1, so1, vo1);
        const bool have1 = u1 >= NPAN ? staged != 0 : u1 >= 0 ? true : y_need(c1) <= yready;
        const u32x4 nxt = item_load(p, c1, u1, so1, vo1, have1 && c1 < NITEM, rank, wid, lane);
        {
            const bool isy = u0 < 0;
            const unsigned vo = vo0, bytes = isy ? OUT_B : (unsigned)WS_NEED;
            if (isy && !dp_out) { LOAD_DP(NR); dp_out = true; }
            __builtin_amdgcn_raw_buffer_store_b128(cur, mkrsrc(dp0, bytes), vo, so0, 17);
            __builtin_amdgcn_raw_buffer_store_b128(cur, mkrsrc(dp1, bytes), vo, so0, 17);
            __builtin_amdgcn_raw_buffer_store_b128(cur, mkrsrc(dp2, bytes), vo, so0, 17);
            __builtin_amdgcn_raw_buffer_store_b128(cur, mkrsrc(dp3, bytes), vo, so0, 17);
            __builtin_amdgcn_raw_buffer_store_b128(cur, mkrsrc(dp4, bytes), vo, so0, 17);
            __builtin_amdgcn_raw_buffer_store_b128(cur, mkrsrc(dp5, bytes), vo, so0, 17);
            __builtin_amdgcn_raw_buffer_store_b128(cur, mkrsrc(dp6, bytes), vo, so0, 17);
            __builtin_amdgcn_raw_buffer_store_b128(cur, mkrsrc(me.ws, (unsigned)u0 < (unsigned)NPAN ? (unsigned)WS_NEED : 0u), vo, so0, 17);
        }
        WAIT_VC(8);
        asm volatile("" : "+v"(aux));
        if (wid <= 2) {
            const unsigned long long m = ~__ballot(aux >= (wid == 1 ? want : call));
            const unsigned v = wid == 0 ? aux - cbase : wid == 1 ? (unsigned)(m == 0ull) : (unsigned)(yready + (m ? __builtin_ctzll(m) : 64));
            if (lane == 0) sl[wid] = v;
        }
        WAIT_L(0); BAR; asm volatile("" ::: "memory");
        item_signal(peer_ctl, prev, par, rank);
        STAMP(2 + (it_n < 240 ? it_n : 240));
        prev = u0; c0 = c1; u0 = u1; so0 = so1; vo0 = vo1; have0 = have1; cur = nxt; ++it_n;
        c1 = __builtin_amdgcn_readfirstlane(sl[0]);
        staged |= __builtin_amdgcn_readfirstlane(sl[1]);
        { const int yr = __builtin_amdgcn_readfirstlane(sl[2]); yready = yr < ND ? yr : ND; }
    }
    WAIT_V(0); BAR;
    item_signal(peer_ctl, prev, par, rank);
    STAMP(250);
    if (wid == 0) {
        if (lane < NR) __hip_atomic_fetch_add(peer_ctl + C_DONE, 1u, RLX_SYS);
        if (blockIdx.x == 0) {
            const bool ok = wave_wait_ge<true>(ctl + C_DONE, call * (unsigned)(NR * NPUSH), tmo, call);
            if ((!ok || __hip_atomic_load(tmo, RLX_AGENT) == call) && lane == 0)
                *(GAS unsigned short*)(me.out + (size_t)rank * DB * 512) = (unsigned short)0x7fc0;
            STAMP(251);
        }
    }
}

__device__ __forceinline__ unsigned bpos(unsigned fq, unsigned q, unsigned r) { return r * 256u + (((fq * 4u + q) ^ ((r & 1u) | ((r & 2u) << 2))) * 16u); }

__device__ __forceinline__ void stage_w(KP p, const Self me) {
    const unsigned call = me.call, par = call & 1u;
    const int tid = threadIdx.x, lane = tid & 63, wid = __builtin_amdgcn_readfirstlane(tid >> 6), rank = me.rank;
    const int fq = lane >> 4, c = lane & 15;
    for (int wt = NWG - 1 - (int)blockIdx.x; wt < 256; wt += NCOMP) {
        int B, dest, uid;
        if (wt < NR * DB * 4) { dest = (rank + 1 + (wt & 7)) & 7; const int jb = (wt >> 5); B = dest * DB + jb; uid = NPAN + jb; }
        else { B = wt >> 2; dest = rank; uid = NPAN + DB + (B - NR * DB); }
        const int quarter = wt < NR * DB * 4 ? (wt >> 3) & 3 : wt & 3;
        const int task = quarter * 8 + wid, ktl = task >> 2, ks = (task >> 1) & 1, g = task & 1;
        const gchar* src = (const gchar*)p->w + ((size_t)(ktl * 64 + ks * 32 + fq * 8) * NN + B * 256 + g * 128 + c * 8) * 2;
        u32x4 v[8];
#pragma unroll
        for (int j = 0; j < 8; ++j) v[j] = *(gvec*)(src + (size_t)j * NN * 2);
        gchar* dws = (gchar*)p->buf[dest];
        const __amdgpu_buffer_rsrc_t rs = mkrsrc(dws, (unsigned)WS_NEED);
        const unsigned base = img_off(par, uid) + rank * CHUNK_B + (unsigned)ktl * 32768u + (unsigned)ks * 1024u;
        const unsigned lb = (unsigned)((c >> 2) & 1) * HTB + (unsigned)(2 * g + (c >> 3)) * 4096u;
#pragma unroll
        for (int e = 0; e < 8; ++e) {
            u32x4 o;
#pragma unroll
            for (int d = 0; d < 4; ++d)
                o[d] = __builtin_amdgcn_perm(v[2 * d + 1][e >> 1], v[2 * d][e >> 1], (e & 1) ? 0x07060302u : 0x05040100u);
            __builtin_amdgcn_raw_buffer_store_b128(o, rs, lb + (unsigned)(e >> 2) * 2048u + bpos(fq, c & 3, e & 3), base, 17);
        }
        WAIT_V(0);
        __syncthreads();
        if (tid == 0) __hip_atomic_fetch_add((gu32*)dws + cnt_idx(par, uid, rank), 8u, RLX_SYS);
    }
}

__device__ __forceinline__ void compute(KP p, const Self me, LAS unsigned char* lds) {
    const unsigned call = me.call;
    const int tid = threadIdx.x, wid = __builtin_amdgcn_readfirstlane(tid >> 6), lane = tid & 63, wr = wid >> 2, wc = wid & 3, fr = lane & 15, fq = lane >> 4;
    const int rank = me.rank, cwg = blockIdx.x - NPUSH;
    const unsigned par = call & 1u, want = ((call + 1u) >> 1) * 32u;
    gu32* ctl = (gu32*)me.ws;
    LAS unsigned* badf = (LAS unsigned*)(lds + STAGE_BYTES);
    const __amdgpu_buffer_rsrc_t rw = mkrsrc(me.ws, (unsigned)WS_NEED);
    const unsigned img = img_off(par, 0);
    const unsigned voff = (unsigned)tid * 16u, ldsw = (unsigned)wid * 1024u;
    const int aoff = wr * 8192 + lane * 16, boff = wc * 4096 + (int)bpos(fq, fr >> 2, fr & 3);
    constexpr unsigned kstep = 2 * HTB, hstep = HTB;
#define SA(b, h) (((b) * 2 + (h)) * HTB)
#define SB(b, h) ((4 + (b) * 2 + (h)) * HTB)
#define STAGE(bufoff, gbase) do { _Pragma("unroll") for (int _i = 0; _i < 2; ++_i) \
        __builtin_amdgcn_raw_ptr_buffer_load_lds(rw, (LAS void*)(lds + (bufoff) + ldsw + _i * 8192), 16, voff, (gbase) + _i * 8192, 0, 0); } while (0)
#define LDA(dst, b, h) do { _Pragma("unroll") for (int m = 0; m < 4; ++m) _Pragma("unroll") for (int k = 0; k < 2; ++k) dst[m][k] = *(const LAS bf16x8*)(lds + SA(b, h) + aoff + m * 2048 + k * 1024); } while (0)
#define LDB(dst, b, h) do { _Pragma("unroll") for (int n = 0; n < 2; ++n) _Pragma("unroll") for (int k = 0; k < 2; ++k) dst[n][k] = *(const LAS bf16x8*)(lds + SB(b, h) + boff + n * 2048 + k * 1024); } while (0)
#define MMA(ai, bj, At, Bt) do { __builtin_amdgcn_s_setprio(1); _Pragma("unroll") for (int m = 0; m < 4; ++m) _Pragma("unroll") for (int n = 0; n < 2; ++n) _Pragma("unroll") for (int k = 0; k < 2; ++k) \
        acc[ai][bj][m][n] = __builtin_amdgcn_mfma_f32_16x16x32_bf16(Bt[n][k], At[m][k], acc[ai][bj][m][n], 0, 0, 0); __builtin_amdgcn_s_setprio(0); } while (0)
    const int kclk_idx = C_KCLK + (int)(__builtin_amdgcn_s_getreg(20 | (3 << 11)) & 15u) * LINE;
#define POLL_WORD(tl) (ctl + (lane < 32 ? cnt_idx(par, (lane & 8) ? NPAN + ((tl) >> 4) : ((tl) & 15), lane & 7) : kclk_idx))
    LAS unsigned* gof = badf + 1;
    constexpr int nt = KK / 64;
    int q = cwg, sn = 0;
    f32x4 acc[2][2][4][2];
    bf16x8 At[4][2], B0[2][2], B1[2][2];
#pragma unroll 1
    while (q < NTILE) {
        int cur = t_tile(p, q), pend = -1;
        unsigned cA = img + (unsigned)(cur & 15) * IMG_B, cB = img + (unsigned)(NPAN + (cur >> 4)) * IMG_B;
        STAMP(2 + 4 * sn);
        if (wid == 0) {
            unsigned pv;
            if (!wave_wait_ge<true>(POLL_WORD(cur), lane < 32 ? want : 0u, ctl + C_TMO, call, pv)) { if (lane == 0) *badf = 1u; }
            if (lane == 32) gof[1] = pv + 2u;
            __builtin_amdgcn_fence(__ATOMIC_ACQUIRE, "");
            asm volatile("s_waitcnt vmcnt(0) lgkmcnt(0)" ::: "memory");
        }
        asm volatile("" ::: "memory"); BAR; asm volatile("" ::: "memory");
        unsigned v0 = KCLK ? __builtin_amdgcn_readfirstlane(gof[1]) & ~1u : 0u;
        {
            const unsigned a0 = cA + (v0 & 63u) * kstep, b0 = cB + (v0 & 63u) * kstep;
            STAGE(SB(0, 0), b0); STAGE(SA(0, 0), a0); STAGE(SB(0, 1), b0 + hstep); STAGE(SA(0, 1), a0 + hstep);
            if (wr == 1) BAR;
            WAIT_V(4); BAR;
            STAGE(SB(1, 0), b0 + kstep); STAGE(SA(1, 0), a0 + kstep); STAGE(SB(1, 1), b0 + hstep + kstep);
            WAIT_V(6); BAR;
        }
        for (;;) {
            STAMP(3 + 4 * sn); STAMPC(100 + 2 * sn);
#pragma unroll
            for (int a = 0; a < 2; ++a)
#pragma unroll
                for (int b = 0; b < 2; ++b)
#pragma unroll
                    for (int m = 0; m < 4; ++m)
#pragma unroll
                        for (int n = 0; n < 2; ++n) acc[a][b][m][n] = (f32x4){0.f, 0.f, 0.f, 0.f};
            const bool has_next = q + NCOMP < NTILE;
            const int nxt = has_next ? t_tile(p, q + NCOMP) : cur;
            unsigned nA = cA, nB = cB, nv0 = 0;
            bool go = false;
#pragma unroll 1
            for (int t = 0; t < nt; t += 2) {
                const bool last = (t == nt - 2), probe = (t == nt - 4) && wid == 0;
                if (last) {
                    go = has_next && __builtin_amdgcn_readfirstlane(*gof) != 0u;
                    if (go) { nA = img + (unsigned)(nxt & 15) * IMG_B; nB = img + (unsigned)(NPAN + (nxt >> 4)) * IMG_B; }
                    if (KCLK) nv0 = __builtin_amdgcn_readfirstlane(gof[1]) & ~1u;
                }
                const unsigned ko = ((v0 + (unsigned)t) & 63u) * kstep, k2 = ((last ? nv0 : v0 + (unsigned)t + 2u) & 63u) * kstep;
                const unsigned a1 = cA + ko + kstep;
                const unsigned a2 = (last ? nA : cA) + k2, b2 = (last ? nB : cB) + k2;
                const unsigned a3 = a2 + kstep, b3 = b2 + kstep;
                unsigned pv = 0;
                if (KCLK && tid == 0) __hip_atomic_fetch_max(ctl + kclk_idx, v0 + (unsigned)t, RLX_AGENT);
                if (probe) asm volatile("global_load_dword %0, %1, off sc0 sc1" : "=v"(pv) : "v"(POLL_WORD(nxt)) : "memory");
                if (t == 2 && pend >= 0) { if (tid == 0) __hip_atomic_store(ctl + C_YF + pend, call, RLX_AGENT); pend = -1; }
                LDB(B0, 0, 0); SCHED; LDA(At, 0, 0); STAGE(SA(1, 1), a1 + hstep);
                WAIT_L(8); BAR; WAIT_L(0); MMA(0, 0, At, B0); BAR; SCHED;
                LDB(B1, 0, 1); STAGE(SB(0, 0), b2);
                BAR; WAIT_L(0); MMA(0, 1, At, B1); BAR;
                LDA(At, 0, 1); STAGE(SA(0, 0), a2);
                BAR; WAIT_L(0); MMA(1, 0, At, B0); BAR; SCHED;
                STAGE(SB(0, 1), b2 + hstep);
                WAIT_V(6);
                if (probe) {
                    asm volatile("buffer_inv sc0 sc1" : "+v"(pv) :: "memory");
                    const bool ok = __all(lane >= 32 || pv >= want);
                    if (lane == 0) gof[0] = ok;
                    if (lane == 32) gof[1] = pv + 4u;
                }
                BAR; MMA(1, 1, At, B1); BAR;
                LDB(B0, 1, 0); SCHED; LDA(At, 1, 0); STAGE(SA(0, 1), a2 + hstep);
                WAIT_L(8); BAR; WAIT_L(0); MMA(0, 0, At, B0); BAR; SCHED;
                LDB(B1, 1, 1); STAGE(SB(1, 0), b3);
                BAR; WAIT_L(0); MMA(0, 1, At, B1); BAR;
                LDA(At, 1, 1); STAGE(SA(1, 0), a3);
                BAR; WAIT_L(0); MMA(1, 0, At, B0); BAR; SCHED;
                STAGE(SB(1, 1), b3 + hstep);
                WAIT_V(6); BAR; MMA(1, 1, At, B1); BAR;
            }
            {
                STAMP(4 + 4 * sn); STAMPC(101 + 2 * sn);
                const bool bad = *badf != 0u;
                const __amdgpu_buffer_rsrc_t ro = mkrsrc(me.out, OUT_B);
                const unsigned so = (unsigned)((cur & 15) * 256 + wr * 64) * (NN * 2) + (unsigned)gblock(rank, cur >> 4) * 512u + (unsigned)wc * 128u;
                const unsigned vo = (unsigned)fr * (NN * 2) + (unsigned)fq * 16u;
#pragma unroll
                for (int ai = 0; ai < 2; ++ai)
#pragma unroll
                    for (int m = 0; m < 4; ++m)
#pragma unroll
                        for (int bj = 0; bj < 2; ++bj) {
                            const f32x4 v0 = acc[ai][bj][m][0], v1 = acc[ai][bj][m][1];
                            u32x4 w;
                            w.x = cvt_pk_bf16(silu(v0[0]), silu(v0[1])); w.y = cvt_pk_bf16(silu(v0[2]), silu(v0[3]));
                            w.z = cvt_pk_bf16(silu(v1[0]), silu(v1[1])); w.w = cvt_pk_bf16(silu(v1[2]), silu(v1[3]));
                            if (bad) w = (u32x4){0x7fc07fc0u, 0x7fc07fc0u, 0x7fc07fc0u, 0x7fc07fc0u};
                            __builtin_amdgcn_raw_buffer_store_b128(w, ro, vo + bj * 64, so + (unsigned)(ai * 128 + m * 16) * (NN * 2), 16);
                        }
                if ((cur >> 4) < DB) pend = t_ypos(p, (cur & 15) * DB + (cur >> 4));
                STAMP(5 + 4 * sn);
                ++sn;
            }
            q += NCOMP;
            if (!go) break;
            cur = nxt; cA = nA; cB = nB; v0 = nv0;
        }
        WAIT_V(0);
        if (wr == 0) BAR;
        BAR;
        if (pend >= 0 && tid == 0) __hip_atomic_store(ctl + C_YF + pend, call, RLX_AGENT);
    }
    STAMP(250);
}

__global__ __launch_bounds__(512, 2) void fused_gemm_silu(const Params params) {
    KP p = (KP)__builtin_amdgcn_kernarg_segment_ptr();
    extern __shared__ __attribute__((aligned(16))) unsigned char shm[];
    LAS unsigned char* lds = (LAS unsigned char*)shm;
    LAS unsigned* ctlw = (LAS unsigned*)(lds + STAGE_BYTES);
    Self me;
    me.out = uni(p->out_self); me.ws = uni(p->ws_self); me.rank = __builtin_amdgcn_readfirstlane(p->rank);
    STAMP(0);
    gu32* cc = (gu32*)me.ws + C_CALL + blockIdx.x * LINE;
    if (threadIdx.x < 128) ctlw[threadIdx.x] = 0u;
    if (threadIdx.x == 128) { const unsigned c = __hip_atomic_load(cc, RLX_AGENT) + 1u; ctlw[128] = c; __hip_atomic_store(cc, c, RLX_AGENT); }
    __syncthreads();
    me.call = __builtin_amdgcn_readfirstlane(ctlw[128]);
    STAMP(1);
    if (blockIdx.x < NPUSH) {
        pusher(p, me, ctlw);
    } else {
        stage_w(p, me);
        compute(p, me, lds);
    }
}

void build_tables(Params& pp) {
    unsigned char *uorder = (unsigned char*)pp.uorder4, *yorder = (unsigned char*)pp.yorder4, *ypos = (unsigned char*)pp.ypos4;
    unsigned short* tile = (unsigned short*)pp.tile2;
    for (int i = 0; i < 80; ++i) uorder[i] = 0;
    for (int i = 0; i < 128; ++i) yorder[i] = ypos[i] = 0;
    int n = 0, nx = 0, nw = DB, upos[NIN];
    while (n < NBU) {
        if (nx < NPAN && (nx <= nw || nw == NLB)) uorder[n++] = nx++;
        else uorder[n++] = NPAN + nw++;
    }
    for (int jb = 0; jb < DB; ++jb) upos[NPAN + jb] = -1;
    for (int i = 0; i < NBU; ++i) upos[uorder[i]] = i;
    int t = 0, y = 0, nq[8] = {0, 0, 0, 0, 0, 0, 0, 0};
    for (int pos = 0; pos < NBU; ++pos)
        for (int jb = 0; jb < NLB; ++jb)
            for (int i = 0; i < NPAN; ++i) {
                const int key = upos[i] > upos[NPAN + jb] ? upos[i] : upos[NPAN + jb];
                if (key != pos) continue;
                if (XMAP && NLB % 2 == 0) { const int x = (i & 3) + 4 * (jb & 1); tile[nq[x]++ * 8 + x] = (unsigned short)(i | jb << 4); }
                else tile[t++] = (unsigned short)(i | jb << 4);
                if (jb < DB) { ypos[i * DB + jb] = (unsigned char)y; yorder[y++] = (unsigned char)(i * DB + jb); }
            }
}
}

extern "C" void kernel_launch(int rank, int nranks, ncclComm_t comm, hipStream_t stream,
                              void* const* d_in, const int* in_sizes, int n_in,
                              void* d_out, int out_size, const int* in_dtypes, int out_dtype,
                              void* const* d_out_all, void* const* d_ws_all, size_t ws_bytes) {
    if (nranks != NR || (unsigned)rank >= (unsigned)NR || n_in < 2 || in_sizes[0] != MM * KS || in_sizes[1] != KS * NN ||
        out_size != MM * NN || ws_bytes < WS_NEED) {
        fprintf(stderr, "unexpected problem shape\n");
        return;
    }
    static bool attr_set = false;
    if (!attr_set) {
        HIP_CHECK(hipFuncSetAttribute((const void*)fused_gemm_silu, hipFuncAttributeMaxDynamicSharedMemorySize, LDS_BYTES));
        attr_set = true;
    }
    Params p;
    p.x = (const char*)d_in[0]; p.w = (const char*)d_in[1]; p.rank = rank;
    p.out_self = (char*)d_out_all[rank]; p.ws_self = (char*)d_ws_all[rank];
    for (int r = 0; r < NR; ++r) { p.buf[NR + r] = (char*)d_out_all[r]; p.buf[r] = (char*)d_ws_all[r]; }
    p.buf[2 * NR] = (char*)d_in[0];
    build_tables(p);
    hipLaunchKernelGGL(fused_gemm_silu, dim3(NWG), dim3(512), LDS_BYTES, stream, p);
    HIP_CHECK(hipGetLastError());
}
```

```cpp
#include <hip/hip_runtime.h>
#include <rccl/rccl.h>
#include <cstdio>
#include <cstdint>

#define HIP_CHECK(expr) do { hipError_t _e = (expr); if (_e != hipSuccess) { \
    fprintf(stderr, "HIP error %s at %s:%d\n", hipGetErrorString(_e), __FILE__, __LINE__); \
    return; } } while (0)

#define NRB 32
#define NPULL 56
#define NPRE 1
#define CH 8
#define NURG 3
#define STDELAY 300
#define XSLACK 4.f
#define HELP 0
#define KCLK 1
#define XMAP 1

namespace {
#define GAS __attribute__((address_space(1)))
#define LAS __attribute__((address_space(3)))
typedef short bf16x8 __attribute__((ext_vector_type(8)));
typedef float f32x4 __attribute__((ext_vector_type(4)));
typedef unsigned u32x4 __attribute__((ext_vector_type(4)));
typedef GAS unsigned gu32;

constexpr int NR = 8, MM = 4096, KK = 4096, NN = 16384, KS = KK / NR;
constexpr int NPAN = MM / 256, NBLK = NN / 256;
constexpr int DB = (NBLK - NRB) / NR;
constexpr int NLB = DB + NRB;
constexpr int NIN = NPAN + NLB;
constexpr int NTILE = NPAN * NLB, ND = NPAN * DB;
constexpr int NWG = 256, NCOMP = NWG - NPULL;
constexpr int UCH = 256 / CH, NCH_IN = NIN * UCH, YCH = 128 * DB / CH, NCH = NCH_IN + NPAN * YCH;
constexpr int NSTASK = (NPAN - NPRE + NBLK) * 4 + NPRE * 32;
constexpr int MAXT = 4;
static_assert(NRB % 8 == 0 && DB >= 1 && NPULL % 8 == 0 && NSTASK <= 2 * NCOMP && NTILE <= MAXT * NCOMP && (CH == 2 || CH == 4 || CH == 8 || CH == 16), "config");
constexpr unsigned CHUNK_B = 256u << 10;
constexpr unsigned IMG_B = 2u << 20;
constexpr unsigned CTRL_B = 1u << 20;
constexpr unsigned SCR_OFF = CTRL_B + 2u * NIN * IMG_B;
constexpr size_t WS_NEED = (size_t)SCR_OFF + (size_t)NR * DB * CHUNK_B;
constexpr unsigned OUT_B = (unsigned)((size_t)MM * NN * 2);
constexpr int LINE = 32;
constexpr int C_CALL = 0;
constexpr int C_CNT = C_CALL + NWG * LINE;
constexpr int C_AV = C_CNT + 2 * 80 * 8 * LINE;
constexpr int C_TMO = C_AV + 2 * 8 * 192;
constexpr int C_CLAIM = C_TMO + LINE;
constexpr int C_KCLK = C_CLAIM + 16 * LINE;
static_assert((C_KCLK + 16 * LINE) * 4 <= (512 << 10) && NIN <= 80 && NIN + NPAN + 64 <= 192, "ctrl");
constexpr unsigned long long DEADLINE_TICKS = 50000000ull;
constexpr int HTB = 16384, STAGE_BYTES = 8 * HTB, LDS_BYTES = STAGE_BYTES + 1024;

struct Params {
    const char* x; const char* w;
    char* out_self; char* ws_self;
    char* buf[2 * NR];
    int rank;
    unsigned uorder4[80 / 4];
    unsigned upos4[80 / 4];
    unsigned stask2[NWG];
    unsigned tile2[NCOMP * MAXT / 2 + 1];
};
typedef const __attribute__((address_space(4))) Params* KP;
__device__ __forceinline__ int t_uorder(KP p, int i) { return (p->uorder4[i >> 2] >> ((i & 3) * 8)) & 255; }
__device__ __forceinline__ int t_upos(KP p, int i) { return (p->upos4[i >> 2] >> ((i & 3) * 8)) & 255; }
__device__ __forceinline__ int t_stask(KP p, int i) { return (p->stask2[i >> 1] >> ((i & 1) * 16)) & 65535; }
__device__ __forceinline__ int t_tile(KP p, int i) { return (p->tile2[i >> 1] >> ((i & 1) * 16)) & 65535; }

#define RLX_AGENT __ATOMIC_RELAXED, __HIP_MEMORY_SCOPE_AGENT
#define RLX_SYS __ATOMIC_RELAXED, __HIP_MEMORY_SCOPE_SYSTEM
#define WAIT_V(n) asm volatile("s_waitcnt vmcnt(" #n ")" ::: "memory")
#define WAIT_L(n) asm volatile("s_waitcnt lgkmcnt(" #n ")" ::: "memory")
#define WAIT_VC(n) do { asm volatile("" ::: "memory"); __builtin_amdgcn_s_waitcnt(0x0F70 | ((n) & 15) | (((n) >> 4) << 14)); asm volatile("" ::: "memory"); } while (0)
#define BAR __builtin_amdgcn_s_barrier()
#define SCHED __builtin_amdgcn_sched_barrier(0)

#define STAMP(slot) do { } while (0)
#define STAMPC(slot) do { } while (0)
typedef GAS char gchar;
typedef const GAS u32x4 gvec;
__device__ __forceinline__ gchar* uni(const void* p) {
    const unsigned long long v = (unsigned long long)p;
    const unsigned lo = __builtin_amdgcn_readfirstlane((unsigned)v), hi = __builtin_amdgcn_readfirstlane((unsigned)(v >> 32));
    return (gchar*)(((unsigned long long)hi << 32) | lo);
}
struct Self { gchar* out; gchar* ws; int rank; unsigned call; };
__device__ __forceinline__ int cnt_idx(unsigned par, int uid, int sender) { return C_CNT + (((int)par * 80 + uid) * 8 + sender) * LINE; }
__device__ __forceinline__ int av_idx(unsigned par, int source, int pos) { return C_AV + ((int)par * 8 + source) * 192 + pos; }
__device__ __forceinline__ unsigned img_off(unsigned par, unsigned uid) { return CTRL_B + (par * NIN + uid) * IMG_B; }
template <class P> __device__ __forceinline__ __amdgpu_buffer_rsrc_t mkrsrc(P p, unsigned bytes) {
    return __builtin_amdgcn_make_buffer_rsrc((void*)p, 0, bytes, 0x00020000);
}
#define ST128(v, rs, vo, so, aux) __builtin_amdgcn_raw_buffer_store_b128(v, rs, (vo) + (so), 0, aux)
__device__ __forceinline__ unsigned cvt_pk_bf16(float lo, float hi) { unsigned r; asm("v_cvt_pk_bf16_f32 %0, %1, %2" : "=v"(r) : "v"(lo), "v"(hi)); return r; }
__device__ __forceinline__ float silu(float y) {
    const float c = __builtin_amdgcn_fmed3f(y, -60.0f, 60.0f);
    return y * __builtin_amdgcn_rcpf(1.0f + __builtin_amdgcn_exp2f(c * -1.44269504089f));
}
__device__ __forceinline__ int gblock(int rank, int jb) { return jb < DB ? rank * DB + jb : NR * DB + (jb - DB); }

template <bool SYS> __device__ __forceinline__ bool wave_wait_ge(gu32* word, unsigned want, gu32* tmo, unsigned call, unsigned& v) {
    const unsigned long long t0 = __builtin_amdgcn_s_memrealtime();
    for (;;) {
        v = SYS ? __hip_atomic_load(word, RLX_SYS) : __hip_atomic_load(word, RLX_AGENT);
        if (__all(v >= want)) return true;
        if ((unsigned)__builtin_amdgcn_readfirstlane(__hip_atomic_load(tmo, RLX_AGENT)) == call ||
            __builtin_amdgcn_s_memrealtime() - t0 > DEADLINE_TICKS) {
            __hip_atomic_store(tmo, call, RLX_AGENT);
            return false;
        }
        __builtin_amdgcn_s_sleep(4);
    }
}

template <bool SYS> __device__ __forceinline__ bool wave_wait_ge(gu32* word, unsigned want, gu32* tmo, unsigned call) {
    unsigned v;
    return wave_wait_ge<SYS>(word, want, tmo, call, v);
}

__device__ __forceinline__ int chunk_pos(int c) { return c < NCH_IN ? c / UCH : NIN + (c - NCH_IN) / YCH; }
__device__ __forceinline__ unsigned y_voff(int c, int k, int lane) {
    const unsigned o = (unsigned)((c - NCH_IN) % YCH * CH + k) * 1024u + (unsigned)lane * 16u;
    return (o / (DB * 512u)) * (NN * 2) + o % (DB * 512u);
}
__device__ __forceinline__ void puller(KP p, const Self me) {
    const unsigned call = me.call, par = call & 1u, want = ((call + 1u) >> 1) * (4u * DB);
    const int tid = threadIdx.x, lane = tid & 63, wid = __builtin_amdgcn_readfirstlane(tid >> 6), rank = me.rank;
    const int gw = blockIdx.x * 8 + wid;
    gu32* ctl = (gu32*)me.ws;
    gu32* tmo = ctl + C_TMO;
    const unsigned l16 = (unsigned)lane * 16u;
    bool bad = false;
    int sn = 0, j = gw % (NR - 1);
    const __amdgpu_buffer_rsrc_t rc = mkrsrc(me.ws, CTRL_B);
    const int one = lane == 0;
#define ATOM_OFF(idx) (lane == 0 ? (unsigned)(idx) * 4u : CTRL_B)
#pragma unroll 1
    for (;;) {
        const int src = (rank + 1 + j) & (NR - 1);
        const char* s_ws = p->buf[src]; const char* s_out = p->buf[NR + src];
        const unsigned head = ATOM_OFF(C_CLAIM + ((int)par * 8 + j) * LINE);
        int ready = 0;
        unsigned prev = CTRL_B;
        int c = __builtin_amdgcn_readfirstlane(__builtin_amdgcn_raw_ptr_buffer_atomic_add_i32(one, rc, head, 0, 0)) + NPRE * UCH;
#pragma unroll 1
        while (c < NCH) {
            const int pos = chunk_pos(c);
            if (pos >= ready) {
                WAIT_VC(0);
                __builtin_amdgcn_raw_ptr_buffer_atomic_add_i32(one * CH, rc, prev, 0, 0);
                prev = CTRL_B;
                const unsigned long long t0 = __builtin_amdgcn_s_memrealtime();
                for (;;) {
                    const unsigned long long m = ~__ballot(__hip_atomic_load(ctl + av_idx(par, src, pos + lane), RLX_SYS) >= want);
                    const int n = m ? __builtin_ctzll(m) : 64;
                    if (n > 0) { ready = pos + n; break; }
                    if ((unsigned)__builtin_amdgcn_readfirstlane(__hip_atomic_load(tmo, RLX_AGENT)) == call ||
                        __builtin_amdgcn_s_memrealtime() - t0 > DEADLINE_TICKS) {
                        __hip_atomic_store(tmo, call, RLX_AGENT); bad = true; ready = 1 << 20; break;
                    }
                    __builtin_amdgcn_s_sleep(4);
                }
                asm volatile("" ::: "memory");
            }
            const bool isy = c >= NCH_IN;
            const int uid = isy ? 0 : t_uorder(p, pos);
            const unsigned po = (unsigned)(c % UCH) * (CH * 1024u);
            const unsigned dso = isy ? (unsigned)(pos - NIN) * (256u * NN * 2) + (unsigned)(src * DB) * 512u : img_off(par, uid) + src * CHUNK_B + po;
            const unsigned sso = (unsigned)(uid - NPAN) < (unsigned)DB ? SCR_OFF + (unsigned)(rank * DB + uid - NPAN) * CHUNK_B + po : dso;
            const __amdgpu_buffer_rsrc_t rs = mkrsrc(isy ? s_out : s_ws, bad ? 0u : isy ? OUT_B : (unsigned)WS_NEED);
            const __amdgpu_buffer_rsrc_t rd = mkrsrc(isy ? me.out : me.ws, isy ? OUT_B : (unsigned)WS_NEED);
            int cv = 0;
#pragma unroll
            for (int k = 0; k < CH; ++k) {
                const unsigned vo = isy ? y_voff(c, k, lane) : l16 + k * 1024u;
                u32x4 v = __builtin_amdgcn_raw_buffer_load_b128(rs, vo, sso, 17);
                asm volatile("" ::: "memory");
                if (k == 0) cv = __builtin_amdgcn_raw_ptr_buffer_atomic_add_i32(one, rc, head, 0, 0);
                if (k == 1) __builtin_amdgcn_raw_ptr_buffer_atomic_add_i32(one * CH, rc, prev, 0, 0);
                if (k < 2) WAIT_VC(1); else WAIT_VC(0);
                if (bad) v = (u32x4){0x7fc07fc0u, 0x7fc07fc0u, 0x7fc07fc0u, 0x7fc07fc0u};
                ST128(v, rd, vo, dso, 16);
                asm volatile("" ::: "memory");
            }
            prev = isy ? CTRL_B : ATOM_OFF(cnt_idx(par, uid, src));
            c = __builtin_amdgcn_readfirstlane(cv) + NPRE * UCH;
            STAMP(2 + (sn < 238 ? sn : 238));
            ++sn;
        }
        const unsigned hv = __hip_atomic_load(ctl + C_CLAIM + ((int)par * 8 + (j + 1 + lane) % (NR - 1)) * LINE, RLX_AGENT);
        const unsigned long long left = __ballot(lane < NR - 1 && (int)hv + NPRE * UCH + 8 < NCH);
        WAIT_VC(0);
        __builtin_amdgcn_raw_ptr_buffer_atomic_add_i32(one * CH, rc, prev, 0, 0);
        if (!left || !HELP) break;
        j = (j + 1 + __builtin_ctzll(left)) % (NR - 1);
    }
    STAMP(250);
}

__device__ __forceinline__ unsigned bpos(unsigned fq, unsigned q, unsigned r) { return r * 256u + (((fq * 4u + q) ^ ((r & 1u) | ((r & 2u) << 2))) * 16u); }

__device__ __forceinline__ void stage(KP p, const Self me) {
    const unsigned call = me.call, par = call & 1u;
    const int tid = threadIdx.x, lane = tid & 63, wid = __builtin_amdgcn_readfirstlane(tid >> 6), rank = me.rank;
    const int fq = lane >> 4, c = lane & 15;
    const __amdgpu_buffer_rsrc_t rs = mkrsrc(me.ws, (unsigned)WS_NEED);
#pragma unroll 1
    for (int rnd = 0; rnd < 2; ++rnd) {
        const int code = t_stask(p, blockIdx.x * 2 + rnd);
        if (code == 0xffff) break;
        const int un = (code & 0x3fff) >> 2, task = (code & 3) * 8 + wid;
        int uid, dest = -1;
        u32x4 v[8];
        if (code & 0x4000) {
            const unsigned long long t0 = __builtin_amdgcn_s_memrealtime();
            while (__builtin_amdgcn_s_memrealtime() - t0 < STDELAY) __builtin_amdgcn_s_sleep(8);
        }
        if (code & 0x8000) {
            const int g = (code & 0x3fff) * 8 + wid, pc = g & 255;
            uid = g >> 8;
            v[0] = *(gvec*)((const gchar*)p->x + ((size_t)(uid * 256 + ((pc >> 1) & 15) * 16 + c) * KS + (pc >> 5) * 64 + (pc & 1) * 32 + fq * 8) * 2);
            const unsigned base = img_off(par, uid) + rank * CHUNK_B + (unsigned)pc * 1024u;
#pragma unroll
            for (int d = 0; d < NR; ++d) ST128(v[0], mkrsrc(p->buf[(rank + 1 + g + d) & 7], (unsigned)WS_NEED), (unsigned)lane * 16u, base, 17);
        } else if (un < NPAN) {
            uid = un;
            const gchar* src = (const gchar*)p->x + ((size_t)(un * 256 + (task & 3) * 64 + c) * KS + (task >> 2) * 64 + fq * 8) * 2;
#pragma unroll
            for (int j = 0; j < 8; ++j) v[j] = *(gvec*)(src + (j >> 1) * (16 * KS * 2) + (j & 1) * 64);
            const unsigned base = img_off(par, uid) + rank * CHUNK_B + (unsigned)task * 8192u;
#pragma unroll
            for (int j = 0; j < 8; ++j) ST128(v[j], rs, (unsigned)lane * 16u + j * 1024u, base, 17);
        } else {
            const int B = un - NPAN, ktl = task >> 2, ks = (task >> 1) & 1, g = task & 1;
            const gchar* src = (const gchar*)p->w + ((size_t)(ktl * 64 + ks * 32 + fq * 8) * NN + B * 256 + g * 128 + c * 8) * 2;
#pragma unroll
            for (int j = 0; j < 8; ++j) v[j] = *(gvec*)(src + (size_t)j * NN * 2);
            if (B < NR * DB) { dest = B / DB; uid = NPAN + B % DB; } else uid = NPAN + DB + (B - NR * DB);
            const unsigned base = (dest >= 0 && dest != rank ? SCR_OFF + (unsigned)B * CHUNK_B : img_off(par, uid) + rank * CHUNK_B)
                                  + (unsigned)ktl * 32768u + (unsigned)ks * 1024u;
            const unsigned lb = (unsigned)((c >> 2) & 1) * HTB + (unsigned)(2 * g + (c >> 3)) * 4096u;
#pragma unroll
            for (int e = 0; e < 8; ++e) {
                u32x4 o;
#pragma unroll
                for (int d = 0; d < 4; ++d)
                    o[d] = __builtin_amdgcn_perm(v[2 * d + 1][e >> 1], v[2 * d][e >> 1], (e & 1) ? 0x07060302u : 0x05040100u);
                ST128(o, rs, lb + (unsigned)(e >> 2) * 2048u + bpos(fq, c & 3, e & 3), base, 17);
            }
        }
        WAIT_V(0);
        __syncthreads();
        if (tid < NR && (dest < 0 || dest == tid)) {
            const bool landed = tid == rank || (code & 0x8000);
            __hip_atomic_fetch_add((gu32*)p->buf[tid] + (landed ? cnt_idx(par, uid, rank) : av_idx(par, rank, t_upos(p, uid))),
                                   (code & 0x8000) ? 8u : landed ? 64u : (unsigned)DB, RLX_SYS);
        }
    }
}

__device__ __forceinline__ void compute(KP p, const Self me, LAS unsigned char* lds) {
    const unsigned call = me.call;
    const int tid = threadIdx.x, wid = __builtin_amdgcn_readfirstlane(tid >> 6), lane = tid & 63, wr = wid >> 2, wc = wid & 3, fr = lane & 15, fq = lane >> 4;
    const int rank = me.rank, cwg = blockIdx.x - NPULL;
    const unsigned par = call & 1u, want = ((call + 1u) >> 1) * 256u;
    gu32* peer_av = (gu32*)p->buf[lane & 7] + av_idx(par, rank, NIN);
    gu32* ctl = (gu32*)me.ws;
    LAS unsigned* badf = (LAS unsigned*)(lds + STAGE_BYTES);
    const __amdgpu_buffer_rsrc_t rw = mkrsrc(me.ws, (unsigned)WS_NEED);
    const unsigned img = img_off(par, 0);
    const unsigned voff = (unsigned)tid * 16u, ldsw = (unsigned)wid * 1024u;
    const int aoff = wr * 8192 + lane * 16, boff = wc * 4096 + (int)bpos(fq, fr >> 2, fr & 3);
    constexpr unsigned kstep = 2 * HTB, hstep = HTB;
#define SA(b, h) (((b) * 2 + (h)) * HTB)
#define SB(b, h) ((4 + (b) * 2 + (h)) * HTB)
#define STAGE(bufoff, gbase) do { _Pragma("unroll") for (int _i = 0; _i < 2; ++_i) \
        __builtin_amdgcn_raw_ptr_buffer_load_lds(rw, (LAS void*)(lds + (bufoff) + ldsw + _i * 8192), 16, voff, (gbase) + _i * 8192, 0, 0); } while (0)
#define LDA(dst, b, h) do { _Pragma("unroll") for (int m = 0; m < 4; ++m) _Pragma("unroll") for (int k = 0; k < 2; ++k) dst[m][k] = *(const LAS bf16x8*)(lds + SA(b, h) + aoff + m * 2048 + k * 1024); } while (0)
#define LDB(dst, b, h) do { _Pragma("unroll") for (int n = 0; n < 2; ++n) _Pragma("unroll") for (int k = 0; k < 2; ++k) dst[n][k] = *(const LAS bf16x8*)(lds + SB(b, h) + boff + n * 2048 + k * 1024); } while (0)
#define MMA(ai, bj, At, Bt) do { __builtin_amdgcn_s_setprio(1); _Pragma("unroll") for (int m = 0; m < 4; ++m) _Pragma("unroll") for (int n = 0; n < 2; ++n) _Pragma("unroll") for (int k = 0; k < 2; ++k) \
        acc[ai][bj][m][n] = __builtin_amdgcn_mfma_f32_16x16x32_bf16(Bt[n][k], At[m][k], acc[ai][bj][m][n], 0, 0, 0); __builtin_amdgcn_s_setprio(0); } while (0)
    const int kclk_idx = C_KCLK + (int)(__builtin_amdgcn_s_getreg(20 | (3 << 11)) & 15u) * LINE;
#define POLL_WORD(tl) (ctl + (lane < 32 ? cnt_idx(par, (lane & 8) ? NPAN + ((tl) >> 4) : ((tl) & 15), lane & 7) : kclk_idx))
    LAS unsigned* gof = badf + 1;
    constexpr int nt = KK / 64;
    int q = cwg * MAXT, sn = 0;
    f32x4 acc[2][2][4][2];
    bf16x8 At[4][2], B0[2][2], B1[2][2];
#pragma unroll 1
    while (q < (cwg + 1) * MAXT && t_tile(p, q) != 0xffff) {
        int cur = t_tile(p, q), pend = -1;
        unsigned cA = img + (unsigned)(cur & 15) * IMG_B, cB = img + (unsigned)(NPAN + (cur >> 4)) * IMG_B;
        STAMP(2 + 4 * sn);
        if (wid == 0) {
            unsigned pv;
            if (!wave_wait_ge<true>(POLL_WORD(cur), lane < 32 ? want : 0u, ctl + C_TMO, call, pv)) { if (lane == 0) *badf = 1u; }
            if (lane == 32) gof[1] = pv + 2u;
            __builtin_amdgcn_fence(__ATOMIC_ACQUIRE, "");
            asm volatile("s_waitcnt vmcnt(0) lgkmcnt(0)" ::: "memory");
        }
        asm volatile("" ::: "memory"); BAR; asm volatile("" ::: "memory");
        unsigned v0 = KCLK ? __builtin_amdgcn_readfirstlane(gof[1]) & ~1u : 0u;
        {
            const unsigned a0 = cA + (v0 & 63u) * kstep, b0 = cB + (v0 & 63u) * kstep;
            STAGE(SB(0, 0), b0); STAGE(SA(0, 0), a0); STAGE(SB(0, 1), b0 + hstep); STAGE(SA(0, 1), a0 + hstep);
            if (wr == 1) BAR;
            WAIT_V(4); BAR;
            STAGE(SB(1, 0), b0 + kstep); STAGE(SA(1, 0), a0 + kstep); STAGE(SB(1, 1), b0 + hstep + kstep);
            WAIT_V(6); BAR;
        }
        for (;;) {
            STAMP(3 + 4 * sn); STAMPC(100 + 2 * sn);
#pragma unroll
            for (int a = 0; a < 2; ++a)
#pragma unroll
                for (int b = 0; b < 2; ++b)
#pragma unroll
                    for (int m = 0; m < 4; ++m)
#pragma unroll
                        for (int n = 0; n < 2; ++n) acc[a][b][m][n] = (f32x4){0.f, 0.f, 0.f, 0.f};
            const bool has_next = q + 1 < (cwg + 1) * MAXT && t_tile(p, q + 1) != 0xffff;
            const int nxt = has_next ? t_tile(p, q + 1) : cur;
            unsigned nA = cA, nB = cB, nv0 = 0;
            bool go = false;
#pragma unroll 1
            for (int t = 0; t < nt; t += 2) {
                const bool last = (t == nt - 2), probe = (t == nt - 4) && wid == 0;
                if (last) {
                    go = has_next && __builtin_amdgcn_readfirstlane(*gof) != 0u;
                    if (go) { nA = img + (unsigned)(nxt & 15) * IMG_B; nB = img + (unsigned)(NPAN + (nxt >> 4)) * IMG_B; }
                    if (KCLK) nv0 = __builtin_amdgcn_readfirstlane(gof[1]) & ~1u;
                }
                const unsigned ko = ((v0 + (unsigned)t) & 63u) * kstep, k2 = ((last ? nv0 : v0 + (unsigned)t + 2u) & 63u) * kstep;
                const unsigned a1 = cA + ko + kstep;
                const unsigned a2 = (last ? nA : cA) + k2, b2 = (last ? nB : cB) + k2;
                const unsigned a3 = a2 + kstep, b3 = b2 + kstep;
                unsigned pv = 0;
                if (KCLK && ((t >> 1) + (cwg >> 3) & 3) == 0 && tid == 0) __hip_atomic_fetch_max(ctl + kclk_idx, v0 + (unsigned)t, RLX_AGENT);
                if (probe) asm volatile("global_load_dword %0, %1, off sc0 sc1" : "=v"(pv) : "v"(POLL_WORD(nxt)) : "memory");
                if (t == 2 && pend >= 0) { if (tid < NR && tid != rank) __hip_atomic_fetch_add(peer_av + pend, 4u, RLX_SYS); pend = -1; }
                LDB(B0, 0, 0); SCHED; LDA(At, 0, 0); STAGE(SA(1, 1), a1 + hstep);
                WAIT_L(8); BAR; WAIT_L(0); MMA(0, 0, At, B0); BAR; SCHED;
                LDB(B1, 0, 1); STAGE(SB(0, 0), b2);
                BAR; WAIT_L(0); MMA(0, 1, At, B1); BAR;
                LDA(At, 0, 1); STAGE(SA(0, 0), a2);
                BAR; WAIT_L(0); MMA(1, 0, At, B0); BAR; SCHED;
                STAGE(SB(0, 1), b2 + hstep);
                WAIT_V(6);
                if (probe) {
                    asm volatile("buffer_inv sc0 sc1" : "+v"(pv) :: "memory");
                    const bool ok = __all(lane >= 32 || pv >= want);
                    if (lane == 0) gof[0] = ok;
                    if (lane == 32) gof[1] = pv + 4u;
                }
                BAR; MMA(1, 1, At, B1); BAR;
                LDB(B0, 1, 0); SCHED; LDA(At, 1, 0); STAGE(SA(0, 1), a2 + hstep);
                WAIT_L(8); BAR; WAIT_L(0); MMA(0, 0, At, B0); BAR; SCHED;
                LDB(B1, 1, 1); STAGE(SB(1, 0), b3);
                BAR; WAIT_L(0); MMA(0, 1, At, B1); BAR;
                LDA(At, 1, 1); STAGE(SA(1, 0), a3);
                BAR; WAIT_L(0); MMA(1, 0, At, B0); BAR; SCHED;
                STAGE(SB(1, 1), b3 + hstep);
                WAIT_V(6); BAR; MMA(1, 1, At, B1); BAR;
            }
            {
                STAMP(4 + 4 * sn); STAMPC(101 + 2 * sn);
                const bool bad = *badf != 0u;
                const __amdgpu_buffer_rsrc_t ro = mkrsrc(me.out, OUT_B);
                const unsigned so = (unsigned)((cur & 15) * 256 + wr * 64) * (NN * 2) + (unsigned)gblock(rank, cur >> 4) * 512u + (unsigned)wc * 128u;
                const unsigned vo = (unsigned)fr * (NN * 2) + (unsigned)fq * 16u;
#pragma unroll
                for (int ai = 0; ai < 2; ++ai)
#pragma unroll
                    for (int m = 0; m < 4; ++m)
#pragma unroll
                        for (int bj = 0; bj < 2; ++bj) {
                            const f32x4 v0 = acc[ai][bj][m][0], v1 = acc[ai][bj][m][1];
                            u32x4 w;
                            w.x = cvt_pk_bf16(silu(v0[0]), silu(v0[1])); w.y = cvt_pk_bf16(silu(v0[2]), silu(v0[3]));
                            w.z = cvt_pk_bf16(silu(v1[0]), silu(v1[1])); w.w = cvt_pk_bf16(silu(v1[2]), silu(v1[3]));
                            if (bad) w = (u32x4){0x7fc07fc0u, 0x7fc07fc0u, 0x7fc07fc0u, 0x7fc07fc0u};
                            ST128(w, ro, vo + bj * 64, so + (unsigned)(ai * 128 + m * 16) * (NN * 2), 17);
                        }
                if ((cur >> 4) < DB) pend = cur & 15;
                STAMP(5 + 4 * sn);
                ++sn;
            }
            ++q;
            if (!go) break;
            cur = nxt; cA = nA; cB = nB; v0 = nv0;
        }
        WAIT_V(0);
        if (wr == 0) BAR;
        BAR;
        if (pend >= 0 && tid < NR && tid != rank) __hip_atomic_fetch_add(peer_av + pend, 4u, RLX_SYS);
    }
    STAMP(250);
}

__global__ __launch_bounds__(512, 2) void fused_gemm_silu(const Params params) {
    KP p = (KP)__builtin_amdgcn_kernarg_segment_ptr();
    extern __shared__ __attribute__((aligned(16))) unsigned char shm[];
    LAS unsigned char* lds = (LAS unsigned char*)shm;
    LAS unsigned* ctlw = (LAS unsigned*)(lds + STAGE_BYTES);
    Self me;
    me.out = uni(p->out_self); me.ws = uni(p->ws_self); me.rank = __builtin_amdgcn_readfirstlane(p->rank);
    STAMP(0);
    gu32* cc = (gu32*)me.ws + C_CALL + blockIdx.x * LINE;
    if (threadIdx.x < 128) ctlw[threadIdx.x] = 0u;
    if (threadIdx.x == 128) { const unsigned c = __hip_atomic_load(cc, RLX_AGENT) + 1u; ctlw[128] = c; __hip_atomic_store(cc, c, RLX_AGENT); }
    __syncthreads();
    me.call = __builtin_amdgcn_readfirstlane(ctlw[128]);
    STAMP(1);
    if (blockIdx.x == 0 && threadIdx.x < 8)
        __hip_atomic_store((gu32*)me.ws + C_CLAIM + (((me.call & 1u) ^ 1u) * 8 + threadIdx.x) * LINE, 0u, RLX_AGENT);
    stage(p, me);
    if (blockIdx.x < NPULL) puller(p, me);
    else compute(p, me, lds);
}

void build_tables(Params& pp) {
    unsigned char *uorder = (unsigned char*)pp.uorder4, *upos = (unsigned char*)pp.upos4;
    unsigned short *tile = (unsigned short*)pp.tile2, *stask = (unsigned short*)pp.stask2;
    for (int i = 0; i < 80; ++i) uorder[i] = upos[i] = 0;
    int n = 0, nx = 0, nw = 0;
    while (n < NIN) {
        if (nx < NPAN && (nx < NPRE || nx <= nw || nw == NLB)) uorder[n++] = nx++;
        else uorder[n++] = NPAN + nw++;
    }
    unsigned short list[NSTASK];
    int ns = 0, nurg = 0;
    for (int i = 0; i < NIN; ++i) {
        const int u = uorder[i];
        upos[u] = i;
        if (i == NPRE + NURG) nurg = ns;
        for (int q = 0; q < 4 && u >= NPRE; ++q) {
            if (u < NPAN) list[ns++] = u * 4 + q;
            else if (u >= NPAN + DB) list[ns++] = (NPAN + NR * DB + (u - NPAN - DB)) * 4 + q;
            else for (int d = 0; d < NR; ++d) list[ns++] = (NPAN + d * DB + (u - NPAN)) * 4 + q;
        }
    }
    for (int i = 0; i < 2 * NWG; ++i) stask[i] = 0xffff;
    int li = 0, pi = 0;
    for (int b = 0; b < NWG; ++b) if (!(b & 4) && li < nurg) stask[2 * b] = list[li++];
    for (int b = NPULL; b < NWG; ++b) if ((b & 4) && pi < NPRE * 32) stask[2 * b] = 0x8000 | pi++;
    for (int b = NPULL; b < NWG; ++b) if (stask[2 * b] == 0xffff && li < ns) stask[2 * b] = 0x4000 | list[li++];
    for (int b = NWG - (ns - li); b < NWG; ++b) stask[2 * b + 1] = list[li++];
    unsigned short ql[8][NTILE];
    float qe[8][NTILE], freeat[NCOMP];
    int qn[8] = {0, 0, 0, 0, 0, 0, 0, 0}, qh[8] = {0, 0, 0, 0, 0, 0, 0, 0}, cnt[NCOMP], left = NTILE;
    for (int pos = 0; pos < NIN; ++pos)
        for (int jb = 0; jb < NLB; ++jb)
            for (int i = 0; i < NPAN; ++i) {
                const int key = upos[i] > upos[NPAN + jb] ? upos[i] : upos[NPAN + jb];
                if (key != pos) continue;
                const int x = ((i + (i >> 2)) & 3) + 4 * ((jb ^ (jb >> 1)) & 1);
                ql[x][qn[x]] = (unsigned short)(i | jb << 4); qe[x][qn[x]++] = 14.f + 4.4f * pos;
            }
    for (int w = 0; w < NCOMP; ++w) { freeat[w] = 0.f; cnt[w] = 0; }
    for (int i = 0; i < NCOMP * MAXT + 2; ++i) tile[i] = 0xffff;
    for (float t = 0.f; left > 0; t += 0.5f)
        for (int steal = 0; steal < 2; ++steal)
            for (int pass = 0; pass <= MAXT; ++pass)
                for (int w = 0; w < NCOMP; ++w) {
                    if (cnt[w] != pass || cnt[w] >= MAXT || freeat[w] > t) continue;
                    int x = (w + NPULL) % 8;
                    if (steal) {
                        if (qh[x] < qn[x] && qe[x][qh[x]] <= t + XSLACK) continue;
                        int bx = -1;
                        for (int y = 0; y < 8; ++y) {
                            if (qh[y] >= qn[y] || qe[y][qh[y]] > t) continue;
                            float nf = 1e9f;
                            for (int v = 0; v < NCOMP; ++v) if ((v + NPULL) % 8 == y && cnt[v] < MAXT && freeat[v] < nf) nf = freeat[v];
                            if (nf > t + XSLACK && (bx < 0 || qn[y] - qh[y] > qn[bx] - qh[bx])) bx = y;
                        }
                        if (bx < 0) continue;
                        x = bx;
                    } else if (qh[x] >= qn[x] || qe[x][qh[x]] > t) continue;
                    tile[w * MAXT + cnt[w]++] = ql[x][qh[x]++];
                    freeat[w] = t + (t < 90.f ? 72.f : 78.f);
                    --left;
                }
}
}

extern "C" void kernel_launch(int rank, int nranks, ncclComm_t comm, hipStream_t stream,
                              void* const* d_in, const int* in_sizes, int n_in,
                              void* d_out, int out_size, const int* in_dtypes, int out_dtype,
                              void* const* d_out_all, void* const* d_ws_all, size_t ws_bytes) {
    if (nranks != NR || (unsigned)rank >= (unsigned)NR || n_in < 2 || in_sizes[0] != MM * KS || in_sizes[1] != KS * NN ||
        out_size != MM * NN || ws_bytes < WS_NEED) {
        fprintf(stderr, "unexpected problem shape\n");
        return;
    }
    static bool attr_set = false;
    if (!attr_set) {
        HIP_CHECK(hipFuncSetAttribute((const void*)fused_gemm_silu, hipFuncAttributeMaxDynamicSharedMemorySize, LDS_BYTES));
        attr_set = true;
    }
    static Params tables;
    if (!tables.tile2[NCOMP * MAXT / 2]) build_tables(tables);
    Params p = tables;
    p.x = (const char*)d_in[0]; p.w = (const char*)d_in[1]; p.rank = rank;
    p.out_self = (char*)d_out_all[rank]; p.ws_self = (char*)d_ws_all[rank];
    for (int r = 0; r < NR; ++r) { p.buf[NR + r] = (char*)d_out_all[r]; p.buf[r] = (char*)d_ws_all[r]; }
    hipLaunchKernelGGL(fused_gemm_silu, dim3(NWG), dim3(512), LDS_BYTES, stream, p);
    HIP_CHECK(hipGetLastError());
}
```

```cpp
#include <hip/hip_runtime.h>
#include <rccl/rccl.h>
#include <cstdio>
#include <cstdint>

#define HIP_CHECK(expr) do { hipError_t _e = (expr); if (_e != hipSuccess) { \
    fprintf(stderr, "HIP error %s at %s:%d\n", hipGetErrorString(_e), __FILE__, __LINE__); \
    return; } } while (0)

#define NRB 32
#define PWL 30
#define NPULL ((7 * PWL + 7) / 8)
#define NPRE 1
#define CH 8
#define NURG 3
#define STDELAY 300
#define XSLACK 4.f
#define TM1 82.f
#define PD 2
#define KCLK 1
#define XMAP 1

namespace {
#define GAS __attribute__((address_space(1)))
#define LAS __attribute__((address_space(3)))
typedef short bf16x8 __attribute__((ext_vector_type(8)));
typedef float f32x4 __attribute__((ext_vector_type(4)));
typedef unsigned u32x4 __attribute__((ext_vector_type(4)));
typedef GAS unsigned gu32;

constexpr int NR = 8, MM = 4096, KK = 4096, NN = 16384, KS = KK / NR;
constexpr int NPAN = MM / 256, NBLK = NN / 256;
constexpr int DB = (NBLK - NRB) / NR;
constexpr int NLB = DB + NRB;
constexpr int NIN = NPAN + NLB;
constexpr int NTILE = NPAN * NLB, ND = NPAN * DB;
constexpr int NWG = 256, NCOMP = NWG - NPULL;
constexpr int UCH = 256 / CH, NCH_IN = NIN * UCH, YCH = 128 * DB / CH, NCH = NCH_IN + NPAN * YCH;
constexpr int NSTASK = (NPAN - NPRE + NBLK) * 4 + NPRE * 32;
constexpr int MAXT = 4;
static_assert(NRB % 8 == 0 && DB >= 1 && NSTASK <= 2 * NCOMP && NTILE <= MAXT * NCOMP && (CH == 2 || CH == 4 || CH == 8 || CH == 16), "config");
constexpr unsigned CHUNK_B = 256u << 10;
constexpr unsigned IMG_B = 2u << 20;
constexpr unsigned CTRL_B = 1u << 20;
constexpr unsigned SCR_OFF = CTRL_B + 2u * NIN * IMG_B;
constexpr size_t WS_NEED = (size_t)SCR_OFF + (size_t)NR * DB * CHUNK_B;
constexpr unsigned OUT_B = (unsigned)((size_t)MM * NN * 2);
constexpr int LINE = 32;
constexpr int C_CALL = 0;
constexpr int C_CNT = C_CALL + NWG * LINE;
constexpr int C_AV = C_CNT + 2 * 80 * 8 * LINE;
constexpr int C_TMO = C_AV + 2 * 8 * 192;
constexpr int C_CLAIM = C_TMO + LINE;
constexpr int C_KCLK = C_CLAIM + 16 * LINE;
static_assert((C_KCLK + 16 * LINE) * 4 <= (512 << 10) && NIN <= 80 && NIN + NPAN + 64 <= 192, "ctrl");
constexpr unsigned long long DEADLINE_TICKS = 50000000ull;
constexpr int HTB = 16384, STAGE_BYTES = 8 * HTB, LDS_BYTES = STAGE_BYTES + 1024;

struct Params {
    const char* x; const char* w;
    char* out_self; char* ws_self;
    char* buf[2 * NR];
    int rank;
    unsigned uorder4[80 / 4];
    unsigned upos4[80 / 4];
    unsigned stask2[NWG];
    unsigned tile2[NCOMP * MAXT / 2 + 1];
};
typedef const __attribute__((address_space(4))) Params* KP;
__device__ __forceinline__ int t_uorder(KP p, int i) { return (p->uorder4[i >> 2] >> ((i & 3) * 8)) & 255; }
__device__ __forceinline__ int t_upos(KP p, int i) { return (p->upos4[i >> 2] >> ((i & 3) * 8)) & 255; }
__device__ __forceinline__ int t_stask(KP p, int i) { return (p->stask2[i >> 1] >> ((i & 1) * 16)) & 65535; }
__device__ __forceinline__ int t_tile(KP p, int i) { return (p->tile2[i >> 1] >> ((i & 1) * 16)) & 65535; }

#define RLX_AGENT __ATOMIC_RELAXED, __HIP_MEMORY_SCOPE_AGENT
#define RLX_SYS __ATOMIC_RELAXED, __HIP_MEMORY_SCOPE_SYSTEM
#define WAIT_V(n) asm volatile("s_waitcnt vmcnt(" #n ")" ::: "memory")
#define WAIT_L(n) asm volatile("s_waitcnt lgkmcnt(" #n ")" ::: "memory")
#define WAIT_VC(n) do { asm volatile("" ::: "memory"); __builtin_amdgcn_s_waitcnt(0x0F70 | ((n) & 15) | (((n) >> 4) << 14)); asm volatile("" ::: "memory"); } while (0)
#define BAR __builtin_amdgcn_s_barrier()
#define SCHED __builtin_amdgcn_sched_barrier(0)

#define STAMP(slot) do { } while (0)
#define STAMPC(slot) do { } while (0)
typedef GAS char gchar;
typedef const GAS u32x4 gvec;
__device__ __forceinline__ gchar* uni(const void* p) {
    const unsigned long long v = (unsigned long long)p;
    const unsigned lo = __builtin_amdgcn_readfirstlane((unsigned)v), hi = __builtin_amdgcn_readfirstlane((unsigned)(v >> 32));
    return (gchar*)(((unsigned long long)hi << 32) | lo);
}
struct Self { gchar* out; gchar* ws; int rank; unsigned call; };
__device__ __forceinline__ int cnt_idx(unsigned par, int uid, int sender) { return C_CNT + (((int)par * 80 + uid) * 8 + sender) * LINE; }
__device__ __forceinline__ int av_idx(unsigned par, int source, int pos) { return C_AV + ((int)par * 8 + source) * 192 + pos; }
__device__ __forceinline__ unsigned img_off(unsigned par, unsigned uid) { return CTRL_B + (par * NIN + uid) * IMG_B; }
template <class P> __device__ __forceinline__ __amdgpu_buffer_rsrc_t mkrsrc(P p, unsigned bytes) {
    return __builtin_amdgcn_make_buffer_rsrc((void*)p, 0, bytes, 0x00020000);
}
#define ST128(v, rs, vo, so, aux) __builtin_amdgcn_raw_buffer_store_b128(v, rs, (vo) + (so), 0, aux)
__device__ __forceinline__ unsigned cvt_pk_bf16(float lo, float hi) { unsigned r; asm("v_cvt_pk_bf16_f32 %0, %1, %2" : "=v"(r) : "v"(lo), "v"(hi)); return r; }
__device__ __forceinline__ float silu(float y) {
    const float c = __builtin_amdgcn_fmed3f(y, -60.0f, 60.0f);
    return y * __builtin_amdgcn_rcpf(1.0f + __builtin_amdgcn_exp2f(c * -1.44269504089f));
}
__device__ __forceinline__ int gblock(int rank, int jb) { return jb < DB ? rank * DB + jb : NR * DB + (jb - DB); }

template <bool SYS> __device__ __forceinline__ bool wave_wait_ge(gu32* word, unsigned want, gu32* tmo, unsigned call, unsigned& v) {
    const unsigned long long t0 = __builtin_amdgcn_s_memrealtime();
    for (;;) {
        v = SYS ? __hip_atomic_load(word, RLX_SYS) : __hip_atomic_load(word, RLX_AGENT);
        if (__all(v >= want)) return true;
        if ((unsigned)__builtin_amdgcn_readfirstlane(__hip_atomic_load(tmo, RLX_AGENT)) == call ||
            __builtin_amdgcn_s_memrealtime() - t0 > DEADLINE_TICKS) {
            __hip_atomic_store(tmo, call, RLX_AGENT);
            return false;
        }
        __builtin_amdgcn_s_sleep(4);
    }
}

template <bool SYS> __device__ __forceinline__ bool wave_wait_ge(gu32* word, unsigned want, gu32* tmo, unsigned call) {
    unsigned v;
    return wave_wait_ge<SYS>(word, want, tmo, call, v);
}

__device__ __forceinline__ int chunk_pos(int c) { return c < NCH_IN ? c / UCH : NIN + (c - NCH_IN) / YCH; }
struct Chunk { unsigned sso, dso, yo, cnt; bool isy; };
__device__ __forceinline__ Chunk chunk_decode(KP p, int c, unsigned par, int rank, int src) {
    Chunk d;
    d.isy = c >= NCH_IN;
    const int pos = chunk_pos(c < NCH ? c : NCH - 1), uid = d.isy ? 0 : t_uorder(p, pos);
    const unsigned po = (unsigned)(c % UCH) * (CH * 1024u);
    d.yo = (unsigned)((c - NCH_IN) % YCH) * (CH * 1024u);
    d.dso = d.isy ? (unsigned)(pos - NIN) * (256u * NN * 2) + (unsigned)(src * DB) * 512u : img_off(par, uid) + src * CHUNK_B + po;
    d.sso = (unsigned)(uid - NPAN) < (unsigned)DB ? SCR_OFF + (unsigned)(rank * DB + uid - NPAN) * CHUNK_B + po : d.dso;
    d.cnt = d.isy ? CTRL_B : (unsigned)cnt_idx(par, uid, src) * 4u;
    return d;
}
__device__ __forceinline__ unsigned piece_voff(bool isy, unsigned yo, int k, int lane) {
    const unsigned l = (unsigned)k * 1024u + (unsigned)lane * 16u, o = yo + l;
    return isy ? (o / (DB * 512u)) * (NN * 2) + o % (DB * 512u) : l;
}
__device__ __forceinline__ void puller(KP p, const Self me) {
    const unsigned call = me.call, par = call & 1u, want = ((call + 1u) >> 1) * (4u * DB);
    const int tid = threadIdx.x, lane = tid & 63, wid = __builtin_amdgcn_readfirstlane(tid >> 6), rank = me.rank;
    const int gw = blockIdx.x * 8 + wid, j = gw % (NR - 1), src = (rank + 1 + j) & (NR - 1);
    gu32* ctl = (gu32*)me.ws;
    gu32* tmo = ctl + C_TMO;
    const unsigned long long s_ws = (unsigned long long)p->buf[src], s_out = (unsigned long long)p->buf[NR + src];
    const unsigned long long d_ws = (unsigned long long)me.ws, d_out = (unsigned long long)me.out;
    const __amdgpu_buffer_rsrc_t rc = mkrsrc(me.ws, CTRL_B);
    const int one = lane == 0;
    const unsigned head = lane == 0 ? (unsigned)(C_CLAIM + ((int)par * 8 + j) * LINE) * 4u : CTRL_B;
    constexpr int R = PD == 1 ? 2 : 4, C0 = NPRE * UCH;
    static_assert(CH % R == 0 && CH >= 2 * PD + 2, "ring");
    bool bad = false, primed = false;
    int sn = 0, ready = 0;
    unsigned prev = CTRL_B;
    int c = C0 + gw / (NR - 1);
    if (gw >= PWL * (NR - 1)) return;
    Chunk cur;
    u32x4 v[R];
#define PLOAD(d, k, on) __builtin_amdgcn_raw_buffer_load_b128(mkrsrc((d).isy ? s_out : s_ws, !(on) ? 0u : (d).isy ? OUT_B : (unsigned)WS_NEED), \
                                                              piece_voff((d).isy, (d).yo, k, lane), (d).sso, 17)
#pragma unroll 1
    while (c < NCH) {
        if (!primed) {
            const int pos = chunk_pos(c);
            WAIT_VC(0);
            __builtin_amdgcn_raw_ptr_buffer_atomic_add_i32(one * CH, rc, lane == 0 ? prev : CTRL_B, 0, 0);
            prev = CTRL_B;
            const unsigned long long t0 = __builtin_amdgcn_s_memrealtime();
            for (;;) {
                const unsigned long long m = ~__ballot(__hip_atomic_load(ctl + av_idx(par, src, pos + lane), RLX_SYS) >= want);
                const int n = m ? __builtin_ctzll(m) : 64;
                if (n > 0) { ready = pos + n; break; }
                if ((unsigned)__builtin_amdgcn_readfirstlane(__hip_atomic_load(tmo, RLX_AGENT)) == call ||
                    __builtin_amdgcn_s_memrealtime() - t0 > DEADLINE_TICKS) {
                    __hip_atomic_store(tmo, call, RLX_AGENT); bad = true; ready = 1 << 20; break;
                }
                __builtin_amdgcn_s_sleep(4);
            }
            asm volatile("" ::: "memory");
            cur = chunk_decode(p, c, par, rank, src);
#pragma unroll
            for (int k = 0; k < PD; ++k) {
                v[k] = PLOAD(cur, k, !bad);
                asm volatile("" ::: "memory");
                __builtin_amdgcn_raw_ptr_buffer_atomic_add_i32(0, rc, CTRL_B + lane * 4u, 0, 0);
                asm volatile("" ::: "memory");
            }
        }
        const int cv = __builtin_amdgcn_raw_ptr_buffer_atomic_add_i32(one, rc, head, 0, 0);
        const __amdgpu_buffer_rsrc_t rd = mkrsrc(cur.isy ? d_out : d_ws, cur.isy ? OUT_B : (unsigned)WS_NEED);
        Chunk nx = cur;
        int cn = 0;
#pragma unroll
        for (int k = 0; k < CH; ++k) {
            if (k == CH - PD - 1) {
                cn = C0 + PWL + __builtin_amdgcn_readfirstlane(cv);
                nx = chunk_decode(p, cn, par, rank, src);
                primed = cn < NCH && chunk_pos(cn) < ready;
            }
            const unsigned vo = piece_voff(cur.isy, cur.yo, k, lane) + cur.dso;
            asm volatile("" : "+v"(v[k % R]) :: "memory");
            if (k + PD < CH) v[(k + PD) % R] = PLOAD(cur, k + PD, !bad);
            else v[(k + PD) % R] = PLOAD(nx, k + PD - CH, primed && !bad);
            asm volatile("" ::: "memory");
            if (bad) v[k % R] = (u32x4){0x7fc07fc0u, 0x7fc07fc0u, 0x7fc07fc0u, 0x7fc07fc0u};
            ST128(v[k % R], rd, vo, 0u, 16);
            if (k == PD) __builtin_amdgcn_raw_ptr_buffer_atomic_add_i32(one * CH, rc, lane == 0 ? prev : CTRL_B, 0, 0);
            asm volatile("" ::: "memory");
        }
        prev = cur.cnt; cur = nx; c = cn;
        STAMP(2 + (sn < 238 ? sn : 238));
        ++sn;
    }
    WAIT_VC(0);
    __builtin_amdgcn_raw_ptr_buffer_atomic_add_i32(one * CH, rc, lane == 0 ? prev : CTRL_B, 0, 0);
    STAMP(250);
}

__device__ __forceinline__ unsigned bpos(unsigned fq, unsigned q, unsigned r) { return r * 256u + (((fq * 4u + q) ^ ((r & 1u) | ((r & 2u) << 2))) * 16u); }

__device__ __forceinline__ void stage(KP p, const Self me) {
    const unsigned call = me.call, par = call & 1u;
    const int tid = threadIdx.x, lane = tid & 63, wid = __builtin_amdgcn_readfirstlane(tid >> 6), rank = me.rank;
    const int fq = lane >> 4, c = lane & 15;
    const __amdgpu_buffer_rsrc_t rs = mkrsrc(me.ws, (unsigned)WS_NEED);
#pragma unroll 1
    for (int rnd = 0; rnd < 2; ++rnd) {
        const int code = t_stask(p, blockIdx.x * 2 + rnd);
        if (code == 0xffff) break;
        const int un = (code & 0x3fff) >> 2, task = (code & 3) * 8 + wid;
        int uid, dest = -1;
        u32x4 v[8];
        if (code & 0x4000) {
            const unsigned long long t0 = __builtin_amdgcn_s_memrealtime();
            while (__builtin_amdgcn_s_memrealtime() - t0 < STDELAY) __builtin_amdgcn_s_sleep(8);
        }
        if (code & 0x8000) {
            const int g = (code & 0x3fff) * 8 + wid, pc = g & 255;
            uid = g >> 8;
            v[0] = *(gvec*)((const gchar*)p->x + ((size_t)(uid * 256 + ((pc >> 1) & 15) * 16 + c) * KS + (pc >> 5) * 64 + (pc & 1) * 32 + fq * 8) * 2);
            const unsigned base = img_off(par, uid) + rank * CHUNK_B + (unsigned)pc * 1024u;
#pragma unroll
            for (int d = 0; d < NR; ++d) ST128(v[0], mkrsrc(p->buf[(rank + 1 + g + d) & 7], (unsigned)WS_NEED), (unsigned)lane * 16u, base, 17);
        } else if (un < NPAN) {
            uid = un;
            const gchar* src = (const gchar*)p->x + ((size_t)(un * 256 + (task & 3) * 64 + c) * KS + (task >> 2) * 64 + fq * 8) * 2;
#pragma unroll
            for (int j = 0; j < 8; ++j) v[j] = *(gvec*)(src + (j >> 1) * (16 * KS * 2) + (j & 1) * 64);
            const unsigned base = img_off(par, uid) + rank * CHUNK_B + (unsigned)task * 8192u;
#pragma unroll
            for (int j = 0; j < 8; ++j) ST128(v[j], rs, (unsigned)lane * 16u + j * 1024u, base, 17);
        } else {
            const int B = un - NPAN, ktl = task >> 2, ks = (task >> 1) & 1, g = task & 1;
            const gchar* src = (const gchar*)p->w + ((size_t)(ktl * 64 + ks * 32 + fq * 8) * NN + B * 256 + g * 128 + c * 8) * 2;
#pragma unroll
            for (int j = 0; j < 8; ++j) v[j] = *(gvec*)(src + (size_t)j * NN * 2);
            if (B < NR * DB) { dest = B / DB; uid = NPAN + B % DB; } else uid = NPAN + DB + (B - NR * DB);
            const unsigned base = (dest >= 0 && dest != rank ? SCR_OFF + (unsigned)B * CHUNK_B : img_off(par, uid) + rank * CHUNK_B)
                                  + (unsigned)ktl * 32768u + (unsigned)ks * 1024u;
            const unsigned lb = (unsigned)((c >> 2) & 1) * HTB + (unsigned)(2 * g + (c >> 3)) * 4096u;
#pragma unroll
            for (int e = 0; e < 8; ++e) {
                u32x4 o;
#pragma unroll
                for (int d = 0; d < 4; ++d)
                    o[d] = __builtin_amdgcn_perm(v[2 * d + 1][e >> 1], v[2 * d][e >> 1], (e & 1) ? 0x07060302u : 0x05040100u);
                ST128(o, rs, lb + (unsigned)(e >> 2) * 2048u + bpos(fq, c & 3, e & 3), base, 17);
            }
        }
        WAIT_V(0);
        __syncthreads();
        if (tid < NR && (dest < 0 || dest == tid)) {
            const bool landed = tid == rank || (code & 0x8000);
            __hip_atomic_fetch_add((gu32*)p->buf[tid] + (landed ? cnt_idx(par, uid, rank) : av_idx(par, rank, t_upos(p, uid))),
                                   (code & 0x8000) ? 8u : landed ? 64u : (unsigned)DB, RLX_SYS);
        }
    }
}

__device__ __forceinline__ void compute(KP p, const Self me, LAS unsigned char* lds) {
    const unsigned call = me.call;
    const int tid = threadIdx.x, wid = __builtin_amdgcn_readfirstlane(tid >> 6), lane = tid & 63, wr = wid >> 2, wc = wid & 3, fr = lane & 15, fq = lane >> 4;
    const int rank = me.rank, cwg = blockIdx.x - NPULL;
    const unsigned par = call & 1u, want = ((call + 1u) >> 1) * 256u;
    gu32* peer_av = (gu32*)p->buf[lane & 7] + av_idx(par, rank, NIN);
    gu32* ctl = (gu32*)me.ws;
    LAS unsigned* badf = (LAS unsigned*)(lds + STAGE_BYTES);
    const __amdgpu_buffer_rsrc_t rw = mkrsrc(me.ws, (unsigned)WS_NEED);
    const unsigned img = img_off(par, 0);
    const unsigned voff = (unsigned)tid * 16u, ldsw = (unsigned)wid * 1024u;
    const int aoff = wr * 8192 + lane * 16, boff = wc * 4096 + (int)bpos(fq, fr >> 2, fr & 3);
    constexpr unsigned kstep = 2 * HTB, hstep = HTB;
#define SA(b, h) (((b) * 2 + (h)) * HTB)
#define SB(b, h) ((4 + (b) * 2 + (h)) * HTB)
#define STAGE(bufoff, gbase) do { _Pragma("unroll") for (int _i = 0; _i < 2; ++_i) \
        __builtin_amdgcn_raw_ptr_buffer_load_lds(rw, (LAS void*)(lds + (bufoff) + ldsw + _i * 8192), 16, voff, (gbase) + _i * 8192, 0, 0); } while (0)
#define LDA(dst, b, h) do { _Pragma("unroll") for (int m = 0; m < 4; ++m) _Pragma("unroll") for (int k = 0; k < 2; ++k) dst[m][k] = *(const LAS bf16x8*)(lds + SA(b, h) + aoff + m * 2048 + k * 1024); } while (0)
#define LDB(dst, b, h) do { _Pragma("unroll") for (int n = 0; n < 2; ++n) _Pragma("unroll") for (int k = 0; k < 2; ++k) dst[n][k] = *(const LAS bf16x8*)(ldsB + (SB(b, h) - SB(0, 0)) + n * 2048 + k * 1024); } while (0)
#define MMA(ai, bj, At, Bt) do { __builtin_amdgcn_s_setprio(1); _Pragma("unroll") for (int m = 0; m < 4; ++m) _Pragma("unroll") for (int n = 0; n < 2; ++n) _Pragma("unroll") for (int k = 0; k < 2; ++k) \
        acc[ai][bj][m][n] = __builtin_amdgcn_mfma_f32_16x16x32_bf16(Bt[n][k], At[m][k], acc[ai][bj][m][n], 0, 0, 0); __builtin_amdgcn_s_setprio(0); } while (0)
    const int kclk_idx = C_KCLK + (int)(__builtin_amdgcn_s_getreg(20 | (3 << 11)) & 15u) * LINE;
#define POLL_WORD(tl) (ctl + (lane < 32 ? cnt_idx(par, (lane & 8) ? NPAN + ((tl) >> 4) : ((tl) & 15), lane & 7) : kclk_idx))
    LAS unsigned* gof = badf + 1;
    LAS unsigned char* ldsB = lds + SB(0, 0) + boff;
    asm volatile("" : "+v"(ldsB));
    constexpr int NIT = KK / 128;
    const unsigned ev8w0 = (1u << (NIT - 3)) | 1u | (KCLK ? 0x11111111u << ((cwg >> 3) & 3) : 0u);
    const unsigned ev8 = (1u << (NIT - 2)) | (wid == 0 ? ev8w0 : 0u), ev4 = wid == 0 ? 1u << (NIT - 2) : 0u;
    int q = cwg * MAXT, sn = 0;
    f32x4 acc[2][2][4][2];
    bf16x8 At[4][2], B0[2][2], B1[2][2];
#pragma unroll 1
    while (q < (cwg + 1) * MAXT && t_tile(p, q) != 0xffff) {
        int cur = t_tile(p, q), pend = -1;
        unsigned cA = img + (unsigned)(cur & 15) * IMG_B, cB = img + (unsigned)(NPAN + (cur >> 4)) * IMG_B;
        STAMP(2 + 4 * sn);
        if (wid == 0) {
            unsigned pv;
            if (!wave_wait_ge<true>(POLL_WORD(cur), lane < 32 ? want : 0u, ctl + C_TMO, call, pv)) { if (lane == 0) *badf = 1u; }
            if (lane == 32) gof[1] = pv + 2u;
            __builtin_amdgcn_fence(__ATOMIC_ACQUIRE, "");
            asm volatile("s_waitcnt vmcnt(0) lgkmcnt(0)" ::: "memory");
        }
        asm volatile("" ::: "memory"); BAR; asm volatile("" ::: "memory");
        unsigned vk = KCLK ? __builtin_amdgcn_readfirstlane(gof[1]) & ~1u : 0u;
        unsigned a2 = cA + (vk & 63u) * kstep, b2 = cB + (vk & 63u) * kstep;
        STAGE(SB(0, 0), b2); STAGE(SA(0, 0), a2); STAGE(SB(0, 1), b2 + hstep); STAGE(SA(0, 1), a2 + hstep);
        if (wr == 1) BAR;
        WAIT_V(4); BAR;
        STAGE(SB(1, 0), b2 + kstep); STAGE(SA(1, 0), a2 + kstep); STAGE(SB(1, 1), b2 + hstep + kstep);
        unsigned a1h = a2 + kstep + hstep;
        vk += 2u; a2 = cA + (vk & 63u) * kstep; b2 = cB + (vk & 63u) * kstep;
        WAIT_V(6); BAR;
        for (;;) {
            STAMP(3 + 4 * sn); STAMPC(100 + 2 * sn);
#pragma unroll
            for (int a = 0; a < 2; ++a)
#pragma unroll
                for (int b = 0; b < 2; ++b)
#pragma unroll
                    for (int m = 0; m < 4; ++m)
#pragma unroll
                        for (int n = 0; n < 2; ++n) acc[a][b][m][n] = (f32x4){0.f, 0.f, 0.f, 0.f};
            const bool has_next = q + 1 < (cwg + 1) * MAXT && t_tile(p, q + 1) != 0xffff;
            const int nxt = has_next ? t_tile(p, q + 1) : cur;
            bool go = false;
            unsigned pv = 0;
#pragma unroll 1
            for (int it = 0; it < NIT; ++it) {
                LDB(B0, 0, 0); SCHED; LDA(At, 0, 0); STAGE(SA(1, 1), a1h);
                WAIT_L(8); BAR; WAIT_L(0); MMA(0, 0, At, B0); BAR; SCHED;
                LDB(B1, 0, 1); STAGE(SB(0, 0), b2);
                BAR; WAIT_L(0); MMA(0, 1, At, B1); BAR;
                LDA(At, 0, 1); STAGE(SA(0, 0), a2);
                BAR; WAIT_L(0); MMA(1, 0, At, B0); BAR; SCHED;
                STAGE(SB(0, 1), b2 + hstep);
                WAIT_V(6);
                if (__builtin_expect((ev4 >> it) & 1u, 0)) {
                    asm volatile("buffer_inv sc0 sc1" : "+v"(pv) :: "memory");
                    const bool ok = __all(lane >= 32 || pv >= want);
                    if (lane == 0) gof[0] = ok;
                    if (lane == 32) gof[1] = pv + 4u;
                }
                BAR; MMA(1, 1, At, B1); BAR;
                LDB(B0, 1, 0); SCHED; LDA(At, 1, 0); STAGE(SA(0, 1), a2 + hstep);
                WAIT_L(8); BAR; WAIT_L(0); MMA(0, 0, At, B0); BAR; SCHED;
                LDB(B1, 1, 1); STAGE(SB(1, 0), b2 + kstep);
                BAR; WAIT_L(0); MMA(0, 1, At, B1); BAR;
                LDA(At, 1, 1); STAGE(SA(1, 0), a2 + kstep);
                BAR; WAIT_L(0); MMA(1, 0, At, B0); BAR; SCHED;
                STAGE(SB(1, 1), b2 + kstep + hstep);
                a1h = a2 + kstep + hstep;
                vk += 2u;
                WAIT_V(6);
                if (__builtin_expect((ev8 >> it) & 1u, 0)) {
                    if (wid == 0) {
                        if (KCLK && lane == 0) __hip_atomic_fetch_max(ctl + kclk_idx, vk - 2u, RLX_AGENT);
                        if (it == NIT - 3) asm volatile("global_load_dword %0, %1, off sc0 sc1" : "=v"(pv) : "v"(POLL_WORD(nxt)) : "memory");
                        if (it == 0 && pend >= 0 && lane < NR && lane != rank) __hip_atomic_fetch_add(peer_av + pend, 4u, RLX_SYS);
                    }
                    if (it == NIT - 2) {
                        go = has_next && __builtin_amdgcn_readfirstlane(gof[0]) != 0u;
                        if (go) {
                            cA = img + (unsigned)(nxt & 15) * IMG_B; cB = img + (unsigned)(NPAN + (nxt >> 4)) * IMG_B;
                            if (KCLK) vk = __builtin_amdgcn_readfirstlane(gof[1]) & ~1u;
                        }
                    }
                }
                a2 = cA + (vk & 63u) * kstep; b2 = cB + (vk & 63u) * kstep;
                BAR; MMA(1, 1, At, B1); BAR;
            }
            pend = -1;
            {
                STAMP(4 + 4 * sn); STAMPC(101 + 2 * sn);
                const bool bad = *badf != 0u;
                const __amdgpu_buffer_rsrc_t ro = mkrsrc(me.out, OUT_B);
                const unsigned so = (unsigned)((cur & 15) * 256 + wr * 64) * (NN * 2) + (unsigned)gblock(rank, cur >> 4) * 512u + (unsigned)wc * 128u;
                const unsigned vo = (unsigned)fr * (NN * 2) + (unsigned)fq * 16u;
#pragma unroll
                for (int ai = 0; ai < 2; ++ai)
#pragma unroll
                    for (int m = 0; m < 4; ++m)
#pragma unroll
                        for (int bj = 0; bj < 2; ++bj) {
                            const f32x4 v0 = acc[ai][bj][m][0], v1 = acc[ai][bj][m][1];
                            u32x4 w;
                            w.x = cvt_pk_bf16(silu(v0[0]), silu(v0[1])); w.y = cvt_pk_bf16(silu(v0[2]), silu(v0[3]));
                            w.z = cvt_pk_bf16(silu(v1[0]), silu(v1[1])); w.w = cvt_pk_bf16(silu(v1[2]), silu(v1[3]));
                            if (bad) w = (u32x4){0x7fc07fc0u, 0x7fc07fc0u, 0x7fc07fc0u, 0x7fc07fc0u};
                            ST128(w, ro, vo + bj * 64, so + (unsigned)(ai * 128 + m * 16) * (NN * 2), 17);
                        }
                if ((cur >> 4) < DB) pend = cur & 15;
                STAMP(5 + 4 * sn);
                ++sn;
            }
            ++q;
            if (!go) break;
            cur = nxt;
        }
        WAIT_V(0);
        if (wr == 0) BAR;
        BAR;
        if (pend >= 0 && tid < NR && tid != rank) __hip_atomic_fetch_add(peer_av + pend, 4u, RLX_SYS);
    }
    STAMP(250);
}

__global__ __launch_bounds__(512, 2) void fused_gemm_silu(const Params params) {
    KP p = (KP)__builtin_amdgcn_kernarg_segment_ptr();
    extern __shared__ __attribute__((aligned(16))) unsigned char shm[];
    LAS unsigned char* lds = (LAS unsigned char*)shm;
    LAS unsigned* ctlw = (LAS unsigned*)(lds + STAGE_BYTES);
    Self me;
    me.out = uni(p->out_self); me.ws = uni(p->ws_self); me.rank = __builtin_amdgcn_readfirstlane(p->rank);
    STAMP(0);
    gu32* cc = (gu32*)me.ws + C_CALL + blockIdx.x * LINE;
    if (threadIdx.x < 128) ctlw[threadIdx.x] = 0u;
    if (threadIdx.x == 128) { const unsigned c = __hip_atomic_load(cc, RLX_AGENT) + 1u; ctlw[128] = c; __hip_atomic_store(cc, c, RLX_AGENT); }
    __syncthreads();
    me.call = __builtin_amdgcn_readfirstlane(ctlw[128]);
    STAMP(1);
    if (blockIdx.x == 0 && threadIdx.x < 8)
        __hip_atomic_store((gu32*)me.ws + C_CLAIM + (((me.call & 1u) ^ 1u) * 8 + threadIdx.x) * LINE, 0u, RLX_AGENT);
    stage(p, me);
    if (blockIdx.x < NPULL) puller(p, me);
    else compute(p, me, lds);
}

void build_tables(Params& pp) {
    unsigned char *uorder = (unsigned char*)pp.uorder4, *upos = (unsigned char*)pp.upos4;
    unsigned short *tile = (unsigned short*)pp.tile2, *stask = (unsigned short*)pp.stask2;
    for (int i = 0; i < 80; ++i) uorder[i] = upos[i] = 0;
    int n = 0, nx = 0, nw = 0;
    while (n < NIN) {
        if (nx < NPAN && (nx < NPRE || nx <= nw || nw == NLB)) uorder[n++] = nx++;
        else uorder[n++] = NPAN + nw++;
    }
    unsigned short list[NSTASK];
    int ns = 0, nurg = 0;
    for (int i = 0; i < NIN; ++i) {
        const int u = uorder[i];
        upos[u] = i;
        if (i == NPRE + NURG) nurg = ns;
        for (int q = 0; q < 4 && u >= NPRE; ++q) {
            if (u < NPAN) list[ns++] = u * 4 + q;
            else if (u >= NPAN + DB) list[ns++] = (NPAN + NR * DB + (u - NPAN - DB)) * 4 + q;
            else for (int d = 0; d < NR; ++d) list[ns++] = (NPAN + d * DB + (u - NPAN)) * 4 + q;
        }
    }
    for (int i = 0; i < 2 * NWG; ++i) stask[i] = 0xffff;
    int li = 0, pi = 0;
    for (int b = 0; b < NWG; ++b) if (!(b & 4) && li < nurg) stask[2 * b] = list[li++];
    for (int b = NPULL; b < NWG; ++b) if ((b & 4) && pi < NPRE * 32) stask[2 * b] = 0x8000 | pi++;
    for (int b = NPULL; b < NWG; ++b) if (stask[2 * b] == 0xffff && li < ns) stask[2 * b] = 0x4000 | list[li++];
    for (int b = NWG - (ns - li); b < NWG; ++b) stask[2 * b + 1] = list[li++];
    unsigned short ql[8][NTILE];
    float qe[8][NTILE], freeat[NCOMP];
    int qn[8] = {0, 0, 0, 0, 0, 0, 0, 0}, qh[8] = {0, 0, 0, 0, 0, 0, 0, 0}, cnt[NCOMP], left = NTILE;
    for (int pos = 0; pos < NIN; ++pos)
        for (int jb = 0; jb < NLB; ++jb)
            for (int i = 0; i < NPAN; ++i) {
                const int key = upos[i] > upos[NPAN + jb] ? upos[i] : upos[NPAN + jb];
                if (key != pos) continue;
                const int x = ((i + (i >> 2)) & 3) + 4 * ((jb ^ (jb >> 1)) & 1);
                ql[x][qn[x]] = (unsigned short)(i | jb << 4); qe[x][qn[x]++] = 18.f + 4.35f * pos;
            }
    for (int w = 0; w < NCOMP; ++w) { freeat[w] = 0.f; cnt[w] = 0; }
    for (int i = 0; i < NCOMP * MAXT + 2; ++i) tile[i] = 0xffff;
    for (float t = 0.f; left > 0; t += 0.5f)
        for (int steal = 0; steal < 2; ++steal)
            for (int pass = 0; pass <= MAXT; ++pass)
                for (int w = 0; w < NCOMP; ++w) {
                    if (cnt[w] != pass || cnt[w] >= MAXT || freeat[w] > t) continue;
                    int x = (w + NPULL) % 8;
                    if (steal) {
                        if (qh[x] < qn[x] && qe[x][qh[x]] <= t + XSLACK) continue;
                        int bx = -1;
                        for (int y = 0; y < 8; ++y) {
                            if (qh[y] >= qn[y] || qe[y][qh[y]] > t) continue;
                            float nf = 1e9f;
                            for (int v = 0; v < NCOMP; ++v) if ((v + NPULL) % 8 == y && cnt[v] < MAXT && freeat[v] < nf) nf = freeat[v];
                            if (nf > t + XSLACK && (bx < 0 || qn[y] - qh[y] > qn[bx] - qh[bx])) bx = y;
                        }
                        if (bx < 0) continue;
                        x = bx;
                    } else if (qh[x] >= qn[x] || qe[x][qh[x]] > t) continue;
                    tile[w * MAXT + cnt[w]++] = ql[x][qh[x]++];
                    freeat[w] = t + (t < 50.f ? TM1 - 8.f : t < 90.f ? TM1 - 4.f : TM1);
                    --left;
                }
}
}

extern "C" void kernel_launch(int rank, int nranks, ncclComm_t comm, hipStream_t stream,
                              void* const* d_in, const int* in_sizes, int n_in,
                              void* d_out, int out_size, const int* in_dtypes, int out_dtype,
                              void* const* d_out_all, void* const* d_ws_all, size_t ws_bytes) {
    if (nranks != NR || (unsigned)rank >= (unsigned)NR || n_in < 2 || in_sizes[0] != MM * KS || in_sizes[1] != KS * NN ||
        out_size != MM * NN || ws_bytes < WS_NEED) {
        fprintf(stderr, "unexpected problem shape\n");
        return;
    }
    static bool attr_set = false;
    if (!attr_set) {
        HIP_CHECK(hipFuncSetAttribute((const void*)fused_gemm_silu, hipFuncAttributeMaxDynamicSharedMemorySize, LDS_BYTES));
        attr_set = true;
    }
    static Params tables;
    if (!tables.tile2[NCOMP * MAXT / 2]) build_tables(tables);
    Params p = tables;
    p.x = (const char*)d_in[0]; p.w = (const char*)d_in[1]; p.rank = rank;
    p.out_self = (char*)d_out_all[rank]; p.ws_self = (char*)d_ws_all[rank];
    for (int r = 0; r < NR; ++r) { p.buf[NR + r] = (char*)d_out_all[r]; p.buf[r] = (char*)d_ws_all[r]; }
    hipLaunchKernelGGL(fused_gemm_silu, dim3(NWG), dim3(512), LDS_BYTES, stream, p);
    HIP_CHECK(hipGetLastError());
}
```

```cpp
#include <hip/hip_runtime.h>
#include <rccl/rccl.h>
#include <cstdio>
#include <cstdint>

#define HIP_CHECK(expr) do { hipError_t _e = (expr); if (_e != hipSuccess) { \
    fprintf(stderr, "HIP error %s at %s:%d\n", hipGetErrorString(_e), __FILE__, __LINE__); \
    return; } } while (0)

#define NRB 40
#define PWL 29
#define NPULL ((7 * PWL + 7) / 8)
#define PD 2
#define CH 8
#define NPRE 2
#define NURG 12
#define STDELAY 300
#define LMAP 0
#define XSLACK 4.f
#define TM1 84.f

namespace {
#define GAS __attribute__((address_space(1)))
#define LAS __attribute__((address_space(3)))
typedef short bf16x8 __attribute__((ext_vector_type(8)));
typedef float f32x4 __attribute__((ext_vector_type(4)));
typedef unsigned u32x4 __attribute__((ext_vector_type(4)));
typedef GAS unsigned gu32;

constexpr int NR = 8, MM = 4096, KK = 4096, NN = 16384, KS = KK / NR;
constexpr int NPAN = MM / 256, NBLK = NN / 256;
constexpr int DB = (NBLK - NRB) / NR;
constexpr int NLB = DB + NRB;
constexpr int NIN = NPAN + NLB;
constexpr int NTILE = NPAN * NLB, ND = NPAN * DB;
constexpr int NWG = 256, NCOMP = NWG - NPULL;
constexpr int NSEG = NIN * 4, SCH = 64 / CH, NCH_IN = NSEG * SCH, YCH = 128 * DB / CH, NCH = NCH_IN + NPAN * YCH;
constexpr int NAV1 = 32;
constexpr int NSTASK = (NPAN + NBLK) * 4 - NPRE + NPRE * 8;
constexpr int MAXT = 4;
static_assert(NRB % 8 == 0 && DB >= 1 && NSTASK <= 2 * NCOMP && NTILE <= MAXT * NCOMP && NIN <= 64 && NPRE <= NPAN, "config");
constexpr unsigned PART_B = 64u << 10;
constexpr unsigned SEG_B = NR * PART_B, IMG_B = 4 * SEG_B;
constexpr unsigned CTRL_B = 1u << 20;
constexpr unsigned SCR_OFF = CTRL_B + 2u * NIN * IMG_B;
constexpr size_t WS_NEED = (size_t)SCR_OFF + (size_t)NR * DB * 4 * PART_B;
constexpr unsigned OUT_B = (unsigned)((size_t)MM * NN * 2);
constexpr int LINE = 32;
constexpr int C_CALL = 0;
constexpr int C_CNT = C_CALL + NWG * LINE;
constexpr int C_AV = C_CNT + 2 * 256 * LINE;
constexpr int C_TMO = C_AV + 2 * 8 * 64;
constexpr int C_CLAIM = C_TMO + LINE;
static_assert((C_CLAIM + 16 * LINE) * 4 <= (512 << 10) && NAV1 + 1 + NPAN <= 64, "ctrl");
constexpr unsigned long long DEADLINE_TICKS = 50000000ull;
constexpr int HTB = 16384, STAGE_BYTES = 8 * HTB, LDS_BYTES = STAGE_BYTES + 1024;

struct Params {
    const char* x; const char* w;
    char* out_self; char* ws_self;
    char* buf[2 * NR];
    int rank;
    unsigned seg4[256 / 4];
    unsigned segpos4[256 / 4];
    unsigned stask2[NWG];
    unsigned tile2[NCOMP * MAXT / 2 + 1];
};
typedef const __attribute__((address_space(4))) Params* KP;
__device__ __forceinline__ int t_seg(KP p, int i) { return (p->seg4[i >> 2] >> ((i & 3) * 8)) & 255; }
__device__ __forceinline__ int t_segpos(KP p, int i) { return (p->segpos4[i >> 2] >> ((i & 3) * 8)) & 255; }
__device__ __forceinline__ int t_stask(KP p, int i) { return (p->stask2[i >> 1] >> ((i & 1) * 16)) & 65535; }
__device__ __forceinline__ int t_tile(KP p, int i) { return (p->tile2[i >> 1] >> ((i & 1) * 16)) & 65535; }

#define RLX_AGENT __ATOMIC_RELAXED, __HIP_MEMORY_SCOPE_AGENT
#define RLX_SYS __ATOMIC_RELAXED, __HIP_MEMORY_SCOPE_SYSTEM
#define WAIT_V(n) asm volatile("s_waitcnt vmcnt(" #n ")" ::: "memory")
#define WAIT_L(n) asm volatile("s_waitcnt lgkmcnt(" #n ")" ::: "memory")
#define WAIT_VC(n) do { asm volatile("" ::: "memory"); __builtin_amdgcn_s_waitcnt(0x0F70 | ((n) & 15) | (((n) >> 4) << 14)); asm volatile("" ::: "memory"); } while (0)
#define BAR __builtin_amdgcn_s_barrier()
#define SCHED __builtin_amdgcn_sched_barrier(0)

#define STAMP(slot) do { } while (0)
#define STAMPC(slot) do { } while (0)
typedef GAS char gchar;
typedef const GAS u32x4 gvec;
__device__ __forceinline__ gchar* uni(const void* p) {
    const unsigned long long v = (unsigned long long)p;
    const unsigned lo = __builtin_amdgcn_readfirstlane((unsigned)v), hi = __builtin_amdgcn_readfirstlane((unsigned)(v >> 32));
    return (gchar*)(((unsigned long long)hi << 32) | lo);
}
struct Self { gchar* out; gchar* ws; int rank; unsigned call; };
__device__ __forceinline__ int cnt_idx(unsigned par, int sg) { return C_CNT + ((int)par * 256 + sg) * LINE; }
__device__ __forceinline__ int av_idx(unsigned par, int source, int slot) { return C_AV + ((int)par * 8 + source) * 64 + slot; }
__device__ __forceinline__ int av_slot(int pos) { return pos - NPRE < NAV1 ? pos - NPRE : NAV1; }
__device__ __forceinline__ unsigned seg_off(unsigned par, unsigned sg) { return CTRL_B + (par * NSEG + sg) * SEG_B; }
template <class P> __device__ __forceinline__ __amdgpu_buffer_rsrc_t mkrsrc(P p, unsigned bytes) {
    return __builtin_amdgcn_make_buffer_rsrc((void*)p, 0, bytes, 0x00020000);
}
#define ST128(v, rs, vo, so, aux) __builtin_amdgcn_raw_buffer_store_b128(v, rs, (vo) + (so), 0, aux)
__device__ __forceinline__ unsigned cvt_pk_bf16(float lo, float hi) { unsigned r; asm("v_cvt_pk_bf16_f32 %0, %1, %2" : "=v"(r) : "v"(lo), "v"(hi)); return r; }
__device__ __forceinline__ float silu(float y) {
    const float c = __builtin_amdgcn_fmed3f(y, -60.0f, 60.0f);
    return y * __builtin_amdgcn_rcpf(1.0f + __builtin_amdgcn_exp2f(c * -1.44269504089f));
}
__device__ __forceinline__ int gblock(int rank, int jb) { return jb < DB ? rank * DB + jb : NR * DB + (jb - DB); }

__device__ __forceinline__ bool timed_out(gu32* tmo, unsigned call, unsigned long long t0) {
    if ((unsigned)__builtin_amdgcn_readfirstlane(__hip_atomic_load(tmo, RLX_AGENT)) == call ||
        __builtin_amdgcn_s_memrealtime() - t0 > DEADLINE_TICKS) {
        __hip_atomic_store(tmo, call, RLX_AGENT);
        return true;
    }
    __builtin_amdgcn_s_sleep(4);
    return false;
}

__device__ __forceinline__ int chunk_slot(int c) { return c < NCH_IN ? av_slot(c / SCH) : NAV1 + 1 + (c - NCH_IN) / YCH; }
struct Chunk { unsigned sso, dso, yo, cnt; bool isy; };
__device__ __forceinline__ Chunk chunk_decode(KP p, int c, unsigned par, int rank, int src) {
    Chunk d;
    d.isy = c >= NCH_IN;
    const int sg = d.isy ? 0 : t_seg(p, c / SCH), cy = (c < NCH ? c : NCH - 1) - NCH_IN;
    const unsigned po = (unsigned)(c % SCH) * (CH * 1024u);
    d.yo = (unsigned)(cy % YCH) * (CH * 1024u);
    d.dso = d.isy ? (unsigned)(cy / YCH) * (256u * NN * 2) + (unsigned)(src * DB) * 512u : seg_off(par, sg) + src * PART_B + po;
    d.sso = (unsigned)(sg - NPAN * 4) < (unsigned)(DB * 4) ? SCR_OFF + (unsigned)(rank * DB * 4 + sg - NPAN * 4) * PART_B + po : d.dso;
    d.cnt = d.isy ? CTRL_B : (unsigned)cnt_idx(par, sg) * 4u;
    return d;
}
__device__ __forceinline__ unsigned piece_voff(bool isy, unsigned yo, int k, int lane) {
    const unsigned l = (unsigned)k * 1024u + (unsigned)lane * 16u, o = yo + l;
    return isy ? (o / (DB * 512u)) * (NN * 2) + o % (DB * 512u) : l;
}
__device__ __forceinline__ void puller(KP p, const Self me) {
    const int tid = threadIdx.x, lane = tid & 63, wid = __builtin_amdgcn_readfirstlane(tid >> 6), rank = me.rank;
    const unsigned call = me.call, par = call & 1u;
    const unsigned want = ((call + 1u) >> 1) * DB * (lane == NAV1 ? NSEG - NPRE - NAV1 : 1);
    const int gw = blockIdx.x * 8 + wid, j = LMAP ? gw / PWL : gw % (NR - 1), src = (rank + 1 + j) & (NR - 1);
    gu32* ctl = (gu32*)me.ws;
    gu32* tmo = ctl + C_TMO;
    const unsigned long long s_ws = (unsigned long long)p->buf[src], s_out = (unsigned long long)p->buf[NR + src];
    const unsigned long long d_ws = (unsigned long long)me.ws, d_out = (unsigned long long)me.out;
    const __amdgpu_buffer_rsrc_t rc = mkrsrc(me.ws, CTRL_B);
    const int one = lane == 0;
    const unsigned head = lane == 0 ? (unsigned)(C_CLAIM + ((int)par * 8 + j) * LINE) * 4u : CTRL_B;
    constexpr int R = PD == 1 ? 2 : 4, C0 = NPRE * SCH;
    static_assert(CH % R == 0 && CH >= 2 * PD + 2, "ring");
    bool bad = false, primed = false;
    int sn = 0;
    unsigned prev = CTRL_B;
    int c = C0 + (LMAP ? gw % PWL : gw / (NR - 1));
    if (gw >= PWL * (NR - 1)) return;
    Chunk cur;
    u32x4 v[R];
    unsigned vo[R];
#define PLOAD(d, o, on) __builtin_amdgcn_raw_buffer_load_b128(mkrsrc((d).isy ? s_out : s_ws, !(on) ? 0u : (d).isy ? OUT_B : (unsigned)WS_NEED), o, (d).sso, 17)
#pragma unroll 1
    while (c < NCH) {
        if (!primed) {
            const int slot = chunk_slot(c);
            WAIT_VC(0);
            __builtin_amdgcn_raw_ptr_buffer_atomic_add_i32(one * CH, rc, lane == 0 ? prev : CTRL_B, 0, 0);
            prev = CTRL_B;
            const unsigned long long t0 = __builtin_amdgcn_s_memrealtime();
            for (;;) {
                if ((__ballot(__hip_atomic_load(ctl + av_idx(par, src, lane), RLX_SYS) >= want) >> slot) & 1ull) break;
                if (timed_out(tmo, call, t0)) { bad = true; break; }
            }
            asm volatile("" ::: "memory");
            cur = chunk_decode(p, c, par, rank, src);
#pragma unroll
            for (int k = 0; k < PD; ++k) {
                vo[k] = piece_voff(cur.isy, cur.yo, k, lane);
                v[k] = PLOAD(cur, vo[k], !bad);
                asm volatile("" ::: "memory");
                __builtin_amdgcn_raw_ptr_buffer_atomic_add_i32(0, rc, CTRL_B + lane * 4u, 0, 0);
                asm volatile("" ::: "memory");
            }
        }
        const int cv = __builtin_amdgcn_raw_ptr_buffer_atomic_add_i32(one, rc, head, 0, 0);
        const unsigned av = __hip_atomic_load(ctl + av_idx(par, src, lane), RLX_SYS);
        const __amdgpu_buffer_rsrc_t rd = mkrsrc(cur.isy ? d_out : d_ws, cur.isy ? OUT_B : (unsigned)WS_NEED);
        Chunk nx = cur;
        int cn = 0;
#pragma unroll
        for (int k = 0; k < CH; ++k) {
            if (k == CH - PD - 1) {
                cn = C0 + PWL + __builtin_amdgcn_readfirstlane(cv);
                nx = chunk_decode(p, cn, par, rank, src);
                primed = cn < NCH && (bad || ((__ballot(av >= want) >> chunk_slot(cn)) & 1ull));
            }
            vo[(k + PD) % R] = k + PD < CH ? piece_voff(cur.isy, cur.yo, k + PD, lane) : piece_voff(nx.isy, nx.yo, k + PD - CH, lane);
            asm volatile("" : "+v"(vo[(k + PD) % R]));
            asm volatile("" : "+v"(v[k % R]) :: "memory");
            if (k + PD < CH) v[(k + PD) % R] = PLOAD(cur, vo[(k + PD) % R], !bad);
            else v[(k + PD) % R] = PLOAD(nx, vo[(k + PD) % R], primed && !bad);
            asm volatile("" ::: "memory");
            if (bad) v[k % R] = (u32x4){0x7fc07fc0u, 0x7fc07fc0u, 0x7fc07fc0u, 0x7fc07fc0u};
            ST128(v[k % R], rd, vo[k % R], cur.dso, 16);
            if (k == PD) __builtin_amdgcn_raw_ptr_buffer_atomic_add_i32(one * CH, rc, lane == 0 ? prev : CTRL_B, 0, 0);
            asm volatile("" ::: "memory");
        }
        prev = cur.cnt; cur = nx; c = cn;
        STAMP(2 + (sn < 238 ? sn : 238));
        ++sn;
    }
    WAIT_VC(0);
    __builtin_amdgcn_raw_ptr_buffer_atomic_add_i32(one * CH, rc, lane == 0 ? prev : CTRL_B, 0, 0);
    STAMP(250);
}

__device__ __forceinline__ unsigned bpos(unsigned fq, unsigned q, unsigned r) { return r * 256u + (((fq * 4u + q) ^ ((r & 1u) | ((r & 2u) << 2))) * 16u); }

__device__ __forceinline__ void stage(KP p, const Self me, unsigned& craw) {
    const int tid = threadIdx.x, lane = tid & 63, wid = __builtin_amdgcn_readfirstlane(tid >> 6), rank = me.rank;
    const int fq = lane >> 4, c = lane & 15;
    const __amdgpu_buffer_rsrc_t rs = mkrsrc(me.ws, (unsigned)WS_NEED);
#define GET_PAR() asm volatile("" : "+v"(craw) :: "memory"); const unsigned par = (__builtin_amdgcn_readfirstlane(craw) + 1u) & 1u
#pragma unroll 1
    for (int rnd = 0; rnd < 2; ++rnd) {
        const int code = t_stask(p, blockIdx.x * 2 + rnd);
        if (code == 0xffff) break;
        const int un = (code & 0x3fff) >> 2, qt = code & 3;
        int sg, dest = -1;
        u32x4 v[8];
        if (code & 0x4000) {
            const unsigned long long t0 = __builtin_amdgcn_s_memrealtime();
            while (__builtin_amdgcn_s_memrealtime() - t0 < STDELAY) __builtin_amdgcn_s_sleep(8);
        }
        if (code & 0x8000) {
            const int g = (code & 0x3fff) * 8 + wid, pc = g & 63;
            sg = t_seg(p, g >> 6);
            v[0] = *(gvec*)((const gchar*)p->x + ((size_t)((sg >> 2) * 256 + ((pc >> 1) & 15) * 16 + c) * KS + (sg & 3) * 128 + (pc >> 5) * 64 + (pc & 1) * 32 + fq * 8) * 2);
            GET_PAR();
            const unsigned base = seg_off(par, sg) + rank * PART_B + (unsigned)pc * 1024u;
#pragma unroll
            for (int d = 0; d < NR; ++d) ST128(v[0], mkrsrc(p->buf[(rank + 1 + g + d) & 7], (unsigned)WS_NEED), (unsigned)lane * 16u, base, 17);
        } else if (un < NPAN) {
            sg = un * 4 + qt;
            const gchar* src = (const gchar*)p->x + ((size_t)(un * 256 + (wid & 3) * 64 + c) * KS + qt * 128 + (wid >> 2) * 64 + fq * 8) * 2;
#pragma unroll
            for (int j = 0; j < 8; ++j) v[j] = *(gvec*)(src + (j >> 1) * (16 * KS * 2) + (j & 1) * 64);
            GET_PAR();
            const unsigned base = seg_off(par, sg) + rank * PART_B + (unsigned)wid * 8192u;
#pragma unroll
            for (int j = 0; j < 8; ++j) ST128(v[j], rs, (unsigned)lane * 16u + j * 1024u, base, 17);
        } else {
            const int B = un - NPAN, kt = wid >> 2, ks = (wid >> 1) & 1, g = wid & 1;
            const gchar* src = (const gchar*)p->w + ((size_t)(qt * 128 + kt * 64 + ks * 32 + fq * 8) * NN + B * 256 + g * 128 + c * 8) * 2;
#pragma unroll
            for (int j = 0; j < 8; ++j) v[j] = *(gvec*)(src + (size_t)j * NN * 2);
            GET_PAR();
            if (B < NR * DB) { dest = B / DB; sg = (NPAN + B % DB) * 4 + qt; } else sg = (NPAN + DB + (B - NR * DB)) * 4 + qt;
            const unsigned base = (dest >= 0 && dest != rank ? SCR_OFF + (unsigned)(B * 4 + qt) * PART_B : seg_off(par, sg) + rank * PART_B)
                                  + (unsigned)kt * 32768u + (unsigned)ks * 1024u;
            const unsigned lb = (unsigned)((c >> 2) & 1) * HTB + (unsigned)(2 * g + (c >> 3)) * 4096u;
#pragma unroll
            for (int e = 0; e < 8; ++e) {
                u32x4 o;
#pragma unroll
                for (int d = 0; d < 4; ++d)
                    o[d] = __builtin_amdgcn_perm(v[2 * d + 1][e >> 1], v[2 * d][e >> 1], (e & 1) ? 0x07060302u : 0x05040100u);
                ST128(o, rs, lb + (unsigned)(e >> 2) * 2048u + bpos(fq, c & 3, e & 3), base, 17);
            }
        }
        WAIT_V(0);
        __syncthreads();
        if (tid < NR && (dest < 0 || dest == tid)) {
            const unsigned par = (craw + 1u) & 1u;
            const bool landed = tid == rank || (code & 0x8000);
            __hip_atomic_fetch_add((gu32*)p->buf[tid] + (landed ? cnt_idx(par, sg) : av_idx(par, rank, av_slot(t_segpos(p, sg)))),
                                   (code & 0x8000) ? 8u : landed ? 64u : (unsigned)DB, RLX_SYS);
        }
    }
}

__device__ __forceinline__ void compute(KP p, const Self me, LAS unsigned char* lds) {
    const unsigned call = me.call;
    const int tid = threadIdx.x, wid = __builtin_amdgcn_readfirstlane(tid >> 6), lane = tid & 63, wr = wid >> 2, wc = wid & 3, fr = lane & 15, fq = lane >> 4;
    const int rank = me.rank, cwg = blockIdx.x - NPULL;
    const unsigned par = call & 1u, want = ((call + 1u) >> 1) * 512u;
    gu32* peer_av = (gu32*)p->buf[lane & 7] + av_idx(par, rank, NAV1 + 1);
    gu32* ctl = (gu32*)me.ws;
    LAS unsigned* badf = (LAS unsigned*)(lds + STAGE_BYTES);
    const __amdgpu_buffer_rsrc_t rw = mkrsrc(me.ws, (unsigned)WS_NEED);
    const unsigned img = seg_off(par, 0);
    const unsigned voff = (unsigned)tid * 16u, ldsw = (unsigned)wid * 1024u;
    const int aoff = wr * 8192 + lane * 16, boff = wc * 4096 + (int)bpos(fq, fr >> 2, fr & 3);
    constexpr unsigned kstep = 2 * HTB, hstep = HTB;
#define SA(b, h) (((b) * 2 + (h)) * HTB)
#define SB(b, h) ((4 + (b) * 2 + (h)) * HTB)
#define STAGE(bufoff, gbase) do { _Pragma("unroll") for (int _i = 0; _i < 2; ++_i) \
        __builtin_amdgcn_raw_ptr_buffer_load_lds(rw, (LAS void*)(lds + (bufoff) + ldsw + _i * 8192), 16, voff, (gbase) + _i * 8192, 0, 0); } while (0)
#define LDA(dst, b, h) do { _Pragma("unroll") for (int m = 0; m < 4; ++m) _Pragma("unroll") for (int k = 0; k < 2; ++k) dst[m][k] = *(const LAS bf16x8*)(lds + SA(b, h) + aoff + m * 2048 + k * 1024); } while (0)
#define LDB(dst, b, h) do { _Pragma("unroll") for (int n = 0; n < 2; ++n) _Pragma("unroll") for (int k = 0; k < 2; ++k) dst[n][k] = *(const LAS bf16x8*)(ldsB + (SB(b, h) - SB(0, 0)) + n * 2048 + k * 1024); } while (0)
#define MMA(ai, bj, At, Bt) do { __builtin_amdgcn_s_setprio(1); _Pragma("unroll") for (int m = 0; m < 4; ++m) _Pragma("unroll") for (int n = 0; n < 2; ++n) _Pragma("unroll") for (int k = 0; k < 2; ++k) \
        acc[ai][bj][m][n] = __builtin_amdgcn_mfma_f32_16x16x32_bf16(Bt[n][k], At[m][k], acc[ai][bj][m][n], 0, 0, 0); __builtin_amdgcn_s_setprio(0); } while (0)
#define POLL_IDX(tl) cnt_idx(par, ((lane & 4) ? NPAN + ((tl) >> 4) : ((tl) & 15)) * 4 + (lane & 3))
#define POLL_WORD(tl) (ctl + POLL_IDX(tl))
#define OKQ(pv) ({ const unsigned m_ = (unsigned)__ballot((pv) >= want); __builtin_ctz(~(m_ & (m_ >> 4) & 15u)); })
#define BLOCK_FOR(tl, need) do { const unsigned long long t0_ = __builtin_amdgcn_s_memrealtime(); \
        while ((okq = OKQ(__hip_atomic_load(POLL_WORD(tl), RLX_SYS))) <= (need)) \
            if (timed_out(ctl + C_TMO, call, t0_)) { if (lane == 0) *badf = 1u; okq = 4; break; } \
        __builtin_amdgcn_fence(__ATOMIC_ACQUIRE, ""); \
        asm volatile("s_waitcnt vmcnt(0) lgkmcnt(0)" ::: "memory"); } while (0)
    LAS unsigned* gof = badf + 1;
    LAS unsigned* prb = badf + 64;
    LAS unsigned char* ldsB = lds + SB(0, 0) + boff;
    asm volatile("" : "+v"(ldsB));
    constexpr int NIT = KK / 128;
    const unsigned ev8 = (1u << (NIT - 2)) | (wid == 0 ? 0x20202021u : 0u), ev4 = wid == 0 ? 0x40404040u : 0u;
    int q = cwg * MAXT, sn = 0, okq = 0;
    f32x4 acc[2][2][4][2];
    bf16x8 At[4][2], B0[2][2], B1[2][2];
#pragma unroll 1
    while (q < (cwg + 1) * MAXT && t_tile(p, q) != 0xffff) {
        int cur = t_tile(p, q), pend = -1;
        unsigned a2 = img + (unsigned)(cur & 15) * IMG_B, b2 = img + (unsigned)(NPAN + (cur >> 4)) * IMG_B;
        STAMP(2 + 4 * sn);
        if (wid == 0) BLOCK_FOR(cur, 0);
        asm volatile("" ::: "memory"); BAR; asm volatile("" ::: "memory");
        STAGE(SB(0, 0), b2); STAGE(SA(0, 0), a2); STAGE(SB(0, 1), b2 + hstep); STAGE(SA(0, 1), a2 + hstep);
        if (wr == 1) BAR;
        WAIT_V(4); BAR;
        STAGE(SB(1, 0), b2 + kstep); STAGE(SA(1, 0), a2 + kstep); STAGE(SB(1, 1), b2 + hstep + kstep);
        unsigned a1h = a2 + kstep + hstep;
        a2 += 2 * kstep; b2 += 2 * kstep;
        WAIT_V(6); BAR;
        for (;;) {
            STAMP(3 + 4 * sn); STAMPC(100 + 2 * sn);
#pragma unroll
            for (int a = 0; a < 2; ++a)
#pragma unroll
                for (int b = 0; b < 2; ++b)
#pragma unroll
                    for (int m = 0; m < 4; ++m)
#pragma unroll
                        for (int n = 0; n < 2; ++n) acc[a][b][m][n] = (f32x4){0.f, 0.f, 0.f, 0.f};
            const bool has_next = q + 1 < (cwg + 1) * MAXT && t_tile(p, q + 1) != 0xffff;
            const int nxt = has_next ? t_tile(p, q + 1) : cur;
            bool go = false;
#pragma unroll 1
            for (int it = 0; it < NIT; ++it) {
                LDB(B0, 0, 0); SCHED; LDA(At, 0, 0); STAGE(SA(1, 1), a1h);
                WAIT_L(8); BAR; WAIT_L(0); MMA(0, 0, At, B0); BAR; SCHED;
                LDB(B1, 0, 1); STAGE(SB(0, 0), b2);
                BAR; WAIT_L(0); MMA(0, 1, At, B1); BAR;
                LDA(At, 0, 1); STAGE(SA(0, 0), a2);
                BAR; WAIT_L(0); MMA(1, 0, At, B0); BAR; SCHED;
                STAGE(SB(0, 1), b2 + hstep);
                WAIT_V(6);
                if (__builtin_expect((ev4 >> it) & 1u, 0)) {
                    const int need = (it + 2) >> 3;
                    if (okq <= need) {
                        asm volatile("buffer_inv sc0 sc1" ::: "memory");
                        okq = OKQ(prb[lane]);
                        if (need == 4) { if (lane == 0) *gof = okq; }
                        else if (okq <= need) BLOCK_FOR(cur, need);
                    }
                }
                BAR; MMA(1, 1, At, B1); BAR;
                LDB(B0, 1, 0); SCHED; LDA(At, 1, 0); STAGE(SA(0, 1), a2 + hstep);
                WAIT_L(8); BAR; WAIT_L(0); MMA(0, 0, At, B0); BAR; SCHED;
                LDB(B1, 1, 1); STAGE(SB(1, 0), b2 + kstep);
                BAR; WAIT_L(0); MMA(0, 1, At, B1); BAR;
                LDA(At, 1, 1); STAGE(SA(1, 0), a2 + kstep);
                BAR; WAIT_L(0); MMA(1, 0, At, B0); BAR; SCHED;
                STAGE(SB(1, 1), b2 + kstep + hstep);
                a1h = a2 + kstep + hstep;
                a2 += 2 * kstep; b2 += 2 * kstep;
                WAIT_V(6);
                if (__builtin_expect((ev8 >> it) & 1u, 0)) {
                    if (it == NIT - 2) {
                        go = has_next && __builtin_amdgcn_readfirstlane(*gof) != 0u;
                        const int tl = go ? nxt : cur;
                        a2 = img + (unsigned)(tl & 15) * IMG_B; b2 = img + (unsigned)(NPAN + (tl >> 4)) * IMG_B;
                    } else if (it == 0) {
                        if (pend >= 0 && lane < NR && lane != rank) __hip_atomic_fetch_add(peer_av + pend, 1u, RLX_SYS);
                    } else if (okq <= (it + 3) >> 3) {
                        const int tl = it == NIT - 3 ? nxt : cur;
                        __builtin_amdgcn_raw_ptr_buffer_load_lds(rw, (LAS void*)prb, 4, (unsigned)POLL_IDX(tl) * 4u, 0, 0, 17);
                    }
                }
                BAR; MMA(1, 1, At, B1); BAR;
            }
            pend = -1;
            {
                STAMP(4 + 4 * sn); STAMPC(101 + 2 * sn);
                const bool bad = *badf != 0u;
                const __amdgpu_buffer_rsrc_t ro = mkrsrc(me.out, OUT_B);
                const unsigned so = (unsigned)((cur & 15) * 256 + wr * 64) * (NN * 2) + (unsigned)gblock(rank, cur >> 4) * 512u + (unsigned)wc * 128u;
                const unsigned vo = (unsigned)fr * (NN * 2) + (unsigned)fq * 16u;
#pragma unroll
                for (int ai = 0; ai < 2; ++ai)
#pragma unroll
                    for (int m = 0; m < 4; ++m)
#pragma unroll
                        for (int bj = 0; bj < 2; ++bj) {
                            const f32x4 v0 = acc[ai][bj][m][0], v1 = acc[ai][bj][m][1];
                            u32x4 w;
                            w.x = cvt_pk_bf16(silu(v0[0]), silu(v0[1])); w.y = cvt_pk_bf16(silu(v0[2]), silu(v0[3]));
                            w.z = cvt_pk_bf16(silu(v1[0]), silu(v1[1])); w.w = cvt_pk_bf16(silu(v1[2]), silu(v1[3]));
                            if (bad) w = (u32x4){0x7fc07fc0u, 0x7fc07fc0u, 0x7fc07fc0u, 0x7fc07fc0u};
                            ST128(w, ro, vo + bj * 64, so + (unsigned)(ai * 128 + m * 16) * (NN * 2), 17);
                        }
                if ((cur >> 4) < DB) pend = cur & 15;
                STAMP(5 + 4 * sn);
                ++sn;
            }
            ++q;
            if (!go) break;
            cur = nxt;
        }
        WAIT_V(0);
        if (wr == 0) BAR;
        BAR;
        if (pend >= 0 && tid < NR && tid != rank) __hip_atomic_fetch_add(peer_av + pend, 1u, RLX_SYS);
    }
    STAMP(250);
}

__global__ __launch_bounds__(512, 2) void fused_gemm_silu(const Params params) {
    KP p = (KP)__builtin_amdgcn_kernarg_segment_ptr();
    extern __shared__ __attribute__((aligned(16))) unsigned char shm[];
    LAS unsigned char* lds = (LAS unsigned char*)shm;
    LAS unsigned* ctlw = (LAS unsigned*)(lds + STAGE_BYTES);
    Self me;
    me.out = uni(p->out_self); me.ws = uni(p->ws_self); me.rank = __builtin_amdgcn_readfirstlane(p->rank);
    STAMP(0);
    gu32* cc = (gu32*)me.ws + C_CALL + blockIdx.x * LINE;
    unsigned craw = __hip_atomic_load(cc, RLX_AGENT);
    if (threadIdx.x < 16) ctlw[threadIdx.x] = 0u;
    stage(p, me, craw);
    me.call = __builtin_amdgcn_readfirstlane(craw) + 1u;
    __syncthreads();
    if (threadIdx.x == 0) __hip_atomic_store(cc, me.call, RLX_AGENT);
    STAMP(1);
    if (blockIdx.x == NWG - 1 && threadIdx.x < 8)
        __hip_atomic_store((gu32*)me.ws + C_CLAIM + (((me.call & 1u) ^ 1u) * 8 + threadIdx.x) * LINE, 0u, RLX_AGENT);
    if (blockIdx.x < NPULL) puller(p, me);
    else compute(p, me, lds);
}

void build_tables(Params& pp) {
    unsigned char *seg = (unsigned char*)pp.seg4, *segpos = (unsigned char*)pp.segpos4;
    unsigned short *tile = (unsigned short*)pp.tile2, *stask = (unsigned short*)pp.stask2;
    for (int i = 0; i < 256; ++i) seg[i] = segpos[i] = 0;
    int n = 0;
    for (int w0 = 0, r = 1; w0 < NLB; ++r) {
        int w1 = (r * NCOMP + NPAN - 1) / NPAN;
        if (w1 > NLB) w1 = NLB;
        for (int qt = 0; qt < 4; ++qt)
            for (int nx = w0 ? NPAN : 0, nw = w0; nx < NPAN || nw < w1;) {
                if (nx < NPAN && (nx < NPRE || nx <= nw || nw == w1)) seg[n++] = nx++ * 4 + qt;
                else seg[n++] = (NPAN + nw++) * 4 + qt;
            }
        w0 = w1;
    }
    unsigned short list[NSTASK];
    int ns = 0, nurg = 0;
    for (int i = 0; i < NSEG; ++i) {
        const int u = seg[i] >> 2, qt = seg[i] & 3;
        segpos[seg[i]] = i;
        if (i == NPRE + NURG) nurg = ns;
        if (i < NPRE) continue;
        if (u < NPAN) list[ns++] = u * 4 + qt;
        else if (u >= NPAN + DB) list[ns++] = (NPAN + NR * DB + (u - NPAN - DB)) * 4 + qt;
        else for (int d = 0; d < NR; ++d) list[ns++] = (NPAN + d * DB + (u - NPAN)) * 4 + qt;
    }
    for (int i = 0; i < 2 * NWG; ++i) stask[i] = 0xffff;
    int li = 0, pi = 0;
    for (int b = 0; b < NWG; ++b) if (!(b & 4) && li < nurg) stask[2 * b] = list[li++];
    for (int b = NPULL; b < NWG; ++b) if ((b & 4) && pi < NPRE * 8) stask[2 * b] = 0x8000 | pi++;
    for (int b = NPULL; b < NWG; ++b) if (stask[2 * b] == 0xffff && li < ns) stask[2 * b] = 0x4000 | list[li++];
    for (int b = NWG - (ns - li); b < NWG; ++b) stask[2 * b + 1] = list[li++];
    unsigned short ql[8][NTILE];
    float qe[8][NTILE], freeat[NCOMP];
    int qn[8] = {0, 0, 0, 0, 0, 0, 0, 0}, qh[8] = {0, 0, 0, 0, 0, 0, 0, 0}, cnt[NCOMP], left = NTILE;
    auto landed = [&](int tl, int qt) {
        const int pa = segpos[(tl & 15) * 4 + qt], pb = segpos[(NPAN + (tl >> 4)) * 4 + qt];
        return 12.f + 1.09f * (pa > pb ? pa : pb);
    };
    for (int pos = 0; pos < NSEG; ++pos)
        for (int jb = 0; jb < NLB; ++jb)
            for (int i = 0; i < NPAN; ++i) {
                const int key = segpos[i * 4] > segpos[(NPAN + jb) * 4] ? segpos[i * 4] : segpos[(NPAN + jb) * 4];
                if (key != pos) continue;
                const int x = (i & 3) + 4 * (jb & 1);
                ql[x][qn[x]] = (unsigned short)(i | jb << 4); qe[x][qn[x]++] = landed(i | jb << 4, 0);
            }
    for (int w = 0; w < NCOMP; ++w) { freeat[w] = 0.f; cnt[w] = 0; }
    for (int i = 0; i < NCOMP * MAXT + 2; ++i) tile[i] = 0xffff;
    for (float t = 0.f; left > 0; t += 0.5f)
        for (int steal = 0; steal < 2; ++steal)
            for (int pass = 0; pass <= MAXT; ++pass)
                for (int w = 0; w < NCOMP; ++w) {
                    if (cnt[w] != pass || cnt[w] >= MAXT || freeat[w] > t) continue;
                    int x = (w + NPULL) % 8;
                    if (steal) {
                        if (qh[x] < qn[x] && qe[x][qh[x]] <= t + XSLACK) continue;
                        int bx = -1;
                        for (int y = 0; y < 8; ++y) {
                            if (qh[y] >= qn[y] || qe[y][qh[y]] > t) continue;
                            float nf = 1e9f;
                            for (int v = 0; v < NCOMP; ++v) if ((v + NPULL) % 8 == y && cnt[v] < MAXT && freeat[v] < nf) nf = freeat[v];
                            if (nf > t + XSLACK && (bx < 0 || qn[y] - qh[y] > qn[bx] - qh[bx])) bx = y;
                        }
                        if (bx < 0) continue;
                        x = bx;
                    } else if (qh[x] >= qn[x] || qe[x][qh[x]] > t) continue;
                    const int tl = ql[x][qh[x]++];
                    tile[w * MAXT + cnt[w]++] = (unsigned short)tl;
                    float f = t;
                    for (int qt = 0; qt < 4; ++qt) { const float e = landed(tl, qt); f = (f > e ? f : e) + TM1 / 4; }
                    freeat[w] = f;
                    --left;
                }
}
}

extern "C" void kernel_launch(int rank, int nranks, ncclComm_t comm, hipStream_t stream,
                              void* const* d_in, const int* in_sizes, int n_in,
                              void* d_out, int out_size, const int* in_dtypes, int out_dtype,
                              void* const* d_out_all, void* const* d_ws_all, size_t ws_bytes) {
    if (nranks != NR || (unsigned)rank >= (unsigned)NR || n_in < 2 || in_sizes[0] != MM * KS || in_sizes[1] != KS * NN ||
        out_size != MM * NN || ws_bytes < WS_NEED) {
        fprintf(stderr, "unexpected problem shape\n");
        return;
    }
    static bool attr_set = false;
    if (!attr_set) {
        HIP_CHECK(hipFuncSetAttribute((const void*)fused_gemm_silu, hipFuncAttributeMaxDynamicSharedMemorySize, LDS_BYTES));
        attr_set = true;
    }
    static Params tables;
    if (!tables.tile2[NCOMP * MAXT / 2]) build_tables(tables);
    Params p = tables;
    p.x = (const char*)d_in[0]; p.w = (const char*)d_in[1]; p.rank = rank;
    p.out_self = (char*)d_out_all[rank]; p.ws_self = (char*)d_ws_all[rank];
    for (int r = 0; r < NR; ++r) { p.buf[NR + r] = (char*)d_out_all[r]; p.buf[r] = (char*)d_ws_all[r]; }
    hipLaunchKernelGGL(fused_gemm_silu, dim3(NWG), dim3(512), LDS_BYTES, stream, p);
    HIP_CHECK(hipGetLastError());
}
```

```cpp
#include <hip/hip_runtime.h>
#include <rccl/rccl.h>
#include <cstdio>
#include <cstdint>

#define HIP_CHECK(expr) do { hipError_t _e = (expr); if (_e != hipSuccess) { \
    fprintf(stderr, "HIP error %s at %s:%d\n", hipGetErrorString(_e), __FILE__, __LINE__); \
    return; } } while (0)

#define NRB 40
#define PWL 29
#define NPULL ((7 * PWL + 7) / 8)
#define PD 2
#define CH 8
#define MA 2
#define MB 4
#define NTAIL 96
#define NPRE 2
#define NURG 12
#define STDELAY 300
#define XSLACK 4.f
#define TM1 84.f

namespace {
#define GAS __attribute__((address_space(1)))
#define LAS __attribute__((address_space(3)))
typedef short bf16x8 __attribute__((ext_vector_type(8)));
typedef float f32x4 __attribute__((ext_vector_type(4)));
typedef unsigned u32x4 __attribute__((ext_vector_type(4)));
typedef GAS unsigned gu32;

constexpr int NR = 8, MM = 4096, KK = 4096, NN = 16384, KS = KK / NR;
constexpr int NPAN = MM / 256, NBLK = NN / 256;
constexpr int DB = (NBLK - NRB) / NR;
constexpr int NLB = DB + NRB;
constexpr int NIN = NPAN + NLB;
constexpr int NTILE = NPAN * NLB;
constexpr int NWG = 256, NCOMP = NWG - NPULL;
constexpr int NSEG = NIN * 4, SCH = 64 / CH, NCH_IN = NSEG * SCH, YCH = 128 * DB / CH, NCH = NCH_IN + NPAN * YCH;
constexpr int NAV1 = 32;
constexpr int NSTASK = (NPAN + NBLK) * 4 - NPRE + NPRE * 8;
constexpr int MAXT = 4;
static_assert(NRB % 8 == 0 && DB >= 1 && NSTASK <= 2 * NCOMP && NTILE <= MAXT * NCOMP && NIN <= 64 && NPRE <= NPAN, "config");
constexpr unsigned PART_B = 64u << 10;
constexpr unsigned SEG_B = NR * PART_B, IMG_B = 4 * SEG_B;
constexpr unsigned CTRL_B = 1u << 20;
constexpr unsigned SCR_OFF = CTRL_B + 2u * NIN * IMG_B;
constexpr size_t WS_NEED = (size_t)SCR_OFF + (size_t)NR * DB * 4 * PART_B;
constexpr unsigned OUT_B = (unsigned)((size_t)MM * NN * 2);
constexpr int LINE = 32;
constexpr int C_CALL = 0;
constexpr int C_CNT = C_CALL + NWG * LINE;
constexpr int C_AV = C_CNT + 2 * 256 * LINE;
constexpr int C_TMO = C_AV + 2 * 8 * 64;
constexpr int C_CLAIM = C_TMO + LINE;
static_assert((C_CLAIM + 16 * LINE) * 4 <= (512 << 10) && NAV1 + 1 + NPAN <= 64, "ctrl");
constexpr unsigned long long DEADLINE_TICKS = 50000000ull;
constexpr int HTB = 16384, STAGE_BYTES = 8 * HTB, LDS_BYTES = STAGE_BYTES + 1024;

struct Params {
    const char* x; const char* w;
    char* out_self; char* ws_self;
    char* buf[2 * NR];
    int rank;
    unsigned seg4[256 / 4];
    unsigned segpos4[256 / 4];
    unsigned stask2[NWG];
    unsigned tile2[NCOMP * MAXT / 2 + 1];
};
typedef const __attribute__((address_space(4))) Params* KP;
__device__ __forceinline__ int t_seg(KP p, int i) { return (p->seg4[i >> 2] >> ((i & 3) * 8)) & 255; }
__device__ __forceinline__ int t_segpos(KP p, int i) { return (p->segpos4[i >> 2] >> ((i & 3) * 8)) & 255; }
__device__ __forceinline__ int t_stask(KP p, int i) { return (p->stask2[i >> 1] >> ((i & 1) * 16)) & 65535; }
__device__ __forceinline__ int t_tile(KP p, int i) { return (p->tile2[i >> 1] >> ((i & 1) * 16)) & 65535; }

#define RLX_AGENT __ATOMIC_RELAXED, __HIP_MEMORY_SCOPE_AGENT
#define RLX_SYS __ATOMIC_RELAXED, __HIP_MEMORY_SCOPE_SYSTEM
#define WAIT_V(n) asm volatile("s_waitcnt vmcnt(" #n ")" ::: "memory")
#define WAIT_L(n) asm volatile("s_waitcnt lgkmcnt(" #n ")" ::: "memory")
#define WAIT_VC(n) do { asm volatile("" ::: "memory"); __builtin_amdgcn_s_waitcnt(0x0F70 | ((n) & 15) | (((n) >> 4) << 14)); asm volatile("" ::: "memory"); } while (0)
#define BAR __builtin_amdgcn_s_barrier()
#define SCHED __builtin_amdgcn_sched_barrier(0)

#define STAMP(slot) do { } while (0)
#define STAMPC(slot) do { } while (0)
typedef GAS char gchar;
typedef const GAS u32x4 gvec;
__device__ __forceinline__ gchar* uni(const void* p) {
    const unsigned long long v = (unsigned long long)p;
    const unsigned lo = __builtin_amdgcn_readfirstlane((unsigned)v), hi = __builtin_amdgcn_readfirstlane((unsigned)(v >> 32));
    return (gchar*)(((unsigned long long)hi << 32) | lo);
}
struct Self { gchar* out; gchar* ws; int rank; unsigned call; };
__device__ __forceinline__ int cnt_idx(unsigned par, int sg) { return C_CNT + ((int)par * 256 + sg) * LINE; }
__device__ __forceinline__ int av_idx(unsigned par, int source, int slot) { return C_AV + ((int)par * 8 + source) * 64 + slot; }
__device__ __forceinline__ int av_slot(int pos) { return pos - NPRE < NAV1 ? pos - NPRE : NAV1; }
__device__ __forceinline__ unsigned seg_off(unsigned par, unsigned sg) { return CTRL_B + (par * NSEG + sg) * SEG_B; }
template <class P> __device__ __forceinline__ __amdgpu_buffer_rsrc_t mkrsrc(P p, unsigned bytes) {
    return __builtin_amdgcn_make_buffer_rsrc((void*)p, 0, bytes, 0x00020000);
}
#define ST128(v, rs, vo, so, aux) __builtin_amdgcn_raw_buffer_store_b128(v, rs, (vo) + (so), 0, aux)
__device__ __forceinline__ unsigned cvt_pk_bf16(float lo, float hi) { unsigned r; asm("v_cvt_pk_bf16_f32 %0, %1, %2" : "=v"(r) : "v"(lo), "v"(hi)); return r; }
typedef float f32x2 __attribute__((ext_vector_type(2)));
__device__ __forceinline__ unsigned silu2(float y0, float y1, float one) {
    const f32x2 c = (f32x2){__builtin_amdgcn_fmed3f(y0, -60.0f, 60.0f), __builtin_amdgcn_fmed3f(y1, -60.0f, 60.0f)} * -1.44269504089f;
    const f32x2 d = (f32x2){__builtin_amdgcn_exp2f(c.x), __builtin_amdgcn_exp2f(c.y)} + one;
    const f32x2 o = (f32x2){y0, y1} * (f32x2){__builtin_amdgcn_rcpf(d.x), __builtin_amdgcn_rcpf(d.y)};
    return cvt_pk_bf16(o.x, o.y);
}
__device__ __forceinline__ int gblock(int rank, int jb) { return jb < DB ? rank * DB + jb : NR * DB + (jb - DB); }

__device__ __forceinline__ bool timed_out(gu32* tmo, unsigned call, unsigned long long t0) {
    if ((unsigned)__builtin_amdgcn_readfirstlane(__hip_atomic_load(tmo, RLX_AGENT)) == call ||
        __builtin_amdgcn_s_memrealtime() - t0 > DEADLINE_TICKS) {
        __hip_atomic_store(tmo, call, RLX_AGENT);
        return true;
    }
    __builtin_amdgcn_s_sleep(4);
    return false;
}

__device__ __forceinline__ int chunk_slot(int c) { return c < NCH_IN ? av_slot(c / SCH) : NAV1 + 1 + (c - NCH_IN) / YCH; }
struct Chunk { unsigned sso, dso, yo, cnt; bool isy; };
__device__ __forceinline__ Chunk chunk_decode(KP p, int c, unsigned par, int rank, int src) {
    Chunk d;
    d.isy = c >= NCH_IN;
    const int sg = d.isy ? 0 : t_seg(p, c / SCH), cy = (c < NCH ? c : NCH - 1) - NCH_IN;
    const unsigned po = (unsigned)(c % SCH) * (CH * 1024u);
    d.yo = (unsigned)(cy % YCH) * (CH * 1024u);
    d.dso = d.isy ? (unsigned)(cy / YCH) * (256u * NN * 2) + (unsigned)(src * DB) * 512u : seg_off(par, sg) + src * PART_B + po;
    d.sso = (unsigned)(sg - NPAN * 4) < (unsigned)(DB * 4) ? SCR_OFF + (unsigned)(rank * DB * 4 + sg - NPAN * 4) * PART_B + po : d.dso;
    d.cnt = d.isy ? CTRL_B : (unsigned)cnt_idx(par, sg) * 4u;
    return d;
}
__device__ __forceinline__ int ticket_chunk(unsigned t, int& m) {
    constexpr unsigned TA = (NCH_IN - NPRE * SCH) / MA, TB = TA + (NCH - NCH_IN - NTAIL) / MB;
    static_assert(SCH % MA == 0 && YCH % MB == 0 && NTAIL % MB == 0, "tickets");
    m = t < TA ? MA : t < TB ? MB : 1;
    return t < TA ? NPRE * SCH + (int)t * MA : t < TB ? NCH_IN + (int)(t - TA) * MB : t - TB < (unsigned)NTAIL ? NCH - NTAIL + (int)(t - TB) : NCH;
}
__device__ __forceinline__ unsigned piece_voff(bool isy, unsigned yo, int k, int lane) {
    const unsigned l = (unsigned)k * 1024u + (unsigned)lane * 16u, o = yo + l;
    return isy ? (o / (DB * 512u)) * (NN * 2) + o % (DB * 512u) : l;
}
__device__ __forceinline__ void puller(KP p, const Self me) {
    const int tid = threadIdx.x, lane = tid & 63, wid = __builtin_amdgcn_readfirstlane(tid >> 6), rank = me.rank;
    const unsigned call = me.call, par = call & 1u;
    const unsigned want = ((call + 1u) >> 1) * DB * (lane == NAV1 ? NSEG - NPRE - NAV1 : 1);
    const int gw = blockIdx.x * 8 + wid, j = gw % (NR - 1), src = (rank + 1 + j) & (NR - 1);
    gu32* ctl = (gu32*)me.ws;
    gu32* tmo = ctl + C_TMO;
    const unsigned long long s_ws = (unsigned long long)p->buf[src], s_out = (unsigned long long)p->buf[NR + src];
    const unsigned long long d_ws = (unsigned long long)me.ws, d_out = (unsigned long long)me.out;
    const __amdgpu_buffer_rsrc_t rc = mkrsrc(me.ws, CTRL_B);
    const int one = lane == 0;
    const unsigned head = lane == 0 ? (unsigned)(C_CLAIM + ((int)par * 8 + j) * LINE) * 4u : CTRL_B;
#define COUNT_IN() __builtin_amdgcn_raw_ptr_buffer_atomic_add_i32(prev < CTRL_B ? one * (CH * MA) : 0, rc, lane == 0 ? prev : CTRL_B, 0, 16)
    constexpr int R = PD == 1 ? 2 : 4;
    static_assert(CH % R == 0 && CH >= 2 * PD + 2, "ring");
    bool bad = false, primed = false;
    int sn = 0;
    unsigned prev = CTRL_B;
    int rem, c = ticket_chunk(gw / (NR - 1), rem);
    if (gw >= PWL * (NR - 1)) return;
    Chunk cur;
    u32x4 v[R];
    unsigned vo[R];
#define PLOAD(d, o, on) __builtin_amdgcn_raw_buffer_load_b128(mkrsrc((d).isy ? s_out : s_ws, !(on) ? 0u : (d).isy ? OUT_B : (unsigned)WS_NEED), o, (d).sso, 17)
#pragma unroll 1
    while (c < NCH) {
        if (!primed) {
            const int slot = chunk_slot(c);
            WAIT_VC(0);
            COUNT_IN();
            prev = CTRL_B;
            const unsigned long long t0 = __builtin_amdgcn_s_memrealtime();
            for (;;) {
                if ((__ballot(__hip_atomic_load(ctl + av_idx(par, src, lane), RLX_SYS) >= want) >> slot) & 1ull) break;
                if (timed_out(tmo, call, t0)) { bad = true; break; }
            }
            asm volatile("" ::: "memory");
            cur = chunk_decode(p, c, par, rank, src);
#pragma unroll
            for (int k = 0; k < PD; ++k) {
                vo[k] = piece_voff(cur.isy, cur.yo, k, lane);
                v[k] = PLOAD(cur, vo[k], !bad);
                asm volatile("" ::: "memory");
                __builtin_amdgcn_raw_ptr_buffer_atomic_add_i32(0, rc, CTRL_B + lane * 4u, 0, 0);
                asm volatile("" ::: "memory");
            }
        }
        const int cv = __builtin_amdgcn_raw_ptr_buffer_atomic_add_i32(rem == 1 ? one : 0, rc, rem == 1 ? head : CTRL_B, 0, 0);
        const unsigned av = __builtin_amdgcn_raw_buffer_load_b32(rc, rem == 1 ? (unsigned)av_idx(par, src, lane) * 4u : CTRL_B, 0, 17);
        const __amdgpu_buffer_rsrc_t rd = mkrsrc(cur.isy ? d_out : d_ws, cur.isy ? OUT_B : (unsigned)WS_NEED);
        Chunk nx = cur;
        int cn = 0, remn = 0;
#pragma unroll
        for (int k = 0; k < CH; ++k) {
            if (k == CH - PD - 1) {
                const unsigned tk = (unsigned)__builtin_amdgcn_readfirstlane(cv);
                cn = ticket_chunk(tk < (unsigned)NCH ? PWL + tk : (unsigned)NCH, remn);
                if (rem > 1) { cn = c + 1; remn = rem - 1; }
                nx = chunk_decode(p, cn, par, rank, src);
                primed = cn < NCH && (bad || rem > 1 || ((__ballot(av >= want) >> chunk_slot(cn)) & 1ull));
            }
            vo[(k + PD) % R] = k + PD < CH ? piece_voff(cur.isy, cur.yo, k + PD, lane) : piece_voff(nx.isy, nx.yo, k + PD - CH, lane);
            asm volatile("" : "+v"(vo[(k + PD) % R]));
            asm volatile("" : "+v"(v[k % R]) :: "memory");
            if (k + PD < CH) v[(k + PD) % R] = PLOAD(cur, vo[(k + PD) % R], !bad);
            else v[(k + PD) % R] = PLOAD(nx, vo[(k + PD) % R], primed && !bad);
            asm volatile("" ::: "memory");
            if (bad) v[k % R] = (u32x4){0x7fc07fc0u, 0x7fc07fc0u, 0x7fc07fc0u, 0x7fc07fc0u};
            ST128(v[k % R], rd, vo[k % R], cur.dso, 16);
            if (k == PD) COUNT_IN();
            asm volatile("" ::: "memory");
        }
        prev = rem == 1 ? cur.cnt : CTRL_B; cur = nx; c = cn; rem = remn;
        STAMP(2 + (sn < 238 ? sn : 238));
        ++sn;
    }
    WAIT_VC(0);
    COUNT_IN();
    STAMP(250);
}

__device__ __forceinline__ unsigned bpos(unsigned fq, unsigned q, unsigned r) { return r * 256u + (((fq * 4u + q) ^ ((r & 1u) | ((r & 2u) << 2))) * 16u); }

__device__ __forceinline__ void stage(KP p, const Self me, unsigned& craw) {
    const int tid = threadIdx.x, lane = tid & 63, wid = __builtin_amdgcn_readfirstlane(tid >> 6), rank = me.rank;
    const int fq = lane >> 4, c = lane & 15;
    const __amdgpu_buffer_rsrc_t rs = mkrsrc(me.ws, (unsigned)WS_NEED);
#define GET_PAR() asm volatile("" : "+v"(craw) :: "memory"); const unsigned par = (__builtin_amdgcn_readfirstlane(craw) + 1u) & 1u
#pragma unroll 1
    for (int rnd = 0; rnd < 2; ++rnd) {
        const int code = t_stask(p, blockIdx.x * 2 + rnd);
        if (code == 0xffff) break;
        const int un = (code & 0x3fff) >> 2, qt = code & 3;
        int sg, dest = -1;
        u32x4 v[8];
        if (code & 0x4000) {
            const unsigned long long t0 = __builtin_amdgcn_s_memrealtime();
            while (__builtin_amdgcn_s_memrealtime() - t0 < STDELAY) __builtin_amdgcn_s_sleep(8);
        }
        if (code & 0x8000) {
            const int g = (code & 0x3fff) * 8 + wid, pc = g & 63;
            sg = t_seg(p, g >> 6);
            v[0] = *(gvec*)((const gchar*)p->x + ((size_t)((sg >> 2) * 256 + ((pc >> 1) & 15) * 16 + c) * KS + (sg & 3) * 128 + (pc >> 5) * 64 + (pc & 1) * 32 + fq * 8) * 2);
            GET_PAR();
            const unsigned base = seg_off(par, sg) + rank * PART_B + (unsigned)pc * 1024u;
#pragma unroll
            for (int d = 0; d < NR; ++d) ST128(v[0], mkrsrc(p->buf[(rank + 1 + g + d) & 7], (unsigned)WS_NEED), (unsigned)lane * 16u, base, 17);
        } else if (un < NPAN) {
            sg = un * 4 + qt;
            const gchar* src = (const gchar*)p->x + ((size_t)(un * 256 + (wid & 3) * 64 + c) * KS + qt * 128 + (wid >> 2) * 64 + fq * 8) * 2;
#pragma unroll
            for (int j = 0; j < 8; ++j) v[j] = *(gvec*)(src + (j >> 1) * (16 * KS * 2) + (j & 1) * 64);
            GET_PAR();
            const unsigned base = seg_off(par, sg) + rank * PART_B + (unsigned)wid * 8192u;
#pragma unroll
            for (int j = 0; j < 8; ++j) ST128(v[j], rs, (unsigned)lane * 16u + j * 1024u, base, 17);
        } else {
            const int B = un - NPAN, kt = wid >> 2, ks = (wid >> 1) & 1, g = wid & 1;
            const gchar* src = (const gchar*)p->w + ((size_t)(qt * 128 + kt * 64 + ks * 32 + fq * 8) * NN + B * 256 + g * 128 + c * 8) * 2;
#pragma unroll
            for (int j = 0; j < 8; ++j) v[j] = *(gvec*)(src + (size_t)j * NN * 2);
            GET_PAR();
            if (B < NR * DB) { dest = B / DB; sg = (NPAN + B % DB) * 4 + qt; } else sg = (NPAN + DB + (B - NR * DB)) * 4 + qt;
            const unsigned base = (dest >= 0 && dest != rank ? SCR_OFF + (unsigned)(B * 4 + qt) * PART_B : seg_off(par, sg) + rank * PART_B)
                                  + (unsigned)kt * 32768u + (unsigned)ks * 1024u;
            const unsigned lb = (unsigned)((c >> 2) & 1) * HTB + (unsigned)(2 * g + (c >> 3)) * 4096u;
#pragma unroll
            for (int e = 0; e < 8; ++e) {
                u32x4 o;
#pragma unroll
                for (int d = 0; d < 4; ++d)
                    o[d] = __builtin_amdgcn_perm(v[2 * d + 1][e >> 1], v[2 * d][e >> 1], (e & 1) ? 0x07060302u : 0x05040100u);
                ST128(o, rs, lb + (unsigned)(e >> 2) * 2048u + bpos(fq, c & 3, e & 3), base, 17);
            }
        }
        WAIT_V(0);
        __syncthreads();
        if (tid < NR && (dest < 0 || dest == tid)) {
            const unsigned par = (craw + 1u) & 1u;
            const bool landed = tid == rank || (code & 0x8000);
            __hip_atomic_fetch_add((gu32*)p->buf[tid] + (landed ? cnt_idx(par, sg) : av_idx(par, rank, av_slot(t_segpos(p, sg)))),
                                   (code & 0x8000) ? 8u : landed ? 64u : (unsigned)DB, RLX_SYS);
        }
    }
}

__device__ __forceinline__ void compute(KP p, const Self me, LAS unsigned char* lds) {
    const unsigned call = me.call;
    const int tid = threadIdx.x, wid = __builtin_amdgcn_readfirstlane(tid >> 6), lane = tid & 63, wr = wid >> 2, wc = wid & 3, fr = lane & 15, fq = lane >> 4;
    const int rank = me.rank, cwg = blockIdx.x - NPULL;
    const unsigned par = call & 1u, want = ((call + 1u) >> 1) * 512u;
    gu32* peer_av = (gu32*)p->buf[lane & 7] + av_idx(par, rank, NAV1 + 1);
    gu32* ctl = (gu32*)me.ws;
    LAS unsigned* badf = (LAS unsigned*)(lds + STAGE_BYTES);
    const __amdgpu_buffer_rsrc_t rw = mkrsrc(me.ws, (unsigned)WS_NEED);
    const unsigned img = seg_off(par, 0);
    const unsigned voff = (unsigned)tid * 16u, ldsw = (unsigned)wid * 1024u;
    const int aoff = wr * 8192 + lane * 16, boff = wc * 4096 + (int)bpos(fq, fr >> 2, fr & 3);
    constexpr unsigned kstep = 2 * HTB, hstep = HTB;
#define SA(b, h) (((b) * 2 + (h)) * HTB)
#define SB(b, h) ((4 + (b) * 2 + (h)) * HTB)
#define STAGE(bufoff, gbase) do { _Pragma("unroll") for (int _i = 0; _i < 2; ++_i) \
        __builtin_amdgcn_raw_ptr_buffer_load_lds(rw, (LAS void*)(lds + (bufoff) + ldsw + _i * 8192), 16, voff, (gbase) + _i * 8192, 0, 0); } while (0)
#define LDA(dst, b, h) do { _Pragma("unroll") for (int m = 0; m < 4; ++m) _Pragma("unroll") for (int k = 0; k < 2; ++k) dst[m][k] = *(const LAS bf16x8*)(lds + SA(b, h) + aoff + m * 2048 + k * 1024); } while (0)
#define LDB(dst, b, h) do { _Pragma("unroll") for (int n = 0; n < 2; ++n) _Pragma("unroll") for (int k = 0; k < 2; ++k) dst[n][k] = *(const LAS bf16x8*)(ldsB + (SB(b, h) - SB(0, 0)) + n * 2048 + k * 1024); } while (0)
#define MMA(ai, bj, At, Bt) do { __builtin_amdgcn_s_setprio(1); _Pragma("unroll") for (int m = 0; m < 4; ++m) _Pragma("unroll") for (int n = 0; n < 2; ++n) _Pragma("unroll") for (int k = 0; k < 2; ++k) \
        acc[ai][bj][m][n] = __builtin_amdgcn_mfma_f32_16x16x32_bf16(Bt[n][k], At[m][k], acc[ai][bj][m][n], 0, 0, 0); __builtin_amdgcn_s_setprio(0); } while (0)
#define POLL_IDX(tl) cnt_idx(par, ((lane & 4) ? NPAN + ((tl) >> 4) : ((tl) & 15)) * 4 + (lane & 3))
#define POLL_WORD(tl) (ctl + POLL_IDX(tl))
#define OKQ(pv) ({ const unsigned m_ = (unsigned)__ballot((pv) >= want); __builtin_ctz(~(m_ & (m_ >> 4) & 15u)); })
#define BLOCK_FOR(tl, need) do { const unsigned long long t0_ = __builtin_amdgcn_s_memrealtime(); \
        while ((okq = OKQ(__hip_atomic_load(POLL_WORD(tl), RLX_SYS))) <= (need)) \
            if (timed_out(ctl + C_TMO, call, t0_)) { if (lane == 0) *badf = 1u; okq = 4; break; } \
        __builtin_amdgcn_fence(__ATOMIC_ACQUIRE, ""); \
        asm volatile("s_waitcnt vmcnt(0) lgkmcnt(0)" ::: "memory"); } while (0)
    LAS unsigned* gof = badf + 1;
    LAS unsigned* prb = badf + 64;
    LAS unsigned char* ldsB = lds + SB(0, 0) + boff;
    asm volatile("" : "+v"(ldsB));
    constexpr int NIT = KK / 128;
    const unsigned ev8 = (1u << (NIT - 2)) | (wid == 0 ? 0x20202021u : 0u), ev4 = wid == 0 ? 0x40404040u : 0u;
    int q = cwg * MAXT, sn = 0, okq = 0;
    f32x4 acc[2][2][4][2];
    bf16x8 At[4][2], B0[2][2], B1[2][2];
#pragma unroll 1
    while (q < (cwg + 1) * MAXT && t_tile(p, q) != 0xffff) {
        int cur = t_tile(p, q), pend = -1;
        unsigned a2 = img + (unsigned)(cur & 15) * IMG_B, b2 = img + (unsigned)(NPAN + (cur >> 4)) * IMG_B;
        STAMP(2 + 4 * sn);
        if (wid == 0) BLOCK_FOR(cur, 0);
        asm volatile("" ::: "memory"); BAR; asm volatile("" ::: "memory");
        STAGE(SB(0, 0), b2); STAGE(SA(0, 0), a2); STAGE(SB(0, 1), b2 + hstep); STAGE(SA(0, 1), a2 + hstep);
        if (wr == 1) BAR;
        WAIT_V(4); BAR;
        STAGE(SB(1, 0), b2 + kstep); STAGE(SA(1, 0), a2 + kstep); STAGE(SB(1, 1), b2 + hstep + kstep);
        unsigned a1h = a2 + kstep + hstep;
        a2 += 2 * kstep; b2 += 2 * kstep;
        WAIT_V(6); BAR;
        for (;;) {
            STAMP(3 + 4 * sn); STAMPC(100 + 2 * sn);
#pragma unroll
            for (int a = 0; a < 2; ++a)
#pragma unroll
                for (int b = 0; b < 2; ++b)
#pragma unroll
                    for (int m = 0; m < 4; ++m)
#pragma unroll
                        for (int n = 0; n < 2; ++n) acc[a][b][m][n] = (f32x4){0.f, 0.f, 0.f, 0.f};
            const bool has_next = q + 1 < (cwg + 1) * MAXT && t_tile(p, q + 1) != 0xffff;
            const int nxt = has_next ? t_tile(p, q + 1) : cur;
            bool go = false;
#pragma unroll 1
            for (int it = 0; it < NIT; ++it) {
                LDB(B0, 0, 0); SCHED; LDA(At, 0, 0); STAGE(SA(1, 1), a1h);
                WAIT_L(8); BAR; WAIT_L(0); MMA(0, 0, At, B0); BAR; SCHED;
                LDB(B1, 0, 1); STAGE(SB(0, 0), b2);
                BAR; WAIT_L(0); MMA(0, 1, At, B1); BAR;
                LDA(At, 0, 1); STAGE(SA(0, 0), a2);
                BAR; WAIT_L(0); MMA(1, 0, At, B0); BAR; SCHED;
                STAGE(SB(0, 1), b2 + hstep);
                WAIT_V(6);
                if (__builtin_expect((ev4 >> it) & 1u, 0)) {
                    const int need = (it + 2) >> 3;
                    if (okq <= need) {
                        asm volatile("buffer_inv sc0 sc1" ::: "memory");
                        okq = OKQ(prb[lane]);
                        if (need == 4) { if (lane == 0) *gof = okq; }
                        else if (okq <= need) BLOCK_FOR(cur, need);
                    }
                }
                BAR; MMA(1, 1, At, B1); BAR;
                LDB(B0, 1, 0); SCHED; LDA(At, 1, 0); STAGE(SA(0, 1), a2 + hstep);
                WAIT_L(8); BAR; WAIT_L(0); MMA(0, 0, At, B0); BAR; SCHED;
                LDB(B1, 1, 1); STAGE(SB(1, 0), b2 + kstep);
                BAR; WAIT_L(0); MMA(0, 1, At, B1); BAR;
                LDA(At, 1, 1); STAGE(SA(1, 0), a2 + kstep);
                BAR; WAIT_L(0); MMA(1, 0, At, B0); BAR; SCHED;
                STAGE(SB(1, 1), b2 + kstep + hstep);
                a1h = a2 + kstep + hstep;
                a2 += 2 * kstep; b2 += 2 * kstep;
                WAIT_V(6);
                if (__builtin_expect((ev8 >> it) & 1u, 0)) {
                    if (it == NIT - 2) {
                        go = has_next && __builtin_amdgcn_readfirstlane(*gof) != 0u;
                        const int tl = go ? nxt : cur;
                        a2 = img + (unsigned)(tl & 15) * IMG_B; b2 = img + (unsigned)(NPAN + (tl >> 4)) * IMG_B;
                    } else if (it == 0) {
                        if (pend >= 0 && lane < NR && lane != rank) __hip_atomic_fetch_add(peer_av + pend, 1u, RLX_SYS);
                    } else if (okq <= (it + 3) >> 3) {
                        const int tl = it == NIT - 3 ? nxt : cur;
                        __builtin_amdgcn_raw_ptr_buffer_load_lds(rw, (LAS void*)prb, 4, (unsigned)POLL_IDX(tl) * 4u, 0, 0, 17);
                    }
                }
                BAR; MMA(1, 1, At, B1); BAR;
            }
            pend = -1;
            {
                STAMP(4 + 4 * sn); STAMPC(101 + 2 * sn);
                const float one = *badf != 0u ? __builtin_nanf("") : 1.0f;
                const __amdgpu_buffer_rsrc_t ro = mkrsrc(me.out, OUT_B);
                const unsigned so = (unsigned)((cur & 15) * 256 + wr * 64) * (NN * 2) + (unsigned)gblock(rank, cur >> 4) * 512u + (unsigned)wc * 128u;
                const unsigned vo = (unsigned)fr * (NN * 2) + (unsigned)fq * 16u;
#pragma unroll
                for (int ai = 0; ai < 2; ++ai)
#pragma unroll
                    for (int m = 0; m < 4; ++m)
#pragma unroll
                        for (int bj = 0; bj < 2; ++bj) {
                            const f32x4 v0 = acc[ai][bj][m][0], v1 = acc[ai][bj][m][1];
                            u32x4 w;
                            w.x = silu2(v0[0], v0[1], one); w.y = silu2(v0[2], v0[3], one);
                            w.z = silu2(v1[0], v1[1], one); w.w = silu2(v1[2], v1[3], one);
                            ST128(w, ro, vo + bj * 64, so + (unsigned)(ai * 128 + m * 16) * (NN * 2), 17);
                        }
                if ((cur >> 4) < DB) pend = cur & 15;
                STAMP(5 + 4 * sn);
                ++sn;
            }
            ++q;
            if (!go) break;
            cur = nxt;
        }
        WAIT_V(0);
        if (wr == 0) BAR;
        BAR;
        if (pend >= 0 && tid < NR && tid != rank) __hip_atomic_fetch_add(peer_av + pend, 1u, RLX_SYS);
    }
    STAMP(250);
}

__global__ __launch_bounds__(512, 2) void fused_gemm_silu(const Params params) {
    KP p = (KP)__builtin_amdgcn_kernarg_segment_ptr();
    extern __shared__ __attribute__((aligned(16))) unsigned char shm[];
    LAS unsigned char* lds = (LAS unsigned char*)shm;
    LAS unsigned* ctlw = (LAS unsigned*)(lds + STAGE_BYTES);
    Self me;
    me.out = uni(p->out_self); me.ws = uni(p->ws_self); me.rank = __builtin_amdgcn_readfirstlane(p->rank);
    STAMP(0);
    gu32* cc = (gu32*)me.ws + C_CALL + blockIdx.x * LINE;
    unsigned craw = __hip_atomic_load(cc, RLX_AGENT);
    if (threadIdx.x < 16) ctlw[threadIdx.x] = 0u;
    stage(p, me, craw);
    me.call = __builtin_amdgcn_readfirstlane(craw) + 1u;
    __syncthreads();
    if (threadIdx.x == 0) __hip_atomic_store(cc, me.call, RLX_AGENT);
    STAMP(1);
    if (blockIdx.x == NWG - 1 && threadIdx.x < 8)
        __hip_atomic_store((gu32*)me.ws + C_CLAIM + (((me.call & 1u) ^ 1u) * 8 + threadIdx.x) * LINE, 0u, RLX_AGENT);
    if (blockIdx.x < NPULL) puller(p, me);
    else compute(p, me, lds);
}

void build_tables(Params& pp) {
    unsigned char *seg = (unsigned char*)pp.seg4, *segpos = (unsigned char*)pp.segpos4;
    unsigned short *tile = (unsigned short*)pp.tile2, *stask = (unsigned short*)pp.stask2;
    for (int i = 0; i < 256; ++i) seg[i] = segpos[i] = 0;
    int n = 0;
    for (int w0 = 0, r = 1; w0 < NLB; ++r) {
        int w1 = (r * NCOMP + NPAN - 1) / NPAN;
        if (w1 > NLB) w1 = NLB;
        for (int qt = 0; qt < 4; ++qt)
            for (int nx = w0 ? NPAN : 0, nw = w0; nx < NPAN || nw < w1;) {
                if (nx < NPAN && (nx < NPRE || nx <= nw || nw == w1)) seg[n++] = nx++ * 4 + qt;
                else seg[n++] = (NPAN + nw++) * 4 + qt;
            }
        w0 = w1;
    }
    unsigned short list[NSTASK];
    int ns = 0, nurg = 0;
    for (int i = 0; i < NSEG; ++i) {
        const int u = seg[i] >> 2, qt = seg[i] & 3;
        segpos[seg[i]] = i;
        if (i == NPRE + NURG) nurg = ns;
        if (i < NPRE) continue;
        if (u < NPAN) list[ns++] = u * 4 + qt;
        else if (u >= NPAN + DB) list[ns++] = (NPAN + NR * DB + (u - NPAN - DB)) * 4 + qt;
        else for (int d = 0; d < NR; ++d) list[ns++] = (NPAN + d * DB + (u - NPAN)) * 4 + qt;
    }
    for (int i = 0; i < 2 * NWG; ++i) stask[i] = 0xffff;
    int li = 0, pi = 0;
    for (int b = 0; b < NWG; ++b) if (!(b & 4) && li < nurg) stask[2 * b] = list[li++];
    for (int b = NPULL; b < NWG; ++b) if ((b & 4) && pi < NPRE * 8) stask[2 * b] = 0x8000 | pi++;
    for (int b = NPULL; b < NWG; ++b) if (stask[2 * b] == 0xffff && li < ns) stask[2 * b] = 0x4000 | list[li++];
    for (int b = NWG - (ns - li); b < NWG; ++b) stask[2 * b + 1] = list[li++];
    unsigned short ql[8][NTILE];
    float qe[8][NTILE], freeat[NCOMP];
    int qn[8] = {0, 0, 0, 0, 0, 0, 0, 0}, qh[8] = {0, 0, 0, 0, 0, 0, 0, 0}, cnt[NCOMP], left = NTILE;
    auto landed = [&](int tl, int qt) {
        const int pa = segpos[(tl & 15) * 4 + qt], pb = segpos[(NPAN + (tl >> 4)) * 4 + qt];
        return 12.f + 1.09f * (pa > pb ? pa : pb);
    };
    for (int pos = 0; pos < NSEG; ++pos)
        for (int jb = 0; jb < NLB; ++jb)
            for (int i = 0; i < NPAN; ++i) {
                const int key = segpos[i * 4] > segpos[(NPAN + jb) * 4] ? segpos[i * 4] : segpos[(NPAN + jb) * 4];
                if (key != pos) continue;
                const int x = (i & 3) + 4 * (jb & 1);
                ql[x][qn[x]] = (unsigned short)(i | jb << 4); qe[x][qn[x]++] = landed(i | jb << 4, 0);
            }
    for (int w = 0; w < NCOMP; ++w) { freeat[w] = 0.f; cnt[w] = 0; }
    for (int i = 0; i < NCOMP * MAXT + 2; ++i) tile[i] = 0xffff;
    for (float t = 0.f; left > 0; t += 0.5f)
        for (int steal = 0; steal < 2; ++steal)
            for (int pass = 0; pass <= MAXT; ++pass)
                for (int w = 0; w < NCOMP; ++w) {
                    if (cnt[w] != pass || cnt[w] >= MAXT || freeat[w] > t) continue;
                    int x = (w + NPULL) % 8;
                    if (steal) {
                        if (qh[x] < qn[x] && qe[x][qh[x]] <= t + XSLACK) continue;
                        int bx = -1;
                        for (int y = 0; y < 8; ++y) {
                            if (qh[y] >= qn[y] || qe[y][qh[y]] > t) continue;
                            float nf = 1e9f;
                            for (int v = 0; v < NCOMP; ++v) if ((v + NPULL) % 8 == y && cnt[v] < MAXT && freeat[v] < nf) nf = freeat[v];
                            if (nf > t + XSLACK && (bx < 0 || qn[y] - qh[y] > qn[bx] - qh[bx])) bx = y;
                        }
                        if (bx < 0) continue;
                        x = bx;
                    } else if (qh[x] >= qn[x] || qe[x][qh[x]] > t) continue;
                    const int tl = ql[x][qh[x]++];
                    tile[w * MAXT + cnt[w]++] = (unsigned short)tl;
                    float f = t;
                    for (int qt = 0; qt < 4; ++qt) { const float e = landed(tl, qt); f = (f > e ? f : e) + TM1 / 4; }
                    freeat[w] = f;
                    --left;
                }
}
}

extern "C" void kernel_launch(int rank, int nranks, ncclComm_t comm, hipStream_t stream,
                              void* const* d_in, const int* in_sizes, int n_in,
                              void* d_out, int out_size, const int* in_dtypes, int out_dtype,
                              void* const* d_out_all, void* const* d_ws_all, size_t ws_bytes) {
    if (nranks != NR || (unsigned)rank >= (unsigned)NR || n_in < 2 || in_sizes[0] != MM * KS || in_sizes[1] != KS * NN ||
        out_size != MM * NN || ws_bytes < WS_NEED) {
        fprintf(stderr, "unexpected problem shape\n");
        return;
    }
    static bool attr_set = false;
    if (!attr_set) {
        HIP_CHECK(hipFuncSetAttribute((const void*)fused_gemm_silu, hipFuncAttributeMaxDynamicSharedMemorySize, LDS_BYTES));
        attr_set = true;
    }
    Params p;
    build_tables(p);
    p.x = (const char*)d_in[0]; p.w = (const char*)d_in[1]; p.rank = rank;
    p.out_self = (char*)d_out_all[rank]; p.ws_self = (char*)d_ws_all[rank];
    for (int r = 0; r < NR; ++r) { p.buf[NR + r] = (char*)d_out_all[r]; p.buf[r] = (char*)d_ws_all[r]; }
    hipLaunchKernelGGL(fused_gemm_silu, dim3(NWG), dim3(512), LDS_BYTES, stream, p);
    HIP_CHECK(hipGetLastError());
}
```

```cpp
#include <hip/hip_runtime.h>
#include <rccl/rccl.h>
#include <cstdio>
#include <cstdint>

#define HIP_CHECK(expr) do { hipError_t _e = (expr); if (_e != hipSuccess) { \
    fprintf(stderr, "HIP error %s at %s:%d\n", hipGetErrorString(_e), __FILE__, __LINE__); \
    return; } } while (0)

#define NRB 40
#define PWL 29
#define NPULL ((7 * PWL + 7) / 8)
#define PD 2
#define CH 8
#define MA 2
#define MB 4
#define NTAIL 96
#define NPRE 2
#define NURG 12
#define STDELAY 300
#define XSLACK 4.f
#define TM1 84.f

namespace {
#define GAS __attribute__((address_space(1)))
#define LAS __attribute__((address_space(3)))
typedef short bf16x8 __attribute__((ext_vector_type(8)));
typedef float f32x4 __attribute__((ext_vector_type(4)));
typedef unsigned u32x4 __attribute__((ext_vector_type(4)));
typedef GAS unsigned gu32;

constexpr int NR = 8, MM = 4096, KK = 4096, NN = 16384, KS = KK / NR;
constexpr int NPAN = MM / 256, NBLK = NN / 256;
constexpr int DB = (NBLK - NRB) / NR;
constexpr int NLB = DB + NRB;
constexpr int NIN = NPAN + NLB;
constexpr int NTILE = NPAN * NLB;
constexpr int NWG = 256, NCOMP = NWG - NPULL;
constexpr int NSEG = NIN * 4, SCH = 64 / CH, NCH_IN = NSEG * SCH, YCH = 128 * DB / CH, NCH = NCH_IN + NPAN * YCH;
constexpr int NAV1 = 32;
constexpr int NSTASK = (NPAN + NBLK) * 4 - NPRE + NPRE * 8;
constexpr int MAXT = 4;
static_assert(NRB % 8 == 0 && DB >= 1 && NSTASK <= 2 * NCOMP && NTILE <= MAXT * NCOMP && NIN <= 64 && NPRE <= NPAN, "config");
constexpr unsigned PART_B = 64u << 10;
constexpr unsigned SEG_B = NR * PART_B, IMG_B = 4 * SEG_B;
constexpr unsigned CTRL_B = 1u << 20;
constexpr unsigned SCR_OFF = CTRL_B + 2u * NIN * IMG_B;
constexpr size_t WS_NEED = (size_t)SCR_OFF + (size_t)NR * DB * 4 * PART_B;
constexpr unsigned OUT_B = (unsigned)((size_t)MM * NN * 2);
constexpr int LINE = 32;
constexpr int C_CALL = 0;
constexpr int C_CNT = C_CALL + NWG * LINE;
constexpr int C_AV = C_CNT + 2 * 256 * LINE;
constexpr int C_TMO = C_AV + 2 * 8 * 64;
constexpr int C_CLAIM = C_TMO + LINE;
static_assert((C_CLAIM + 8 * LINE) * 4 <= (512 << 10) && NAV1 + 1 + NPAN <= 64, "ctrl");
constexpr unsigned long long DEADLINE_TICKS = 50000000ull;
constexpr int HTB = 16384, STAGE_BYTES = 8 * HTB, LDS_BYTES = STAGE_BYTES + 1024;

struct Params {
    const char* x; const char* w;
    char* out_self; char* ws_self;
    char* buf[2 * NR];
    int rank;
    unsigned seg4[256 / 4];
    unsigned segpos4[256 / 4];
    unsigned stask2[NWG];
    unsigned tile2[NCOMP * MAXT / 2 + 1];
};
typedef const __attribute__((address_space(4))) Params* KP;
__device__ __forceinline__ int t_seg(KP p, int i) { return (p->seg4[i >> 2] >> ((i & 3) * 8)) & 255; }
__device__ __forceinline__ int t_segpos(KP p, int i) { return (p->segpos4[i >> 2] >> ((i & 3) * 8)) & 255; }
__device__ __forceinline__ int t_stask(KP p, int i) { return (p->stask2[i >> 1] >> ((i & 1) * 16)) & 65535; }
__device__ __forceinline__ int t_tile(KP p, int i) { return (p->tile2[i >> 1] >> ((i & 1) * 16)) & 65535; }

#define RLX_AGENT __ATOMIC_RELAXED, __HIP_MEMORY_SCOPE_AGENT
#define RLX_SYS __ATOMIC_RELAXED, __HIP_MEMORY_SCOPE_SYSTEM
#define WAIT_V(n) asm volatile("s_waitcnt vmcnt(" #n ")" ::: "memory")
#define WAIT_L(n) asm volatile("s_waitcnt lgkmcnt(" #n ")" ::: "memory")
#define WAIT_VC(n) do { asm volatile("" ::: "memory"); __builtin_amdgcn_s_waitcnt(0x0F70 | ((n) & 15) | (((n) >> 4) << 14)); asm volatile("" ::: "memory"); } while (0)
#define BAR __builtin_amdgcn_s_barrier()
#define SCHED __builtin_amdgcn_sched_barrier(0)

#define STAMP(slot) do { } while (0)
#define STAMPC(slot) do { } while (0)
typedef GAS char gchar;
typedef const GAS u32x4 gvec;
__device__ __forceinline__ gchar* uni(const void* p) {
    const unsigned long long v = (unsigned long long)p;
    const unsigned lo = __builtin_amdgcn_readfirstlane((unsigned)v), hi = __builtin_amdgcn_readfirstlane((unsigned)(v >> 32));
    return (gchar*)(((unsigned long long)hi << 32) | lo);
}
struct Self { gchar* out; gchar* ws; int rank; unsigned call; };
__device__ __forceinline__ int cnt_idx(unsigned par, int sg) { return C_CNT + ((int)par * 256 + sg) * LINE; }
__device__ __forceinline__ int av_idx(unsigned par, int source, int slot) { return C_AV + ((int)par * 8 + source) * 64 + slot; }
__device__ __forceinline__ int av_slot(int pos) { return pos - NPRE < NAV1 ? pos - NPRE : NAV1; }
__device__ __forceinline__ unsigned seg_off(unsigned par, unsigned sg) { return CTRL_B + (par * NSEG + sg) * SEG_B; }
template <class P> __device__ __forceinline__ __amdgpu_buffer_rsrc_t mkrsrc(P p, unsigned bytes) {
    return __builtin_amdgcn_make_buffer_rsrc((void*)p, 0, bytes, 0x00020000);
}
#define ST128(v, rs, vo, so, aux) __builtin_amdgcn_raw_buffer_store_b128(v, rs, (vo) + (so), 0, aux)
__device__ __forceinline__ unsigned cvt_pk_bf16(float lo, float hi) { unsigned r; asm("v_cvt_pk_bf16_f32 %0, %1, %2" : "=v"(r) : "v"(lo), "v"(hi)); return r; }
typedef float f32x2 __attribute__((ext_vector_type(2)));
__device__ __forceinline__ unsigned silu2(float y0, float y1, float one) {
    const f32x2 c = (f32x2){__builtin_amdgcn_fmed3f(y0, -60.0f, 60.0f), __builtin_amdgcn_fmed3f(y1, -60.0f, 60.0f)} * -1.44269504089f;
    const f32x2 d = (f32x2){__builtin_amdgcn_exp2f(c.x), __builtin_amdgcn_exp2f(c.y)} + one;
    const f32x2 o = (f32x2){y0, y1} * (f32x2){__builtin_amdgcn_rcpf(d.x), __builtin_amdgcn_rcpf(d.y)};
    return cvt_pk_bf16(o.x, o.y);
}
__device__ __forceinline__ int gblock(int rank, int jb) { return jb < DB ? rank * DB + jb : NR * DB + (jb - DB); }

__device__ __forceinline__ bool timed_out(gu32* tmo, unsigned call, unsigned long long t0) {
    if ((unsigned)__builtin_amdgcn_readfirstlane(__hip_atomic_load(tmo, RLX_AGENT)) == call ||
        __builtin_amdgcn_s_memrealtime() - t0 > DEADLINE_TICKS) {
        __hip_atomic_store(tmo, call, RLX_AGENT);
        return true;
    }
    __builtin_amdgcn_s_sleep(4);
    return false;
}

__device__ __forceinline__ int chunk_slot(int c) { return c < NCH_IN ? av_slot(c / SCH) : NAV1 + 1 + (c - NCH_IN) / YCH; }
struct Chunk { unsigned sso, dso, yo, cnt; bool isy; };
__device__ __forceinline__ Chunk chunk_decode(KP p, int c, unsigned par, int rank, int src) {
    Chunk d;
    d.isy = c >= NCH_IN;
    const int sg = d.isy ? 0 : t_seg(p, c / SCH), cy = (c < NCH ? c : NCH - 1) - NCH_IN;
    const unsigned po = (unsigned)(c % SCH) * (CH * 1024u);
    d.yo = (unsigned)(cy % YCH) * (CH * 1024u);
    d.dso = d.isy ? (unsigned)(cy / YCH) * (256u * NN * 2) + (unsigned)(src * DB) * 512u : seg_off(par, sg) + src * PART_B + po;
    d.sso = (unsigned)(sg - NPAN * 4) < (unsigned)(DB * 4) ? SCR_OFF + (unsigned)(rank * DB * 4 + sg - NPAN * 4) * PART_B + po : d.dso;
    d.cnt = d.isy ? CTRL_B : (unsigned)cnt_idx(par, sg) * 4u;
    return d;
}
constexpr unsigned TA = (NCH_IN - NPRE * SCH) / MA, TB = TA + (NCH - NCH_IN - NTAIL) / MB, NTICK = TB + NTAIL;
__device__ __forceinline__ int ticket_chunk(unsigned t, int& m) {
    static_assert(SCH % MA == 0 && YCH % MB == 0 && NTAIL % MB == 0, "tickets");
    m = t < TA ? MA : t < TB ? MB : 1;
    return t < TA ? NPRE * SCH + (int)t * MA : t < TB ? NCH_IN + (int)(t - TA) * MB : t - TB < (unsigned)NTAIL ? NCH - NTAIL + (int)(t - TB) : NCH;
}
__device__ __forceinline__ unsigned piece_voff(bool isy, unsigned yo, int k, int lane) {
    const unsigned l = (unsigned)k * 1024u + (unsigned)lane * 16u, o = yo + l;
    return isy ? (o / (DB * 512u)) * (NN * 2) + o % (DB * 512u) : l;
}
__device__ __forceinline__ void puller(KP p, const Self me) {
    const int tid = threadIdx.x, lane = tid & 63, wid = __builtin_amdgcn_readfirstlane(tid >> 6), rank = me.rank;
    const unsigned call = me.call, par = call & 1u;
    const unsigned want = ((call + 1u) >> 1) * DB * (lane == NAV1 ? NSEG - NPRE - NAV1 : 1);
    const int gw = blockIdx.x * 8 + wid, j = gw % (NR - 1), src = (rank + 1 + j) & (NR - 1);
    gu32* ctl = (gu32*)me.ws;
    gu32* tmo = ctl + C_TMO;
    const unsigned long long s_ws = (unsigned long long)p->buf[src], s_out = (unsigned long long)p->buf[NR + src];
    const unsigned long long d_ws = (unsigned long long)me.ws, d_out = (unsigned long long)me.out;
    const __amdgpu_buffer_rsrc_t rc = mkrsrc(me.ws, CTRL_B);
    const int one = lane == 0;
    const unsigned head = lane == 0 ? (unsigned)(C_CLAIM + j * LINE) * 4u : CTRL_B;
#define COUNT_IN() __builtin_amdgcn_raw_ptr_buffer_atomic_add_i32(prev < CTRL_B ? one * (CH * MA) : 0, rc, lane == 0 ? prev : CTRL_B, 0, 16)
    constexpr int R = PD == 1 ? 2 : 4;
    static_assert(CH % R == 0 && CH >= 2 * PD + 2, "ring");
    bool bad = false, primed = false;
    int sn = 0;
    unsigned prev = CTRL_B;
    int rem, c = ticket_chunk(gw / (NR - 1), rem);
    if (gw >= PWL * (NR - 1)) return;
    Chunk cur;
    u32x4 v[R];
    unsigned vo[R];
#define PLOAD(d, o, on) __builtin_amdgcn_raw_buffer_load_b128(mkrsrc((d).isy ? s_out : s_ws, !(on) ? 0u : (d).isy ? OUT_B : (unsigned)WS_NEED), o, (d).sso, 17)
#pragma unroll 1
    while (c < NCH) {
        if (!primed) {
            const int slot = chunk_slot(c);
            WAIT_VC(0);
            COUNT_IN();
            prev = CTRL_B;
            const unsigned long long t0 = __builtin_amdgcn_s_memrealtime();
            for (;;) {
                if ((__ballot(__hip_atomic_load(ctl + av_idx(par, src, lane), RLX_SYS) >= want) >> slot) & 1ull) break;
                if (timed_out(tmo, call, t0)) { bad = true; break; }
            }
            asm volatile("" ::: "memory");
            cur = chunk_decode(p, c, par, rank, src);
#pragma unroll
            for (int k = 0; k < PD; ++k) {
                vo[k] = piece_voff(cur.isy, cur.yo, k, lane);
                v[k] = PLOAD(cur, vo[k], !bad);
                asm volatile("" ::: "memory");
                __builtin_amdgcn_raw_ptr_buffer_atomic_add_i32(0, rc, CTRL_B + lane * 4u, 0, 0);
                asm volatile("" ::: "memory");
            }
        }
        const int cv = __builtin_amdgcn_raw_ptr_buffer_atomic_add_i32(rem == 1 ? one : 0, rc, rem == 1 ? head : CTRL_B, 0, 0);
        const unsigned av = __builtin_amdgcn_raw_buffer_load_b32(rc, rem == 1 ? (unsigned)av_idx(par, src, lane) * 4u : CTRL_B, 0, 17);
        const __amdgpu_buffer_rsrc_t rd = mkrsrc(cur.isy ? d_out : d_ws, cur.isy ? OUT_B : (unsigned)WS_NEED);
        Chunk nx = cur;
        int cn = 0, remn = 0;
#pragma unroll
        for (int k = 0; k < CH; ++k) {
            if (k == CH - PD - 1) {
                const unsigned tk = (unsigned)__builtin_amdgcn_readfirstlane(cv) - (call - 1u) * NTICK;
                cn = ticket_chunk(tk < NTICK ? PWL + tk : NTICK, remn);
                if (rem > 1) { cn = c + 1; remn = rem - 1; }
                nx = chunk_decode(p, cn, par, rank, src);
                primed = cn < NCH && (bad || rem > 1 || ((__ballot(av >= want) >> chunk_slot(cn)) & 1ull));
            }
            vo[(k + PD) % R] = k + PD < CH ? piece_voff(cur.isy, cur.yo, k + PD, lane) : piece_voff(nx.isy, nx.yo, k + PD - CH, lane);
            asm volatile("" : "+v"(vo[(k + PD) % R]));
            asm volatile("" : "+v"(v[k % R]) :: "memory");
            if (k + PD < CH) v[(k + PD) % R] = PLOAD(cur, vo[(k + PD) % R], !bad);
            else v[(k + PD) % R] = PLOAD(nx, vo[(k + PD) % R], primed && !bad);
            asm volatile("" ::: "memory");
            if (bad) v[k % R] = (u32x4){0x7fc07fc0u, 0x7fc07fc0u, 0x7fc07fc0u, 0x7fc07fc0u};
            ST128(v[k % R], rd, vo[k % R], cur.dso, 16);
            if (k == PD) COUNT_IN();
            asm volatile("" ::: "memory");
        }
        prev = rem == 1 ? cur.cnt : CTRL_B; cur = nx; c = cn; rem = remn;
        STAMP(2 + (sn < 238 ? sn : 238));
        ++sn;
    }
    WAIT_VC(0);
    COUNT_IN();
    STAMP(250);
}

__device__ __forceinline__ unsigned bpos(unsigned fq, unsigned q, unsigned r) { return r * 256u + (((fq * 4u + q) ^ ((r & 1u) | ((r & 2u) << 2))) * 16u); }

__device__ __forceinline__ void stage(KP p, const Self me, unsigned& craw) {
    const int tid = threadIdx.x, lane = tid & 63, wid = __builtin_amdgcn_readfirstlane(tid >> 6), rank = me.rank;
    const int fq = lane >> 4, c = lane & 15;
    const __amdgpu_buffer_rsrc_t rs = mkrsrc(me.ws, (unsigned)WS_NEED);
#define GET_PAR() asm volatile("" : "+v"(craw) :: "memory"); const unsigned par = (__builtin_amdgcn_readfirstlane(craw) + 1u) & 1u
#pragma unroll 1
    for (int rnd = 0; rnd < 2; ++rnd) {
        const int code = t_stask(p, blockIdx.x * 2 + rnd);
        if (code == 0xffff) break;
        const int un = (code & 0x3fff) >> 2, qt = code & 3;
        int sg, dest = -1;
        u32x4 v[8];
        if (code & 0x4000) {
            const unsigned long long t0 = __builtin_amdgcn_s_memrealtime();
            while (__builtin_amdgcn_s_memrealtime() - t0 < STDELAY) __builtin_amdgcn_s_sleep(8);
        }
        if (code & 0x8000) {
            const int g = (code & 0x3fff) * 8 + wid, pc = g & 63;
            sg = t_seg(p, g >> 6);
            v[0] = *(gvec*)((const gchar*)p->x + ((size_t)((sg >> 2) * 256 + ((pc >> 1) & 15) * 16 + c) * KS + (sg & 3) * 128 + (pc >> 5) * 64 + (pc & 1) * 32 + fq * 8) * 2);
            GET_PAR();
            const unsigned base = seg_off(par, sg) + rank * PART_B + (unsigned)pc * 1024u;
#pragma unroll
            for (int d = 0; d < NR; ++d) ST128(v[0], mkrsrc(p->buf[(rank + 1 + g + d) & 7], (unsigned)WS_NEED), (unsigned)lane * 16u, base, 17);
        } else if (un < NPAN) {
            sg = un * 4 + qt;
            const gchar* src = (const gchar*)p->x + ((size_t)(un * 256 + (wid & 3) * 64 + c) * KS + qt * 128 + (wid >> 2) * 64 + fq * 8) * 2;
#pragma unroll
            for (int j = 0; j < 8; ++j) v[j] = *(gvec*)(src + (j >> 1) * (16 * KS * 2) + (j & 1) * 64);
            GET_PAR();
            const unsigned base = seg_off(par, sg) + rank * PART_B + (unsigned)wid * 8192u;
#pragma unroll
            for (int j = 0; j < 8; ++j) ST128(v[j], rs, (unsigned)lane * 16u + j * 1024u, base, 17);
        } else {
            const int B = un - NPAN, kt = wid >> 2, ks = (wid >> 1) & 1, g = wid & 1;
            const gchar* src = (const gchar*)p->w + ((size_t)(qt * 128 + kt * 64 + ks * 32 + fq * 8) * NN + B * 256 + g * 128 + c * 8) * 2;
#pragma unroll
            for (int j = 0; j < 8; ++j) v[j] = *(gvec*)(src + (size_t)j * NN * 2);
            GET_PAR();
            if (B < NR * DB) { dest = B / DB; sg = (NPAN + B % DB) * 4 + qt; } else sg = (NPAN + DB + (B - NR * DB)) * 4 + qt;
            const unsigned base = (dest >= 0 && dest != rank ? SCR_OFF + (unsigned)(B * 4 + qt) * PART_B : seg_off(par, sg) + rank * PART_B)
                                  + (unsigned)kt * 32768u + (unsigned)ks * 1024u;
            const unsigned lb = (unsigned)((c >> 2) & 1) * HTB + (unsigned)(2 * g + (c >> 3)) * 4096u;
#pragma unroll
            for (int e = 0; e < 8; ++e) {
                u32x4 o;
#pragma unroll
                for (int d = 0; d < 4; ++d)
                    o[d] = __builtin_amdgcn_perm(v[2 * d + 1][e >> 1], v[2 * d][e >> 1], (e & 1) ? 0x07060302u : 0x05040100u);
                ST128(o, rs, lb + (unsigned)(e >> 2) * 2048u + bpos(fq, c & 3, e & 3), base, 17);
            }
        }
        WAIT_V(0);
        __syncthreads();
        if (tid < NR && (dest < 0 || dest == tid)) {
            const unsigned par = (craw + 1u) & 1u;
            const bool landed = tid == rank || (code & 0x8000);
            __hip_atomic_fetch_add((gu32*)p->buf[tid] + (landed ? cnt_idx(par, sg) : av_idx(par, rank, av_slot(t_segpos(p, sg)))),
                                   (code & 0x8000) ? 8u : landed ? 64u : (unsigned)DB, RLX_SYS);
        }
    }
}

__device__ __forceinline__ void compute(KP p, const Self me, LAS unsigned char* lds) {
    const unsigned call = me.call;
    const int tid = threadIdx.x, wid = __builtin_amdgcn_readfirstlane(tid >> 6), lane = tid & 63, wr = wid >> 2, wc = wid & 3, fr = lane & 15, fq = lane >> 4;
    const int rank = me.rank, cwg = blockIdx.x - NPULL;
    const unsigned par = call & 1u, want = ((call + 1u) >> 1) * 512u;
    gu32* peer_av = (gu32*)p->buf[lane & 7] + av_idx(par, rank, NAV1 + 1);
    gu32* ctl = (gu32*)me.ws;
    LAS unsigned* badf = (LAS unsigned*)(lds + STAGE_BYTES);
    const __amdgpu_buffer_rsrc_t rw = mkrsrc(me.ws, (unsigned)WS_NEED);
    const unsigned img = seg_off(par, 0);
    const unsigned voff = (unsigned)tid * 16u, ldsw = (unsigned)wid * 1024u;
    const int aoff = wr * 8192 + lane * 16, boff = wc * 4096 + (int)bpos(fq, fr >> 2, fr & 3);
    constexpr unsigned kstep = 2 * HTB, hstep = HTB;
#define SA(b, h) (((b) * 2 + (h)) * HTB)
#define SB(b, h) ((4 + (b) * 2 + (h)) * HTB)
#define STAGE(bufoff, gbase) do { _Pragma("unroll") for (int _i = 0; _i < 2; ++_i) \
        __builtin_amdgcn_raw_ptr_buffer_load_lds(rw, (LAS void*)(lds + (bufoff) + ldsw + _i * 8192), 16, voff, (gbase) + _i * 8192, 0, 0); } while (0)
#define LDA(dst, b, h) do { _Pragma("unroll") for (int m = 0; m < 4; ++m) _Pragma("unroll") for (int k = 0; k < 2; ++k) dst[m][k] = *(const LAS bf16x8*)(lds + SA(b, h) + aoff + m * 2048 + k * 1024); } while (0)
#define LDB(dst, b, h) do { _Pragma("unroll") for (int n = 0; n < 2; ++n) _Pragma("unroll") for (int k = 0; k < 2; ++k) dst[n][k] = *(const LAS bf16x8*)(ldsB + (SB(b, h) - SB(0, 0)) + n * 2048 + k * 1024); } while (0)
#define MMA(ai, bj, At, Bt) do { __builtin_amdgcn_s_setprio(1); _Pragma("unroll") for (int m = 0; m < 4; ++m) _Pragma("unroll") for (int n = 0; n < 2; ++n) _Pragma("unroll") for (int k = 0; k < 2; ++k) \
        acc[ai][bj][m][n] = __builtin_amdgcn_mfma_f32_16x16x32_bf16(Bt[n][k], At[m][k], acc[ai][bj][m][n], 0, 0, 0); __builtin_amdgcn_s_setprio(0); } while (0)
#define POLL_IDX(tl) cnt_idx(par, ((lane & 4) ? NPAN + ((tl) >> 4) : ((tl) & 15)) * 4 + (lane & 3))
#define POLL_WORD(tl) (ctl + POLL_IDX(tl))
#define OKQ(pv) ({ const unsigned m_ = (unsigned)__ballot((pv) >= want); __builtin_ctz(~(m_ & (m_ >> 4) & 15u)); })
#define BLOCK_FOR(tl, need) do { const unsigned long long t0_ = __builtin_amdgcn_s_memrealtime(); \
        while ((okq = OKQ(__hip_atomic_load(POLL_WORD(tl), RLX_SYS))) <= (need)) \
            if (timed_out(ctl + C_TMO, call, t0_)) { if (lane == 0) *badf = 1u; okq = 4; break; } \
        __builtin_amdgcn_fence(__ATOMIC_ACQUIRE, ""); \
        asm volatile("s_waitcnt vmcnt(0) lgkmcnt(0)" ::: "memory"); } while (0)
    LAS unsigned* gof = badf + 1;
    LAS unsigned* prb = badf + 64;
    LAS unsigned char* ldsB = lds + SB(0, 0) + boff;
    asm volatile("" : "+v"(ldsB));
    constexpr int NIT = KK / 128;
    const unsigned ev8 = (1u << (NIT - 2)) | (wid == 0 ? 0x20202021u : 0u), ev4 = wid == 0 ? 0x40404040u : 0u;
    int q = cwg * MAXT, sn = 0, okq = 0;
    f32x4 acc[2][2][4][2];
    bf16x8 At[4][2], B0[2][2], B1[2][2];
#pragma unroll 1
    while (q < (cwg + 1) * MAXT && t_tile(p, q) != 0xffff) {
        int cur = t_tile(p, q), pend = -1;
        unsigned a2 = img + (unsigned)(cur & 15) * IMG_B, b2 = img + (unsigned)(NPAN + (cur >> 4)) * IMG_B;
        STAMP(2 + 4 * sn);
        if (wid == 0) BLOCK_FOR(cur, 0);
        asm volatile("" ::: "memory"); BAR; asm volatile("" ::: "memory");
        STAGE(SB(0, 0), b2); STAGE(SA(0, 0), a2); STAGE(SB(0, 1), b2 + hstep); STAGE(SA(0, 1), a2 + hstep);
        if (wr == 1) BAR;
        WAIT_V(4); BAR;
        STAGE(SB(1, 0), b2 + kstep); STAGE(SA(1, 0), a2 + kstep); STAGE(SB(1, 1), b2 + hstep + kstep);
        unsigned a1h = a2 + kstep + hstep;
        a2 += 2 * kstep; b2 += 2 * kstep;
        WAIT_V(6); BAR;
        for (;;) {
            STAMP(3 + 4 * sn); STAMPC(100 + 2 * sn);
#pragma unroll
            for (int a = 0; a < 2; ++a)
#pragma unroll
                for (int b = 0; b < 2; ++b)
#pragma unroll
                    for (int m = 0; m < 4; ++m)
#pragma unroll
                        for (int n = 0; n < 2; ++n) acc[a][b][m][n] = (f32x4){0.f, 0.f, 0.f, 0.f};
            const bool has_next = q + 1 < (cwg + 1) * MAXT && t_tile(p, q + 1) != 0xffff;
            const int nxt = has_next ? t_tile(p, q + 1) : cur;
            bool go = false;
#pragma unroll 1
            for (int it = 0; it < NIT; ++it) {
                LDB(B0, 0, 0); SCHED; LDA(At, 0, 0); STAGE(SA(1, 1), a1h);
                WAIT_L(8); BAR; WAIT_L(0); MMA(0, 0, At, B0); BAR; SCHED;
                LDB(B1, 0, 1); STAGE(SB(0, 0), b2);
                BAR; WAIT_L(0); MMA(0, 1, At, B1); BAR;
                LDA(At, 0, 1); STAGE(SA(0, 0), a2);
                BAR; WAIT_L(0); MMA(1, 0, At, B0); BAR; SCHED;
                STAGE(SB(0, 1), b2 + hstep);
                WAIT_V(6);
                if (__builtin_expect((ev4 >> it) & 1u, 0)) {
                    const int need = (it + 2) >> 3;
                    if (okq <= need) {
                        asm volatile("buffer_inv sc0 sc1" ::: "memory");
                        okq = OKQ(prb[lane]);
                        if (need == 4) { if (lane == 0) *gof = okq; }
                        else if (okq <= need) BLOCK_FOR(cur, need);
                    }
                }
                BAR; MMA(1, 1, At, B1); BAR;
                LDB(B0, 1, 0); SCHED; LDA(At, 1, 0); STAGE(SA(0, 1), a2 + hstep);
                WAIT_L(8); BAR; WAIT_L(0); MMA(0, 0, At, B0); BAR; SCHED;
                LDB(B1, 1, 1); STAGE(SB(1, 0), b2 + kstep);
                BAR; WAIT_L(0); MMA(0, 1, At, B1); BAR;
                LDA(At, 1, 1); STAGE(SA(1, 0), a2 + kstep);
                BAR; WAIT_L(0); MMA(1, 0, At, B0); BAR; SCHED;
                STAGE(SB(1, 1), b2 + kstep + hstep);
                a1h = a2 + kstep + hstep;
                a2 += 2 * kstep; b2 += 2 * kstep;
                WAIT_V(6);
                if (__builtin_expect((ev8 >> it) & 1u, 0)) {
                    if (it == NIT - 2) {
                        go = has_next && __builtin_amdgcn_readfirstlane(*gof) != 0u;
                        const int tl = go ? nxt : cur;
                        a2 = img + (unsigned)(tl & 15) * IMG_B; b2 = img + (unsigned)(NPAN + (tl >> 4)) * IMG_B;
                    } else if (it == 0) {
                        if (pend >= 0 && lane < NR && lane != rank) __hip_atomic_fetch_add(peer_av + pend, 1u, RLX_SYS);
                    } else if (okq <= (it + 3) >> 3) {
                        const int tl = it == NIT - 3 ? nxt : cur;
                        __builtin_amdgcn_raw_ptr_buffer_load_lds(rw, (LAS void*)prb, 4, (unsigned)POLL_IDX(tl) * 4u, 0, 0, 17);
                    }
                }
                BAR; MMA(1, 1, At, B1); BAR;
            }
            pend = -1;
            {
                STAMP(4 + 4 * sn); STAMPC(101 + 2 * sn);
                const float one = *badf != 0u ? __builtin_nanf("") : 1.0f;
                const __amdgpu_buffer_rsrc_t ro = mkrsrc(me.out, OUT_B);
                const unsigned so = (unsigned)((cur & 15) * 256 + wr * 64) * (NN * 2) + (unsigned)gblock(rank, cur >> 4) * 512u + (unsigned)wc * 128u;
                const unsigned vo = (unsigned)fr * (NN * 2) + (unsigned)fq * 16u;
#pragma unroll
                for (int ai = 0; ai < 2; ++ai)
#pragma unroll
                    for (int m = 0; m < 4; ++m)
#pragma unroll
                        for (int bj = 0; bj < 2; ++bj) {
                            const f32x4 v0 = acc[ai][bj][m][0], v1 = acc[ai][bj][m][1];
                            u32x4 w;
                            w.x = silu2(v0[0], v0[1], one); w.y = silu2(v0[2], v0[3], one);
                            w.z = silu2(v1[0], v1[1], one); w.w = silu2(v1[2], v1[3], one);
                            ST128(w, ro, vo + bj * 64, so + (unsigned)(ai * 128 + m * 16) * (NN * 2), 17);
                        }
                if ((cur >> 4) < DB) pend = cur & 15;
                STAMP(5 + 4 * sn);
                ++sn;
            }
            ++q;
            if (!go) break;
            cur = nxt;
        }
        WAIT_V(0);
        if (wr == 0) BAR;
        BAR;
        if (pend >= 0 && tid < NR && tid != rank) __hip_atomic_fetch_add(peer_av + pend, 1u, RLX_SYS);
    }
    STAMP(250);
}

__global__ __launch_bounds__(512, 2) void fused_gemm_silu(const Params params) {
    KP p = (KP)__builtin_amdgcn_kernarg_segment_ptr();
    extern __shared__ __attribute__((aligned(16))) unsigned char shm[];
    LAS unsigned char* lds = (LAS unsigned char*)shm;
    LAS unsigned* ctlw = (LAS unsigned*)(lds + STAGE_BYTES);
    Self me;
    me.out = uni(p->out_self); me.ws = uni(p->ws_self); me.rank = __builtin_amdgcn_readfirstlane(p->rank);
    STAMP(0);
    gu32* cc = (gu32*)me.ws + C_CALL + blockIdx.x * LINE;
    unsigned craw = __hip_atomic_load(cc, RLX_AGENT);
    if (threadIdx.x < 16) ctlw[threadIdx.x] = 0u;
    stage(p, me, craw);
    me.call = __builtin_amdgcn_readfirstlane(craw) + 1u;
    __syncthreads();
    if (threadIdx.x == 0) __hip_atomic_store(cc, me.call, RLX_AGENT);
    STAMP(1);
    if (blockIdx.x < NPULL) puller(p, me);
    else compute(p, me, lds);
}

void build_tables(Params& pp) {
    unsigned char *seg = (unsigned char*)pp.seg4, *segpos = (unsigned char*)pp.segpos4;
    unsigned short *tile = (unsigned short*)pp.tile2, *stask = (unsigned short*)pp.stask2;
    for (int i = 0; i < 256; ++i) seg[i] = segpos[i] = 0;
    int n = 0;
    for (int w0 = 0, r = 1; w0 < NLB; ++r) {
        int w1 = (r * NCOMP + NPAN - 1) / NPAN;
        if (w1 > NLB) w1 = NLB;
        for (int qt = 0; qt < 4; ++qt)
            for (int nx = w0 ? NPAN : 0, nw = w0; nx < NPAN || nw < w1;) {
                if (nx < NPAN && (nx < NPRE || nx <= nw || nw == w1)) seg[n++] = nx++ * 4 + qt;
                else seg[n++] = (NPAN + nw++) * 4 + qt;
            }
        w0 = w1;
    }
    unsigned short list[NSTASK];
    int ns = 0, nurg = 0;
    for (int i = 0; i < NSEG; ++i) {
        const int u = seg[i] >> 2, qt = seg[i] & 3;
        segpos[seg[i]] = i;
        if (i == NPRE + NURG) nurg = ns;
        if (i < NPRE) continue;
        if (u < NPAN) list[ns++] = u * 4 + qt;
        else if (u >= NPAN + DB) list[ns++] = (NPAN + NR * DB + (u - NPAN - DB)) * 4 + qt;
        else for (int d = 0; d < NR; ++d) list[ns++] = (NPAN + d * DB + (u - NPAN)) * 4 + qt;
    }
    for (int i = 0; i < 2 * NWG; ++i) stask[i] = 0xffff;
    int li = 0, pi = 0;
    for (int b = 0; b < NWG; ++b) if (!(b & 4) && li < nurg) stask[2 * b] = list[li++];
    for (int b = NPULL; b < NWG; ++b) if ((b & 4) && pi < NPRE * 8) stask[2 * b] = 0x8000 | pi++;
    for (int b = NPULL; b < NWG; ++b) if (stask[2 * b] == 0xffff && li < ns) stask[2 * b] = 0x4000 | list[li++];
    for (int b = NWG - (ns - li); b < NWG; ++b) stask[2 * b + 1] = list[li++];
    unsigned short ql[8][NTILE];
    float qe[8][NTILE], freeat[NCOMP];
    int qn[8] = {0, 0, 0, 0, 0, 0, 0, 0}, qh[8] = {0, 0, 0, 0, 0, 0, 0, 0}, cnt[NCOMP], left = NTILE;
    auto landed = [&](int tl, int qt) {
        const int pa = segpos[(tl & 15) * 4 + qt], pb = segpos[(NPAN + (tl >> 4)) * 4 + qt];
        return 12.f + 1.09f * (pa > pb ? pa : pb);
    };
    for (int pos = 0; pos < NSEG; ++pos)
        for (int jb = 0; jb < NLB; ++jb)
            for (int i = 0; i < NPAN; ++i) {
                const int key = segpos[i * 4] > segpos[(NPAN + jb) * 4] ? segpos[i * 4] : segpos[(NPAN + jb) * 4];
                if (key != pos) continue;
                const int x = (i & 3) + 4 * (jb & 1);
                ql[x][qn[x]] = (unsigned short)(i | jb << 4); qe[x][qn[x]++] = landed(i | jb << 4, 0);
            }
    for (int w = 0; w < NCOMP; ++w) { freeat[w] = 0.f; cnt[w] = 0; }
    for (int i = 0; i < NCOMP * MAXT + 2; ++i) tile[i] = 0xffff;
    for (float t = 0.f; left > 0; t += 0.5f)
        for (int steal = 0; steal < 2; ++steal)
            for (int pass = 0; pass <= MAXT; ++pass)
                for (int w = 0; w < NCOMP; ++w) {
                    if (cnt[w] != pass || cnt[w] >= MAXT || freeat[w] > t) continue;
                    int x = (w + NPULL) % 8;
                    if (steal) {
                        if (qh[x] < qn[x] && qe[x][qh[x]] <= t + XSLACK) continue;
                        int bx = -1;
                        for (int y = 0; y < 8; ++y) {
                            if (qh[y] >= qn[y] || qe[y][qh[y]] > t) continue;
                            float nf = 1e9f;
                            for (int v = 0; v < NCOMP; ++v) if ((v + NPULL) % 8 == y && cnt[v] < MAXT && freeat[v] < nf) nf = freeat[v];
                            if (nf > t + XSLACK && (bx < 0 || qn[y] - qh[y] > qn[bx] - qh[bx])) bx = y;
                        }
                        if (bx < 0) continue;
                        x = bx;
                    } else if (qh[x] >= qn[x] || qe[x][qh[x]] > t) continue;
                    const int tl = ql[x][qh[x]++];
                    tile[w * MAXT + cnt[w]++] = (unsigned short)tl;
                    float f = t;
                    for (int qt = 0; qt < 4; ++qt) { const float e = landed(tl, qt); f = (f > e ? f : e) + TM1 / 4; }
                    freeat[w] = f;
                    --left;
                }
}
}

extern "C" void kernel_launch(int rank, int nranks, ncclComm_t comm, hipStream_t stream,
                              void* const* d_in, const int* in_sizes, int n_in,
                              void* d_out, int out_size, const int* in_dtypes, int out_dtype,
                              void* const* d_out_all, void* const* d_ws_all, size_t ws_bytes) {
    if (nranks != NR || (unsigned)rank >= (unsigned)NR || n_in < 2 || in_sizes[0] != MM * KS || in_sizes[1] != KS * NN ||
        out_size != MM * NN || ws_bytes < WS_NEED) {
        fprintf(stderr, "unexpected problem shape\n");
        return;
    }
    static bool attr_set = false;
    if (!attr_set) {
        HIP_CHECK(hipFuncSetAttribute((const void*)fused_gemm_silu, hipFuncAttributeMaxDynamicSharedMemorySize, LDS_BYTES));
        attr_set = true;
    }
    Params p;
    build_tables(p);
    p.x = (const char*)d_in[0]; p.w = (const char*)d_in[1]; p.rank = rank;
    p.out_self = (char*)d_out_all[rank]; p.ws_self = (char*)d_ws_all[rank];
    for (int r = 0; r < NR; ++r) { p.buf[NR + r] = (char*)d_out_all[r]; p.buf[r] = (char*)d_ws_all[r]; }
    hipLaunchKernelGGL(fused_gemm_silu, dim3(NWG), dim3(512), LDS_BYTES, stream, p);
    HIP_CHECK(hipGetLastError());
}
```
